```python
import numpy as np

kernel_name = "gpu_ag_gemm_m4096_k4096_n4096_w8a8_dequant"


ABSMAX_RTOL = 0.001

COMPUTE_DTYPE = "int8"
EPILOGUE = "dequant"
IN_QUANT = "int8"
OUT_QUANT = None
IN_DTYPES = {"x": "int8", "w_mat": "int8",
             "scale_x": "float32", "scale_w": "float32"}
WORLD_SIZE = 8

def setup_inputs():

    x = np.random.randint(-127, 128, size=(4096, 4096)).astype(np.int8)
    w_mat = np.random.randint(-127, 128, size=(4096, 4096)).astype(np.int8)
    scale_x = (0.02 * (1.0 + np.random.rand(1))).astype(np.float32)
    scale_w = (0.02 * (1.0 + np.random.rand(1))).astype(np.float32)
    return {"x": x, "w_mat": w_mat, "scale_x": scale_x, "scale_w": scale_w}

def reference(x, w_mat, scale_x, scale_w):
    acc = x.astype(np.float64) @ w_mat.astype(np.float64)
    y = (acc.astype(np.float32) * (scale_x * scale_w)).astype(np.float32)
    return y
```

```cpp
#include <hip/hip_runtime.h>
#include <rccl/rccl.h>
#include <cstdio>
#include <cstdint>

#define HIP_CHECK(expr) do { hipError_t _e = (expr); if (_e != hipSuccess) { \
    fprintf(stderr, "HIP error %s at %s:%d\n", hipGetErrorString(_e), __FILE__, __LINE__); \
    return; } } while (0)

typedef unsigned u32x4 __attribute__((ext_vector_type(4)));
typedef int i32x4 __attribute__((ext_vector_type(4)));
typedef int i32x2 __attribute__((ext_vector_type(2)));
typedef float f32x4 __attribute__((ext_vector_type(4)));
#define LDS_AS __attribute__((address_space(3)))
#define GLB_AS __attribute__((address_space(1)))

constexpr int NR = 8;
constexpr int KDIM = 4096;
constexpr int ML = 512;
constexpr int NL = 512;
constexpr int KC = 256;
constexpr int NCH = KDIM / KC;
constexpr int NSEND = (NR - 1) * (ML / 64);
constexpr int CW = 6;
constexpr int NRB = NR * ML / 16;
constexpr int NJ = (NRB + CW - 1) / CW;
constexpr int NGEMM = ((NJ + 7) / 8) * 32;
constexpr int GRID = NSEND + NGEMM;
constexpr int WPB = 8;
constexpr int NSLOT = 4;
constexpr int BSLOT = KC * 128;
constexpr int LDS_BYTES = NSLOT * BSLOT;
constexpr size_t WS_CNT = 0;
constexpr size_t WS_FLAG = 64 << 10;
constexpr size_t WS_TMO = 96 << 10;
constexpr size_t WS_DIAG = 128 << 10;
constexpr size_t WS_X = 1 << 20;
constexpr size_t XSHARD = (size_t)ML * KDIM;
constexpr size_t WS_NEED = WS_X + 2 * NR * XSHARD;
constexpr unsigned long long DEADLINE_TICKS = 30000000ull;

#define OUT_MODE 0
#define SLAG 2
constexpr int SD = 4;
constexpr int SSLOT = 8;

struct Params {
    const char* x; const char* w; const float* sx; const float* sw; float* out;
    char* ws[NR];
    int rank; int pad;
};

#define WAITV(n) asm volatile("s_waitcnt vmcnt(" #n ")" ::: "memory")
template <int N> __device__ __forceinline__ void waitv() { asm volatile("s_waitcnt vmcnt(%0)" :: "n"(N) : "memory"); }

#define STAMP(idx) do {} while (0)
#define DIAG_ARGS
#define DIAG_PASS

__global__ void __launch_bounds__(512, 2) ag_gemm(Params p) {
    extern __shared__ __attribute__((aligned(16))) char smem[];
    const int tid = threadIdx.x, lane = tid & 63;
    const int wave = __builtin_amdgcn_readfirstlane(tid >> 6);
    const int bid = blockIdx.x;
    char* myws = p.ws[p.rank];
    GLB_AS unsigned* cnt = (GLB_AS unsigned*)(myws + WS_CNT + (size_t)bid * 128);

    if (bid < NSEND) {
        const int grp = bid & 7, pi = bid >> 3;
        const unsigned peer = (unsigned)(p.rank + 1 + pi) % NR;
        if (wave == 7) return;
        if (wave == 4 || wave == 5) {
            const int h = wave - 4;
            const char* src = p.x + (size_t)(grp * 64 + h * 32 + (lane & 15)) * KDIM + (lane >> 4) * 16;
            auto stageX = [&](int c) {
                char* dstl = smem + (c & (SSLOT - 1)) * 16384 + h * 8192;
#pragma unroll
                for (int i = 0; i < 8; ++i)
                    __builtin_amdgcn_global_load_lds((const GLB_AS void*)(src + (size_t)(i >> 2) * 16 * KDIM + c * KC + (i & 3) * 64),
                                                     (LDS_AS void*)(dstl + i * 1024), 16, 0, 0);
            };
#pragma unroll
            for (int c = 0; c < SD; ++c) stageX(c);
#pragma unroll 1
            for (int c = 0; c < NCH - SD; ++c) {
                waitv<8 * (SD - 1)>();
                __builtin_amdgcn_s_barrier();
                stageX(c + SD);
            }
            static_assert(SD == 4, "tail below");
            WAITV(24); __builtin_amdgcn_s_barrier();
            WAITV(16); __builtin_amdgcn_s_barrier();
            WAITV(8); __builtin_amdgcn_s_barrier();
            WAITV(0); __builtin_amdgcn_s_barrier();
            return;
        }
        const unsigned call = __hip_atomic_load(cnt, __ATOMIC_RELAXED, __HIP_MEMORY_SCOPE_AGENT) + 1;
        if (wave == 6) {
            GLB_AS unsigned* flag = (GLB_AS unsigned*)(p.ws[peer] + WS_FLAG + (size_t)(p.rank * 8 + grp) * 128);
            const unsigned base = (call - 1) * NCH;
#pragma unroll 1
            for (int c = 0; c < NCH + SLAG; ++c) {
                __builtin_amdgcn_s_barrier();
                if (c >= SLAG && lane == 0) __hip_atomic_store(flag, base + (c - SLAG + 1), __ATOMIC_RELAXED, __HIP_MEMORY_SCOPE_SYSTEM);
                if (c >= SLAG) STAMP(2 + c - SLAG);
            }
            if (lane == 0) __hip_atomic_store(cnt, call, __ATOMIC_RELAXED, __HIP_MEMORY_SCOPE_AGENT);
            return;
        }
        char* dst = p.ws[peer] + WS_X + ((size_t)(call & 1) * NR + p.rank) * XSHARD + (size_t)grp * (64 * KDIM) + wave * 4096 + lane * 16;
        const char* lsrc = smem + wave * 4096 + lane * 16;
        STAMP(1);
#pragma unroll 1
        for (int c = 0; c < NCH; ++c) {
            waitv<4 * (SLAG - 1)>();
            __builtin_amdgcn_s_barrier();
            const char* l = lsrc + (c & (SSLOT - 1)) * 16384;
            const u32x4 v0 = *(const u32x4*)l, v1 = *(const u32x4*)(l + 1024), v2 = *(const u32x4*)(l + 2048), v3 = *(const u32x4*)(l + 3072);
            asm volatile("global_store_dwordx4 %0, %1, off sc0 sc1\n\t"
                         "global_store_dwordx4 %0, %2, off offset:1024 sc0 sc1\n\t"
                         "global_store_dwordx4 %0, %3, off offset:2048 sc0 sc1\n\t"
                         "global_store_dwordx4 %0, %4, off offset:3072 sc0 sc1\n\ts_nop 1"
                         :: "v"(dst + (size_t)c * 16384), "v"(v0), "v"(v1), "v"(v2), "v"(v3) : "memory");
            if (c == 0) STAMP(2);
        }
        if constexpr (SLAG >= 3) { waitv<8>(); __builtin_amdgcn_s_barrier(); }
        if constexpr (SLAG >= 2) { waitv<4>(); __builtin_amdgcn_s_barrier(); }
        WAITV(0); __builtin_amdgcn_s_barrier();
        return;
    }

    const int gb = bid - NSEND;
    const int cg = (gb >> 3) & 3, js = (gb >> 5) * 8 + (gb & 7);
    if (js >= NJ) return;

    if (wave >= CW) {
        const int h = wave - CW;
        const int brow = lane >> 3;
        const int f_lo = (brow >> 1) & 3;
        const char* w0 = p.w + (size_t)(h * 128 + brow) * NL + cg * 128 + (((lane & 7) ^ f_lo) << 4);
        const char* w1 = p.w + (size_t)(h * 128 + brow) * NL + cg * 128 + (((lane & 7) ^ (f_lo | 4)) << 4);
        auto stageB = [&](int c) {
            char* dstl = smem + (c & (NSLOT - 1)) * BSLOT + h * 16384;
#pragma unroll
            for (int i = 0; i < 16; ++i) {
                const char* s = ((i & 2) ? w1 : w0) + ((size_t)c * KC + i * 8) * NL;
                __builtin_amdgcn_global_load_lds((const GLB_AS void*)s, (LDS_AS void*)(dstl + i * 1024), 16, 0, 0);
            }
        };
        stageB(0); stageB(1); stageB(2);
#pragma unroll 1
        for (int c = 0; c < NCH - 3; ++c) {
            WAITV(32);
            STAMP(2 + c);
            __builtin_amdgcn_s_barrier();
            stageB(c + 3);
        }
        WAITV(32); __builtin_amdgcn_s_barrier();
        WAITV(16); __builtin_amdgcn_s_barrier();
        WAITV(0); __builtin_amdgcn_s_barrier();
        return;
    }

    const int rb = min(js * CW + wave, NRB - 1);
    const bool store_out = js * CW + wave < NRB;
    const int srcr = rb >> 5, rbl = rb & 31;
    const bool local = srcr == p.rank;
    const unsigned call = __hip_atomic_load(cnt, __ATOMIC_RELAXED, __HIP_MEMORY_SCOPE_AGENT) + 1;
    const float scale = p.sx[0] * p.sw[0];
    const unsigned fbase = (call - 1) * NCH;

    const int g = lane >> 4;
    const char* abase = local ? p.x + (size_t)(rbl * 16 + (lane & 15)) * KDIM + g * 16
                              : myws + WS_X + ((size_t)(call & 1) * NR + srcr) * XSHARD
                                    + (size_t)(rbl >> 2) * (64 * KDIM) + (rbl & 3) * 4096 + lane * 16;
    const int astep = local ? 64 : 1024, cstep = local ? KC : 16384;
    GLB_AS unsigned* tmo = (GLB_AS unsigned*)(myws + WS_TMO);
    GLB_AS unsigned* flagp = lane == 0 ? (GLB_AS unsigned*)(myws + WS_FLAG + (size_t)(srcr * 8 + (rbl >> 2)) * 128) : tmo;

    const int q = (lane & 15) >> 1;
    const int fl = ((q >> 1) & 3) | ((g & 1) << 2);
    int boff[8];
#pragma unroll
    for (int t = 0; t < 8; ++t) boff[t] = (16 * g + q) * 128 + 8 * (lane & 1) + 16 * (t ^ fl);

    i32x4 acc[8];
#pragma unroll
    for (int t = 0; t < 8; ++t) acc[t] = (i32x4){0, 0, 0, 0};
    int avail = local ? NCH : 0;
    bool bad = false;
    const unsigned long long t0 = __builtin_amdgcn_s_memrealtime();
    STAMP(1);

    auto note = [&](unsigned fv) {
        const int got = (int)(__builtin_amdgcn_readlane(fv, 0) - fbase);
        if (got > avail) avail = got;
        if (__builtin_amdgcn_readlane(fv, 1) == call) { bad = true; avail = NCH; }
    };
    auto spin = [&](int c) {
        while (avail <= c) {
            note(__hip_atomic_load(flagp, __ATOMIC_RELAXED, __HIP_MEMORY_SCOPE_SYSTEM));
            if (avail > c) break;
            if (__builtin_amdgcn_s_memrealtime() - t0 > DEADLINE_TICKS) {
                if (lane == 0) __hip_atomic_store(tmo, call, __ATOMIC_RELAXED, __HIP_MEMORY_SCOPE_AGENT);
                bad = true; avail = NCH;
                break;
            }
            __builtin_amdgcn_s_sleep(1);
        }
    };
    auto issueA = [&](i32x4 (&a)[4], unsigned& fv, int c) {
        const char* a0 = abase + (size_t)c * cstep;
        asm volatile("global_load_dwordx4 %0, %5, off sc0 sc1\n\t"
                     "global_load_dwordx4 %1, %6, off sc0 sc1\n\t"
                     "global_load_dwordx4 %2, %7, off sc0 sc1\n\t"
                     "global_load_dwordx4 %3, %8, off sc0 sc1\n\t"
                     "global_load_dword %4, %9, off sc0 sc1"
                     : "=&v"(a[0]), "=&v"(a[1]), "=&v"(a[2]), "=&v"(a[3]), "=&v"(fv)
                     : "v"(a0), "v"(a0 + astep), "v"(a0 + 2 * astep), "v"(a0 + 3 * astep), "v"(flagp) : "memory");
    };
    auto step = [&](i32x4 (&a)[4], unsigned& fv, i32x4 (&an)[4], unsigned& fvn, int c) {
        __builtin_amdgcn_s_barrier();
        i32x4 bf[4][8];
        const char* sb = smem + (c & (NSLOT - 1)) * BSLOT;
#pragma unroll
        for (int s = 0; s < 4; ++s)
#pragma unroll
            for (int t = 0; t < 8; ++t) {
                i32x2 lo = __builtin_amdgcn_ds_read_tr8_b64_v2i32((LDS_AS i32x2*)(sb + boff[t] + s * 8192));
                i32x2 hi = __builtin_amdgcn_ds_read_tr8_b64_v2i32((LDS_AS i32x2*)(sb + boff[t] + s * 8192 + 1024));
                bf[s][t] = (i32x4){lo[0], lo[1], hi[0], hi[1]};
            }
        asm volatile("s_waitcnt vmcnt(0)" : "+v"(a[0]), "+v"(a[1]), "+v"(a[2]), "+v"(a[3]), "+v"(fv) :: "memory");
        STAMP(20 + c);
        note(fv);
        const bool more = c + 1 < NCH;
        const bool early = more && avail > c + 1;
        if (early) issueA(an, fvn, c + 1);
#pragma unroll
        for (int s = 0; s < 4; ++s)
#pragma unroll
            for (int t = 0; t < 8; ++t)
                acc[t] = __builtin_amdgcn_mfma_i32_16x16x64_i8(bf[s][t], a[s], acc[t], 0, 0, 0);
        if (more && !early) { spin(c + 1); issueA(an, fvn, c + 1); }
        STAMP(3 + c);
    };

    i32x4 a0[4], a1[4];
    unsigned f0, f1;
    spin(0);
    STAMP(2);
    issueA(a0, f0, 0);
    for (int c = 0; c < NCH; c += 2) {
        step(a0, f0, a1, f1, c);
        step(a1, f1, a0, f0, c + 1);
    }
    float* orow = p.out + (size_t)(rb * 16 + (lane & 15)) * NL + cg * 128 + 4 * g;
    const float qnan = __builtin_nanf("");
    if (store_out) {
#pragma unroll
        for (int t = 0; t < 8; ++t) {
            f32x4 o;
#pragma unroll
            for (int e = 0; e < 4; ++e) o[e] = bad ? qnan : (float)acc[t][e] * scale;
            *(f32x4*)(orow + 16 * t) = o;
        }
    }
    if (tid == 0) __hip_atomic_store(cnt, call, __ATOMIC_RELAXED, __HIP_MEMORY_SCOPE_AGENT);
}

extern "C" void kernel_launch(int rank, int nranks, ncclComm_t comm,
                              hipStream_t stream,
                              void* const* d_in, const int* in_sizes, int n_in,
                              void* d_out, int out_size,
                              const int* in_dtypes, int out_dtype,
                              void* const* d_out_all,
                              void* const* d_ws_all, size_t ws_bytes) {
    if (nranks != NR || (unsigned)rank >= (unsigned)NR || n_in != 4 || ws_bytes < WS_NEED
        || in_sizes[0] != ML * KDIM || in_sizes[1] != KDIM * NL || out_size != NR * ML * NL) {
        fprintf(stderr, "ag_gemm: unexpected problem shape\n");
        return;
    }
    HIP_CHECK(hipFuncSetAttribute((const void*)ag_gemm, hipFuncAttributeMaxDynamicSharedMemorySize, LDS_BYTES));
    Params p;
    p.x = (const char*)d_in[0]; p.w = (const char*)d_in[1];
    p.sx = (const float*)d_in[2]; p.sw = (const float*)d_in[3];
    p.out = (float*)d_out;
    for (int i = 0; i < NR; ++i) p.ws[i] = (char*)d_ws_all[i];
    p.rank = rank; p.pad = 0;
    hipLaunchKernelGGL(ag_gemm, dim3(GRID), dim3(512), LDS_BYTES, stream, p);
    HIP_CHECK(hipGetLastError());
}
```

```cpp
#include <hip/hip_runtime.h>
#include <rccl/rccl.h>
#include <cstdio>
#include <cstdint>

#define HIP_CHECK(expr) do { hipError_t _e = (expr); if (_e != hipSuccess) { \
    fprintf(stderr, "HIP error %s at %s:%d\n", hipGetErrorString(_e), __FILE__, __LINE__); \
    return; } } while (0)

typedef unsigned u32x4 __attribute__((ext_vector_type(4)));
typedef int i32x4 __attribute__((ext_vector_type(4)));
typedef int i32x2 __attribute__((ext_vector_type(2)));
typedef float f32x4 __attribute__((ext_vector_type(4)));
#define LDS_AS __attribute__((address_space(3)))
#define GLB_AS __attribute__((address_space(1)))

constexpr int NR = 8;
constexpr int KDIM = 4096;
constexpr int ML = 512;
constexpr int NL = 512;
constexpr int NRB = ML / 16;
constexpr int RBB = 16 * KDIM;
constexpr int NSEND = (NR - 1) * 4;
constexpr int GEMM0 = 32;
constexpr int NREP = 24;
constexpr int GRID = GEMM0 + 8 * NREP;
constexpr int WPB = 8;
constexpr int LDS_BYTES = 128 << 10;
constexpr size_t WS_CNT = 0;
constexpr size_t WS_TMO = 48 << 10;
constexpr size_t WS_FLAG = 64 << 10;
constexpr size_t WS_X = 1 << 20;
constexpr size_t XSHARD = (size_t)ML * KDIM;
constexpr size_t WS_DIAG = WS_X + 2 * NR * XSHARD;
constexpr size_t WS_NEED = WS_DIAG + (4 << 20);
constexpr unsigned long long DEADLINE_TICKS = 30000000ull;

#define PACE 3520
#define LEAD 50

struct Params {
    const char* x; const char* w; const float* sx; const float* sw; float* out;
    char* ws[NR];
    int rank; int pad;
};

#define WAITV(n) asm volatile("s_waitcnt vmcnt(" #n ")" ::: "memory")

#define STAMP(idx) do {} while (0)

__global__ void __launch_bounds__(512, 2) ag_gemm(Params p) {
    extern __shared__ __attribute__((aligned(16))) char smem[];
    const int tid = threadIdx.x, lane = tid & 63;
    const int wave = __builtin_amdgcn_readfirstlane(tid >> 6);
    const int bid = blockIdx.x;
    char* myws = p.ws[p.rank];
    GLB_AS unsigned* cnt = (GLB_AS unsigned*)(myws + WS_CNT + (size_t)bid * 128);
    const unsigned lbase = (unsigned)(size_t)(LDS_AS char*)smem;

    if (bid < GEMM0) {
        if (bid >= NSEND) return;
        const int j = bid & 3, pi = bid >> 2;
        const unsigned peer = (unsigned)(p.rank + 1 + pi) % NR;
        const unsigned ts = (unsigned)__builtin_amdgcn_s_memrealtime() + LEAD;
        const char* src = p.x + (size_t)(lane & 15) * KDIM + (j * 16 + wave * 2) * 64 + (lane >> 4) * 16;
        u32x4 v0, v1, u0, u1;
        unsigned c0;
        asm volatile("global_load_dword %4, %7, off sc1\n\t"
                     "global_load_dwordx4 %0, %5, off\n\t"
                     "global_load_dwordx4 %1, %5, off offset:64\n\t"
                     "global_load_dwordx4 %2, %6, off\n\t"
                     "global_load_dwordx4 %3, %6, off offset:64\n\t"
                     "ds_write_b32 %8, %9\n\ts_waitcnt lgkmcnt(0)\n\ts_barrier\n\ts_waitcnt vmcnt(0)"
                     : "=&v"(v0), "=&v"(v1), "=&v"(u0), "=&v"(u1), "=&v"(c0)
                     : "v"(src), "v"(src + RBB), "v"(cnt), "v"(lbase + 4 * lane), "v"(0u) : "memory");
        const unsigned call = __builtin_amdgcn_readfirstlane(c0) + 1;
        char* dst = p.ws[peer] + WS_X + ((size_t)(call & 1) * NR + p.rank) * XSHARD + (j * 16 + wave * 2) * 1024 + lane * 16;
        GLB_AS unsigned* flag = (GLB_AS unsigned*)(p.ws[peer] + WS_FLAG + (size_t)p.rank * NRB * 128) + j;
        auto send = [&](u32x4& a, u32x4& b, int i) {
            const int due = (((i * 8 + wave) * 4 + j) * PACE) >> 10;
            while ((int)((unsigned)__builtin_amdgcn_s_memrealtime() - ts) < due) __builtin_amdgcn_s_sleep(1);
            asm volatile("global_store_dwordx4 %0, %1, off sc0 sc1\n\t"
                         "global_store_dwordx4 %0, %2, off offset:1024 sc0 sc1\n\ts_nop 1"
                         :: "v"(dst + (size_t)i * RBB), "v"(a), "v"(b) : "memory");
            if (i == 0) STAMP(40);
            asm volatile("s_waitcnt vmcnt(0)" : "+v"(v0), "+v"(v1), "+v"(u0), "+v"(u1) :: "memory");
            if (lane == 0) {
                unsigned old;
                asm volatile("ds_add_rtn_u32 %0, %1, %2\n\ts_waitcnt lgkmcnt(0)" : "=&v"(old) : "v"(lbase + 4 * i), "v"(1u) : "memory");
                if (old == WPB - 1) {
                    __hip_atomic_store(flag + i * 32, call, __ATOMIC_RELAXED, __HIP_MEMORY_SCOPE_SYSTEM);
                    if (i == NRB - 1) __hip_atomic_store(cnt, call, __ATOMIC_RELAXED, __HIP_MEMORY_SCOPE_AGENT);
                }
            }
            if (i + 2 < NRB)
                asm volatile("global_load_dwordx4 %0, %2, off\n\tglobal_load_dwordx4 %1, %2, off offset:64"
                             : "+v"(a), "+v"(b) : "v"(src + (size_t)(i + 2) * RBB) : "memory");
        };
#pragma unroll 1
        for (int i = 0; i < NRB; i += 2) { send(v0, v1, i); send(u0, u1, i + 1); }
        return;
    }

    const int gb = bid - GEMM0;
    const int slice = (gb >> 3) & 7, rep = (gb & 7) * 3 + (gb >> 6);
    const int g = lane >> 4;

    i32x4 bf[8][4];
    {
        const int brow = lane >> 2;
        const char* w0 = p.w + (size_t)(wave * 512 + brow) * NL + slice * 64;
        const int f0 = (brow >> 2) & 1;
        const int q = (lane & 15) >> 1;
        const int fr = ((q >> 2) & 1) | ((g & 1) << 1);
        const unsigned rd = lbase + wave * 16384 + (16 * g + q) * 64 + 8 * (lane & 1);
#pragma unroll
        for (int hh = 0; hh < 2; ++hh) {
#pragma unroll
            for (int i = 0; i < 16; ++i)
                __builtin_amdgcn_global_load_lds((const GLB_AS void*)(w0 + (size_t)(hh * 256 + i * 16) * NL + (((lane & 3) ^ (f0 | ((i & 1) << 1))) << 4)),
                                                 (LDS_AS void*)(smem + wave * 16384 + i * 1024), 16, 0, 0);
            WAITV(0);
#pragma unroll
            for (int s = 0; s < 4; ++s)
#pragma unroll
                for (int t = 0; t < 4; ++t) {
                    i32x2 lo = __builtin_amdgcn_ds_read_tr8_b64_v2i32((LDS_AS i32x2*)(size_t)(rd + s * 4096 + ((t ^ fr) << 4)));
                    i32x2 hi = __builtin_amdgcn_ds_read_tr8_b64_v2i32((LDS_AS i32x2*)(size_t)(rd + s * 4096 + 512 + ((t ^ fr) << 4)));
                    bf[hh * 4 + s][t] = (i32x4){lo[0], lo[1], hi[0], hi[1]};
                }
            asm volatile("s_waitcnt lgkmcnt(0)" ::: "memory");
#pragma unroll
            for (int s = 0; s < 4; ++s)
#pragma unroll
                for (int t = 0; t < 4; ++t) asm volatile("" : "+v"(bf[hh * 4 + s][t]));
        }
    }
    const unsigned call = __hip_atomic_load(cnt, __ATOMIC_RELAXED, __HIP_MEMORY_SCOPE_AGENT) + 1;
    const float scale = p.sx[0] * p.sw[0];
    GLB_AS unsigned* tmo = (GLB_AS unsigned*)(myws + WS_TMO);
    const char* xloc = p.x + (size_t)(lane & 15) * KDIM + wave * 512 + g * 16;
    const char* xrem = myws + WS_X + (size_t)(call & 1) * NR * XSHARD + wave * 8192 + lane * 16;
    const unsigned long long t0 = __builtin_amdgcn_s_memrealtime();
    bool bad = false;
    asm volatile("s_barrier" ::: "memory");
    STAMP(1);

    int nt = 0;
#pragma unroll 1
    for (int idx = rep; idx < NR * NRB; idx += NREP, ++nt) {
        const bool local = idx < NRB;
        const int r = idx - NRB;
        const int rb = local ? idx : r / 7;
        const int srcr = local ? p.rank : (p.rank + 1 + r % 7) & (NR - 1);
        i32x4 a[8];
        if (local) {
            const char* s = xloc + (size_t)rb * RBB;
            asm volatile("global_load_dwordx4 %0, %8, off\n\tglobal_load_dwordx4 %1, %8, off offset:64\n\t"
                         "global_load_dwordx4 %2, %8, off offset:128\n\tglobal_load_dwordx4 %3, %8, off offset:192\n\t"
                         "global_load_dwordx4 %4, %8, off offset:256\n\tglobal_load_dwordx4 %5, %8, off offset:320\n\t"
                         "global_load_dwordx4 %6, %8, off offset:384\n\tglobal_load_dwordx4 %7, %8, off offset:448\n\ts_waitcnt vmcnt(0)"
                         : "=&v"(a[0]), "=&v"(a[1]), "=&v"(a[2]), "=&v"(a[3]), "=&v"(a[4]), "=&v"(a[5]), "=&v"(a[6]), "=&v"(a[7])
                         : "v"(s) : "memory");
        } else {
            GLB_AS unsigned* fp = lane == 0 ? (GLB_AS unsigned*)(myws + WS_FLAG + (size_t)(srcr * NRB + rb) * 128) + (wave >> 1) : tmo;
            while (!bad) {
                const unsigned fv = __hip_atomic_load(fp, __ATOMIC_RELAXED, __HIP_MEMORY_SCOPE_SYSTEM);
                if (__builtin_amdgcn_readlane(fv, 1) == call) { bad = true; break; }
                if (__builtin_amdgcn_readlane(fv, 0) >= call) break;
                if (__builtin_amdgcn_s_memrealtime() - t0 > DEADLINE_TICKS) {
                    if (lane == 0) __hip_atomic_store(tmo, call, __ATOMIC_RELAXED, __HIP_MEMORY_SCOPE_AGENT);
                    bad = true;
                    break;
                }
                __builtin_amdgcn_s_sleep(1);
            }
            STAMP(2 + nt);
            const char* s = xrem + (size_t)(srcr * NRB + rb) * RBB;
            asm volatile("global_load_dwordx4 %0, %8, off sc0 sc1\n\tglobal_load_dwordx4 %1, %8, off offset:1024 sc0 sc1\n\t"
                         "global_load_dwordx4 %2, %8, off offset:2048 sc0 sc1\n\tglobal_load_dwordx4 %3, %8, off offset:3072 sc0 sc1\n\t"
                         "global_load_dwordx4 %4, %9, off sc0 sc1\n\tglobal_load_dwordx4 %5, %9, off offset:1024 sc0 sc1\n\t"
                         "global_load_dwordx4 %6, %9, off offset:2048 sc0 sc1\n\tglobal_load_dwordx4 %7, %9, off offset:3072 sc0 sc1\n\ts_waitcnt vmcnt(0)"
                         : "=&v"(a[0]), "=&v"(a[1]), "=&v"(a[2]), "=&v"(a[3]), "=&v"(a[4]), "=&v"(a[5]), "=&v"(a[6]), "=&v"(a[7])
                         : "v"(s), "v"(s + 4096) : "memory");
        }
        STAMP(16 + nt);
        i32x4 acc[4];
#pragma unroll
        for (int t = 0; t < 4; ++t) {
            acc[t] = (i32x4){0, 0, 0, 0};
#pragma unroll
            for (int s = 0; s < 8; ++s) acc[t] = __builtin_amdgcn_mfma_i32_16x16x64_i8(bf[s][t], a[s], acc[t], 0, 0, 0);
        }
        char* pb = smem + (nt & 1) * 36864;
#pragma unroll
        for (int t = 0; t < 4; ++t)
            *(i32x4*)(pb + wave * 4096 + (lane & 15) * 256 + (((4 * t + g) ^ (lane & 15)) << 4)) = acc[t];
        if (lane == 0) *(int*)(pb + 32768 + wave * 4) = bad;
        asm volatile("s_waitcnt lgkmcnt(0)\n\ts_barrier" ::: "memory");
        STAMP(30 + nt);
        if ((wave >> 2) == (nt & 1)) {
            const int row = (wave & 3) * 4 + g;
            const char* rp = pb + row * 256 + (((lane & 15) ^ row) << 4);
            i32x4 sum = *(const i32x4*)rp;
#pragma unroll
            for (int v = 1; v < 8; ++v) sum += *(const i32x4*)(rp + v * 4096);
            const i32x4 b0 = *(const i32x4*)(pb + 32768), b1 = *(const i32x4*)(pb + 32784);
            const bool anybad = (b0[0] | b0[1] | b0[2] | b0[3] | b1[0] | b1[1] | b1[2] | b1[3]) != 0;
            const float sc = anybad ? __builtin_nanf("") : scale;
            f32x4 o;
#pragma unroll
            for (int e = 0; e < 4; ++e) o[e] = (float)sum[e] * sc;
            float* op = p.out + (size_t)(srcr * ML + rb * 16 + row) * NL + slice * 64 + (lane & 15) * 4;
            asm volatile("global_store_dwordx4 %0, %1, off sc0 sc1\n\ts_nop 1" :: "v"(op), "v"(o) : "memory");
            STAMP(44 + nt);
        }
    }
    if (tid == 0) __hip_atomic_store(cnt, call, __ATOMIC_RELAXED, __HIP_MEMORY_SCOPE_AGENT);
}

extern "C" void kernel_launch(int rank, int nranks, ncclComm_t comm,
                              hipStream_t stream,
                              void* const* d_in, const int* in_sizes, int n_in,
                              void* d_out, int out_size,
                              const int* in_dtypes, int out_dtype,
                              void* const* d_out_all,
                              void* const* d_ws_all, size_t ws_bytes) {
    if (nranks != NR || (unsigned)rank >= (unsigned)NR || n_in != 4 || ws_bytes < WS_NEED
        || in_sizes[0] != ML * KDIM || in_sizes[1] != KDIM * NL || out_size != NR * ML * NL) {
        fprintf(stderr, "ag_gemm: unexpected problem shape\n");
        return;
    }
    HIP_CHECK(hipFuncSetAttribute((const void*)ag_gemm, hipFuncAttributeMaxDynamicSharedMemorySize, LDS_BYTES));
    Params p;
    p.x = (const char*)d_in[0]; p.w = (const char*)d_in[1];
    p.sx = (const float*)d_in[2]; p.sw = (const float*)d_in[3];
    p.out = (float*)d_out;
    for (int i = 0; i < NR; ++i) p.ws[i] = (char*)d_ws_all[i];
    p.rank = rank; p.pad = 0;
    hipLaunchKernelGGL(ag_gemm, dim3(GRID), dim3(512), LDS_BYTES, stream, p);
    HIP_CHECK(hipGetLastError());
}
```

```cpp
#include <hip/hip_runtime.h>
#include <rccl/rccl.h>
#include <cstdio>
#include <cstdint>

#define HIP_CHECK(expr) do { hipError_t _e = (expr); if (_e != hipSuccess) { \
    fprintf(stderr, "HIP error %s at %s:%d\n", hipGetErrorString(_e), __FILE__, __LINE__); \
    return; } } while (0)

typedef unsigned u32x4 __attribute__((ext_vector_type(4)));
typedef int i32x4 __attribute__((ext_vector_type(4)));
typedef int i32x2 __attribute__((ext_vector_type(2)));
typedef float f32x4 __attribute__((ext_vector_type(4)));
#define LDS_AS __attribute__((address_space(3)))
#define GLB_AS __attribute__((address_space(1)))

#define NF 7
#define D1 13
#define ROUND 1120
#define LEAD 125
#define SLEAD 10
#define SORD (i + 1 < nround ? (i * 4 + wq) * 8 + j : (i * 8 + ((j + pi) & 7)) * 4 + wq)
#define CATCH 4
#define YSLOT 545
#define LAZY 8
#define WAKE 2
#define RAWPOS 122

constexpr int NR = 8;
constexpr int KDIM = 4096;
constexpr int ML = 512;
constexpr int NL = 512;
constexpr int NRB = ML / 16;
constexpr int ND = NRB - NF;
constexpr int RBB = 16 * KDIM;
constexpr unsigned OFFS = NF == 5 ? 0x20321u : NF == 6 ? 0x310321u : 0x03210321u;
__host__ __device__ constexpr int offs(int f) { return (OFFS >> (4 * f)) & 3; }
static_assert(NF >= 4 && NF <= 8 && D1 <= ND, "plan");
constexpr int GEMM0 = (NR - 1) * 8;
constexpr int GRID = GEMM0 + 8 * 24;
constexpr int WPB = 8;
constexpr int LDS_BYTES = 140 << 10;
constexpr int NSLOT = NR * NRB + 32;
constexpr size_t WS_CNT = 0;
constexpr size_t WS_TMO = 48 << 10;
constexpr size_t WS_FLAG = 64 << 10;
constexpr size_t WS_YFLAG = 128 << 10;
constexpr size_t WS_X = 1 << 20;
constexpr size_t XPAR = (size_t)NSLOT * RBB;
constexpr size_t WS_Y = WS_X + 2 * XPAR;
constexpr size_t YPAR = (size_t)4 * 32 * 8 * 3072;
constexpr size_t WS_DIAG = WS_Y + 2 * YPAR;
constexpr size_t WS_NEED = WS_DIAG + (4 << 20);
constexpr unsigned long long DEADLINE_TICKS = 30000000ull;

struct Params {
    const char* x; const char* w; const float* sx; const float* sw;
    char* ws[NR];
    float* out[NR];
    int rank;
    unsigned fin[4];
    unsigned fout[4];
    unsigned extra[2];
    int pad;
};

__device__ __forceinline__ int list_slot(int r, int pos) {
    if (pos < NRB) return r * NRB + pos;
    const int q = pos - NRB, t = q / 3, s = q - 3 * t;
    const int rb = t < D1 ? t : (t < D1 + NF ? ND + t - D1 : t - NF);
    return ((r & 4) | ((r + 1 + s) & 3)) * NRB + rb;
}
__device__ __forceinline__ bool held_by(int slot, int k) {
    const int rb = slot & (NRB - 1);
    return rb >= ND && (((slot >> 5) + offs(rb - ND)) & 3) == k;
}

#define WAITV(n) asm volatile("s_waitcnt vmcnt(" #n ")" ::: "memory")

#define STAMP(idx) do {} while (0)

__global__ void __launch_bounds__(512, 2) ag_gemm(Params p) {
    extern __shared__ __attribute__((aligned(16))) char smem[];
    const int tid = threadIdx.x, lane = tid & 63;
    const int wave = __builtin_amdgcn_readfirstlane(tid >> 6);
    const int bid = blockIdx.x;
    const unsigned long long t0 = __builtin_amdgcn_s_memrealtime();
    unsigned ts = (unsigned)t0 + LEAD;
    const int rank = p.rank, me = rank & 3, other = (rank & 4) ^ 4;
    char* myws = p.ws[rank];
    GLB_AS unsigned* cnt = (GLB_AS unsigned*)(myws + WS_CNT + (size_t)bid * 128);
    GLB_AS unsigned* tmo = (GLB_AS unsigned*)(myws + WS_TMO);
    const unsigned lbase = (unsigned)(size_t)(LDS_AS char*)smem;
    unsigned call;
    bool bad = false;
    auto wait_flag = [&](const char* f, unsigned want, bool lazy = false) -> unsigned {
        GLB_AS unsigned* fp = lane == 0 ? (GLB_AS unsigned*)f : tmo;
        while (!bad) {
            const unsigned fv = __hip_atomic_load(fp, __ATOMIC_RELAXED, __HIP_MEMORY_SCOPE_SYSTEM);
            if (__builtin_amdgcn_readlane(fv, 1) == call) { bad = true; break; }
            const unsigned v = __builtin_amdgcn_readlane(fv, 0);
            if (v >= want) return v;
            if (__builtin_amdgcn_s_memrealtime() - t0 > DEADLINE_TICKS) {
                if (lane == 0) __hip_atomic_store(tmo, call, __ATOMIC_RELAXED, __HIP_MEMORY_SCOPE_AGENT);
                bad = true;
                break;
            }
            if (lazy) __builtin_amdgcn_s_sleep(LAZY); else __builtin_amdgcn_s_sleep(1);
        }
        return 0;
    };
    auto pace = [&](int due) { while ((int)((unsigned)__builtin_amdgcn_s_memrealtime() - ts) < due) __builtin_amdgcn_s_sleep(1); };
    auto snooze = [&](int due) { while ((int)((unsigned)__builtin_amdgcn_s_memrealtime() - ts) < due) __builtin_amdgcn_s_sleep(8); };
    auto lds_add = [&](unsigned addr, unsigned v) {
        unsigned old;
        asm volatile("ds_add_rtn_u32 %0, %1, %2\n\ts_waitcnt lgkmcnt(0)" : "=&v"(old) : "v"(addr), "v"(v) : "memory");
        return old;
    };

    const int g = lane >> 4;
    if (bid < GEMM0) {
        const int j = bid & 7, pi = bid >> 3;
        const bool cross = pi >= 3;
        const int k = cross ? pi - 3 : (me + 1 + pi) & 3;
        const unsigned peer = (unsigned)((cross ? other : rank & 4) | k);
        const int nround = cross ? 8 + NF : ND;
        const int wq = wave & 3, frag = j * 8 + wq * 2;
        int vslot;
        {
            const unsigned fin = p.fin[k], fout = p.fout[k];
            const int nin = __builtin_popcount(fin);
            unsigned m = lane < nin ? fin : fout;
            const int c = lane < nin ? lane : lane - nin - 8;
#pragma unroll
            for (int t = 0; t < NF - 1; ++t) m &= m - (c > t);
            const int b = __builtin_ctz(m | 0x80000000u);
            vslot = !cross ? rank * NRB + (lane & 31) : lane < nin ? rank * NRB + ND + b
                  : lane < nin + 8 ? NR * NRB + me * 8 + lane - nin : (other | (b & 3)) * NRB + min(ND + (b >> 2), NRB - 1);
        }
        const unsigned stg = lbase + 1024 + wave * 4096;
        unsigned fseen = 0;
        auto fetch = [&](int i) {
            const int slot = __builtin_amdgcn_readlane(vslot, i);
            const int kind = (slot >> 7) - (other >> 2);
            const char* base; unsigned o0, o1;
            if (slot >= NR * NRB) {
                const int r = slot & 7;
                base = p.w + (size_t)(((r & 3) * 8 + j) * 128 + wq * 32) * NL + k * 128 + (r >> 2) * 64;
                o0 = (lane >> 2) * NL + (((lane & 3) ^ (g & 1)) << 4);
                o1 = (o0 ^ 32) + 16 * NL;
            } else if (kind != 0) {
                base = p.x + (size_t)(slot & 31) * RBB + frag * 64;
                o0 = (lane >> 3) * KDIM + (((lane & 7) ^ g) << 4);
                o1 = (o0 ^ 64) + 8 * KDIM;
            } else {
                if (__builtin_amdgcn_readlane(fseen, i) < call) wait_flag(myws + WS_FLAG + slot * 128 + j * 4, call);
                base = myws + WS_X + (call & 1) * XPAR + (size_t)slot * RBB + frag * 1024;
                o0 = lane * 16; o1 = o0 + 1024;
            }
            const unsigned d = stg + ((i >> 1) & 1) * 2048;
            unsigned keep;
            asm volatile("s_mov_b32 %0, m0\n\ts_mov_b32 m0, %4\n\ts_cmp_eq_u32 %5, 0\n\ts_nop 4\n\ts_cbranch_scc1 1f\n\t"
                         "global_load_lds_dwordx4 %1, %3\n\ts_add_u32 m0, m0, 1024\n\ts_nop 0\n\t"
                         "global_load_lds_dwordx4 %2, %3\n\ts_branch 2f\n"
                         "1:\n\tglobal_load_lds_dwordx4 %1, %3 sc0 sc1\n\ts_add_u32 m0, m0, 1024\n\ts_nop 0\n\t"
                         "global_load_lds_dwordx4 %2, %3 sc0 sc1\n"
                         "2:\n\ts_mov_b32 m0, %0"
                         : "=&s"(keep) : "v"(o0), "v"(o1), "s"(base), "s"(d), "s"(kind) : "memory", "scc");
        };
        call = 0;
        fetch(wave >> 2);
        {
            unsigned c0;
            asm volatile("global_load_dword %0, %1, off sc1\n\t"
                         "ds_write_b32 %2, %3\n\ts_waitcnt lgkmcnt(0)\n\ts_barrier\n\ts_waitcnt vmcnt(0)"
                         : "=&v"(c0) : "v"(cnt), "v"(lbase + 4 * lane), "v"(0u) : "memory");
            call = __builtin_amdgcn_readfirstlane(c0) + 1;
        }
        ts = (unsigned)__builtin_amdgcn_s_memrealtime() + SLEAD;
        int shift = 0;
        if (wave >= 4) fetch(3);
        char* dst = p.ws[peer] + WS_X + (call & 1) * XPAR + frag * 1024;
        char* flag = p.ws[peer] + WS_FLAG + j * 4;
        const unsigned l16 = lane * 16;
#pragma unroll 1
        for (int i = wave >> 2; i < nround; i += 2) {
            const int slot = __builtin_amdgcn_readlane(vslot, i);
            const bool own = (slot >> 5) == rank;
            const unsigned r0 = stg + ((i >> 1) & 1) * 2048 + (own ? (lane & 15) * 128 + ((g ^ ((lane & 15) >> 1)) << 4) : l16);
            u32x4 v0, v1;
            if (slot < NR * NRB) {
                asm volatile("ds_read_b128 %0, %2\n\tds_read_b128 %1, %3\n\ts_waitcnt lgkmcnt(0)"
                             : "=&v"(v0), "=&v"(v1) : "v"(r0), "v"(own ? r0 ^ 64 : r0 + 1024) : "memory");
            } else {
                const int q = (lane & 15) >> 1;
                const unsigned rt = stg + ((i >> 1) & 1) * 2048 + (16 * (g & 1) + q) * 64 + 8 * (lane & 1) + (((g >> 1) ^ (q >> 2) ^ ((g & 1) << 1)) << 4);
                i32x2 a, b, c, d;
                asm volatile("ds_read_b64_tr_b8 %0, %4\n\tds_read_b64_tr_b8 %1, %4 offset:512\n\t"
                             "ds_read_b64_tr_b8 %2, %5\n\tds_read_b64_tr_b8 %3, %5 offset:512\n\ts_waitcnt lgkmcnt(0)"
                             : "=&v"(a), "=&v"(b), "=&v"(c), "=&v"(d) : "v"(rt), "v"(rt ^ 32) : "memory");
                v0 = (u32x4){(unsigned)a[0], (unsigned)a[1], (unsigned)b[0], (unsigned)b[1]};
                v1 = (u32x4){(unsigned)c[0], (unsigned)c[1], (unsigned)d[0], (unsigned)d[1]};
            }
            shift = max(shift - CATCH, 0);
            shift = max(shift, (int)((unsigned)__builtin_amdgcn_s_memrealtime() - ts) - (SORD * ROUND) / 320);
            pace((SORD * ROUND) / 320 + shift);
            asm volatile("global_store_dwordx4 %0, %2, %1 sc0 sc1\n\tglobal_store_dwordx4 %0, %3, %1 offset:1024 sc0 sc1\n\ts_nop 1"
                         :: "v"(l16), "s"(dst + (size_t)slot * RBB), "v"(v0), "v"(v1) : "memory");
            if (i == 0) { STAMP(40); fetch(2); }
            if (cross && (i >> 1) == 2) fseen = __hip_atomic_load((GLB_AS unsigned*)(myws + WS_FLAG + (vslot & 255) * 128 + j * 4), __ATOMIC_RELAXED, __HIP_MEMORY_SCOPE_SYSTEM);
            asm volatile("s_waitcnt vmcnt(0)" : "+v"(fseen) :: "memory");
            if (lane == 0 && !bad && lds_add(lbase + 4 * i, 1) == 3) {
                __hip_atomic_store((GLB_AS unsigned*)(flag + slot * 128), call, __ATOMIC_RELAXED, __HIP_MEMORY_SCOPE_SYSTEM);
            }
            if (i + 4 < nround) fetch(i + 4);
        }
        if (lane == 0 && lds_add(lbase + 4 * 63, 1) == WPB - 1) __hip_atomic_store(cnt, call, __ATOMIC_RELAXED, __HIP_MEMORY_SCOPE_AGENT);
        if (!cross) return;

        const float scale = p.sx[0] * p.sw[0];
        const int prod = j * 4 + wq, h = prod >> 4, rep = prod & 15, e0 = (wave >> 2) * 2;
        const char* yf = myws + WS_YFLAG + (k * 32 + prod) * 128;
        const char* yb = myws + WS_Y + (call & 1) * YPAR + (size_t)(k * 32 + prod) * 8 * 3072 + e0 * 768 + lane * 4;
        const int yslot = list_slot(peer, rep + 16 * (lane & 7));
        unsigned todo = (unsigned)__ballot(lane < 8 && !held_by(yslot, me));
#pragma unroll 1
        for (; todo; todo &= todo - 1) {
            const int n = __builtin_ctz(todo), slot = __builtin_amdgcn_readlane(yslot, n);
            const unsigned raw = (wait_flag(yf + n * 4, call * 16) >> e0) & 3;
            STAMP(44 + n);
            if (raw == 3) continue;
            unsigned d[6];
            asm volatile("global_load_dword %0, %6, off sc0 sc1\n\tglobal_load_dword %1, %6, off offset:256 sc0 sc1\n\t"
                         "global_load_dword %2, %6, off offset:512 sc0 sc1\n\tglobal_load_dword %3, %6, off offset:768 sc0 sc1\n\t"
                         "global_load_dword %4, %6, off offset:1024 sc0 sc1\n\tglobal_load_dword %5, %6, off offset:1280 sc0 sc1\n\t"
                         "s_waitcnt vmcnt(0)"
                         : "=&v"(d[0]), "=&v"(d[1]), "=&v"(d[2]), "=&v"(d[3]), "=&v"(d[4]), "=&v"(d[5]) : "v"(yb + n * 3072) : "memory");
            const float sc = bad ? __builtin_nanf("") : scale;
            float* op = p.out[rank] + (size_t)(slot * 16 + e0 * 4 + g) * NL + k * 128 + h * 64 + (lane & 15) * 4;
#pragma unroll
            for (int e = 0; e < 2; ++e) {
                if ((raw >> e) & 1) continue;
                const unsigned d0 = d[3 * e], d1 = d[3 * e + 1], d2 = d[3 * e + 2];
                f32x4 o;
                o[0] = (float)((int)(d0 << 8) >> 8) * sc;
                o[1] = (float)((int)(((d0 >> 16) & 0xff00u) | (d1 << 16)) >> 8) * sc;
                o[2] = (float)((int)(((d1 >> 8) & 0xffff00u) | (d2 << 24)) >> 8) * sc;
                o[3] = (float)((int)d2 >> 8) * sc;
                asm volatile("global_store_dwordx4 %0, %1, off sc0 sc1\n\ts_nop 1" :: "v"(op + e * 4 * NL), "v"(o) : "memory");
            }
        }
        return;
    }

    const int gb = bid - GEMM0;
    const int xcd = gb & 7, sl = (gb >> 3) & 7, set = gb >> 6;
    const bool foreign = set > 0;
    const int rep = foreign ? xcd + 8 * (set - 1) : xcd;
    call = __builtin_amdgcn_readfirstlane(__hip_atomic_load(cnt, __ATOMIC_RELAXED, __HIP_MEMORY_SCOPE_AGENT)) + 1;
    const float scale = p.sx[0] * p.sw[0];
    const char* xws = myws + WS_X + (call & 1) * XPAR;

    if (set == 1 && sl < 4) {
        const int rb = sl * 8 + xcd;
        const char* s = p.x + (size_t)(rb * 16 + (lane & 15)) * KDIM + wave * 512 + g * 16;
        const char* d = xws + (size_t)(rank * NRB + rb) * RBB + wave * 8192 + lane * 16;
        u32x4 v[8];
#pragma unroll
        for (int i = 0; i < 8; ++i) v[i] = *(const u32x4*)(s + i * 64);
        asm volatile("" : "+v"(v[0]), "+v"(v[1]), "+v"(v[2]), "+v"(v[3]), "+v"(v[4]), "+v"(v[5]), "+v"(v[6]), "+v"(v[7]));
#pragma unroll
        for (int i = 0; i < 8; ++i)
            asm volatile("global_store_dwordx4 %0, %1, off sc0 sc1\n\ts_nop 1" :: "v"(d + i * 1024), "v"(v[i]) : "memory");
        WAITV(0);
        if (lane == 0) __hip_atomic_store((GLB_AS unsigned*)(myws + WS_FLAG + (rank * NRB + rb) * 128) + wave, call, __ATOMIC_RELAXED, __HIP_MEMORY_SCOPE_SYSTEM);
        STAMP(2);
    }

    i32x4 bf[8][4];
    if (foreign) {
        const int wslot = NR * NRB + (sl >> 1) * 8 + (sl & 1) * 4 + (wave >> 1);
        snooze(((WAKE + (sl & 1) * 4 + (wave >> 1)) * ROUND) / 10);
#pragma unroll 1
        for (int e = 3; e >= 0; --e) wait_flag(myws + WS_FLAG + wslot * 128 + ((wave & 1) * 4 + e) * 4, call, true);
        asm volatile("" ::: "memory");
        const __amdgpu_buffer_rsrc_t rs = __builtin_amdgcn_make_buffer_rsrc((void*)(xws + (size_t)wslot * RBB + (wave & 1) * 32768), 0, 32768, 0x00020000);
#pragma unroll
        for (int s = 0; s < 8; ++s)
#pragma unroll
            for (int t = 0; t < 4; ++t)
                bf[s][t] = __builtin_bit_cast(i32x4, __builtin_amdgcn_raw_buffer_load_b128(rs, (lane >> 5) * 2048 + (lane & 31) * 16 + t * 512, s * 4096, 17));
    } else {
        const int brow = lane >> 2;
        const int f0 = (brow >> 2) & 1;
        const char* b0 = p.w + (size_t)(wave * 512 + brow) * NL + sl * 64 + (((lane & 3) ^ f0) << 4);
        const char* b1 = p.w + (size_t)(wave * 512 + 16 + brow) * NL + sl * 64 + (((lane & 3) ^ (f0 | 2)) << 4);
        const int q = (lane & 15) >> 1;
        const int fr = ((q >> 2) & 1) | ((g & 1) << 1);
        const unsigned rd = lbase + wave * 16384 + (16 * g + q) * 64 + 8 * (lane & 1);
#pragma unroll
        for (int hh = 0; hh < 2; ++hh) {
#pragma unroll
            for (int i = 0; i < 16; ++i)
                __builtin_amdgcn_global_load_lds((const GLB_AS void*)(((i & 1) ? b1 : b0) + (size_t)(hh * 8 + (i >> 1)) * 32 * NL),
                                                 (LDS_AS void*)(smem + wave * 16384 + i * 1024), 16, 0, 0);
            WAITV(0);
#pragma unroll
            for (int s = 0; s < 4; ++s)
#pragma unroll
                for (int t = 0; t < 4; ++t) {
                    i32x2 lo = __builtin_amdgcn_ds_read_tr8_b64_v2i32((LDS_AS i32x2*)(size_t)(rd + s * 4096 + ((t ^ fr) << 4)));
                    i32x2 hi = __builtin_amdgcn_ds_read_tr8_b64_v2i32((LDS_AS i32x2*)(size_t)(rd + s * 4096 + 512 + ((t ^ fr) << 4)));
                    bf[hh * 4 + s][t] = (i32x4){lo[0], lo[1], hi[0], hi[1]};
                }
            asm volatile("s_waitcnt lgkmcnt(0)" ::: "memory");
#pragma unroll
            for (int s = 0; s < 4; ++s)
#pragma unroll
                for (int t = 0; t < 4; ++t) asm volatile("" : "+v"(bf[hh * 4 + s][t]));
        }
    }
#pragma unroll
    for (int s = 0; s < 8; ++s) asm volatile("" : "+v"(bf[s][0]), "+v"(bf[s][1]), "+v"(bf[s][2]), "+v"(bf[s][3]));
    constexpr int PBUF = 32768 + 64, AST0 = 2 * PBUF, FST0 = AST0 + 65536, PACK0 = FST0 + 8 * 256, ARR0 = PACK0 + 8 * 768;
    static_assert(ARR0 + 64 <= LDS_BYTES, "LDS");
    asm volatile("s_barrier" ::: "memory");
    if (tid < 16) *(unsigned*)(smem + ARR0 + tid * 4) = 0;
    STAMP(1);

    int vslot, vdue;
    unsigned todo;
    {
        const int pos = min(foreign ? rep + 16 * lane : rep + 8 * (lane - 1), 127);
        vslot = pos < 0 ? (int)((p.extra[xcd >> 2] >> (8 * (xcd & 3))) & 255) : list_slot(rank, pos);
        todo = (unsigned)__ballot(foreign ? lane < 8 && !held_by(vslot, sl >> 1) : lane < 17 && (lane > 0 || xcd < NF));
        const int c4 = 4 * pos - min(max(pos - ND, 0), NF) - min(max(pos - NRB - 3 * D1, 0), 3 * NF);
        vdue = ((8 + NF) * ROUND) / 10 + ((c4 + 2 * (sl & 1)) * YSLOT) / 200;
    }
    auto pop = [&]() { const int i = todo ? __builtin_ctz(todo) : -1; todo &= todo - 1; return i; };
    const char* abase = xws + wave * 8192;
    const char* fbase = myws + WS_FLAG + wave * 4;
    char* ydst = p.ws[other | (sl >> 1)] + WS_Y + (call & 1) * YPAR + (size_t)(me * 32 + (sl & 1) * 16 + rep) * 8 * 3072 + (wave & 3) * 768;
    char* yflag = p.ws[other | (sl >> 1)] + WS_YFLAG + (me * 32 + (sl & 1) * 16 + rep) * 128;
    float* obase = (foreign ? p.out[other | (sl >> 1)] + me * 128 + (sl & 1) * 64 : p.out[rank] + sl * 64) + (wave & 3) * 4 * NL;
    const unsigned l16 = lane * 16;
    const int hsw = (sl & 1) * 4096;
    const unsigned ast = lbase + AST0 + wave * 8192, fst = lbase + FST0 + wave * 256;

    auto issue = [&](int m, int mn) {
        const char* s = abase + (size_t)__builtin_amdgcn_readlane(vslot, m) * RBB;
        const unsigned fo = (unsigned)WS_FLAG + wave * 4 + __builtin_amdgcn_readlane(vslot, max(mn, 0)) * 128;
        const unsigned f = lane == 0 ? fo : (unsigned)WS_TMO;
        unsigned keep;
        asm volatile("s_mov_b32 %0, m0\n\ts_mov_b32 m0, %6\n\ts_nop 4\n\t"
                     "global_load_lds_dwordx4 %1, %2 sc0 sc1\n\tglobal_load_lds_dwordx4 %1, %2 offset:1024 sc0 sc1\n\t"
                     "global_load_lds_dwordx4 %1, %2 offset:2048 sc0 sc1\n\tglobal_load_lds_dwordx4 %1, %2 offset:3072 sc0 sc1\n\t"
                     "s_mov_b32 m0, %7\n\ts_nop 0\n\t"
                     "global_load_lds_dwordx4 %1, %3 sc0 sc1\n\tglobal_load_lds_dwordx4 %1, %3 offset:1024 sc0 sc1\n\t"
                     "global_load_lds_dwordx4 %1, %3 offset:2048 sc0 sc1\n\tglobal_load_lds_dwordx4 %1, %3 offset:3072 sc0 sc1\n\t"
                     "s_mov_b32 m0, %8\n\ts_nop 0\n\t"
                     "global_load_lds_dword %4, %5 sc0 sc1\n\ts_mov_b32 m0, %0"
                     : "=&s"(keep) : "v"(l16), "s"(s + hsw), "s"(s + (hsw ^ 4096)), "v"(f), "s"(myws), "s"(ast + hsw), "s"(ast + (hsw ^ 4096)), "s"(fst) : "memory");
    };
    int pend = -1;
    unsigned pendv = 0;
    auto publish = [&]() {
        if (foreign && lane == 0) {
            const unsigned now = lds_add(lbase + ARR0 + pend * 4, pendv) + pendv;
            if ((now & 15) == 4) __hip_atomic_store((GLB_AS unsigned*)(yflag + pend * 4), call * 16 + (now >> 4), __ATOMIC_RELAXED, __HIP_MEMORY_SCOPE_SYSTEM);
        }
        pend = -1;
    };
#define SEG(x) do {} while (0)
    int m = pop(), mn = pop(), nt = 0;
    wait_flag(fbase + __builtin_amdgcn_readlane(vslot, m) * 128, call);
    issue(m, mn);
#pragma unroll 1
    for (; m >= 0; ++nt) {
        SEG(0);
        if (pend >= 0) WAITV(1); else WAITV(0);
        SEG(1);
        i32x4 a[8];
        i32x2 fv;
        asm volatile("ds_read_b64 %8, %10\n\t"
                     "ds_read_b128 %0, %9\n\tds_read_b128 %1, %9 offset:1024\n\tds_read_b128 %2, %9 offset:2048\n\tds_read_b128 %3, %9 offset:3072\n\t"
                     "ds_read_b128 %4, %9 offset:4096\n\tds_read_b128 %5, %9 offset:5120\n\tds_read_b128 %6, %9 offset:6144\n\tds_read_b128 %7, %9 offset:7168\n\t"
                     "s_waitcnt lgkmcnt(0)"
                     : "=&v"(a[0]), "=&v"(a[1]), "=&v"(a[2]), "=&v"(a[3]), "=&v"(a[4]), "=&v"(a[5]), "=&v"(a[6]), "=&v"(a[7]), "=&v"(fv)
                     : "v"(ast + l16), "v"(fst) : "memory");
        if ((unsigned)__builtin_amdgcn_readfirstlane(fv[1]) == call) bad = true;
        bool pf = mn >= 0 && (unsigned)__builtin_amdgcn_readfirstlane(fv[0]) >= call;
        const int mnn = pop();
        char* pb = smem + (nt & 1) * PBUF;
        auto compute = [&]() {
            i32x4 acc[4];
#pragma unroll
            for (int t = 0; t < 4; ++t) {
                acc[t] = (i32x4){0, 0, 0, 0};
#pragma unroll
                for (int s = 0; s < 8; ++s) acc[t] = __builtin_amdgcn_mfma_i32_16x16x64_i8(bf[s][t], a[s], acc[t], 0, 0, 0);
            }
#pragma unroll
            for (int t = 0; t < 4; ++t)
                *(i32x4*)(pb + wave * 4096 + (lane & 15) * 256 + (((4 * t + g) ^ (lane & 15)) << 4)) = acc[t];
            if (lane == 0) *(int*)(pb + 32768 + wave * 4) = bad;
        };
        if (pend >= 0) {
            issue(max(mn, 0), mnn);
            SEG(2);
            compute();
            SEG(3);
            WAITV(9);
            publish();
        } else {
            if (pf) issue(mn, mnn);
            SEG(2);
            compute();
            SEG(3);
        }
        SEG(4);
        asm volatile("s_waitcnt lgkmcnt(0)\n\ts_barrier" ::: "memory");
        SEG(5);
        if ((wave >> 2) == (nt & 1)) {
            const int row = (wave & 3) * 4 + g;
            const char* rp = pb + row * 256 + (((lane & 15) ^ row) << 4);
#pragma unroll
            for (int v = 0; v < 8; ++v) a[v] = *(const i32x4*)(rp + v * 4096);
            const i32x4 b0 = *(const i32x4*)(pb + 32768), b1 = *(const i32x4*)(pb + 32784);
            const i32x4 sum = ((a[0] + a[1]) + (a[2] + a[3])) + ((a[4] + a[5]) + (a[6] + a[7]));
            const bool anybad = (b0[0] | b0[1] | b0[2] | b0[3] | b1[0] | b1[1] | b1[2] | b1[3]) != 0;
            const unsigned big = ((unsigned)sum[0] + 0x800000u) | ((unsigned)sum[1] + 0x800000u) | ((unsigned)sum[2] + 0x800000u) | ((unsigned)sum[3] + 0x800000u);
            const bool raw = !foreign || rep + 16 * m >= RAWPOS || __any((big >> 24) || anybad);
            u32x4 o;
            char* op;
            unsigned ooff = l16;
            if (raw) {
                const float sc = anybad ? __builtin_nanf("") : scale;
#pragma unroll
                for (int e = 0; e < 4; ++e) o[e] = __builtin_bit_cast(unsigned, (float)sum[e] * sc);
                op = (char*)(obase + (size_t)__builtin_amdgcn_readlane(vslot, m) * 16 * NL);
                ooff = g * (NL * 4) + (lane & 15) * 16;
            } else {
                const unsigned s0 = sum[0], s1 = sum[1], s2 = sum[2], s3 = sum[3];
                const unsigned pk = lbase + PACK0 + wave * 768;
                asm volatile("ds_write_b32 %1, %2\n\tds_write_b32 %1, %3 offset:256\n\tds_write_b32 %1, %4 offset:512\n\t"
                             "ds_read_b128 %0, %5\n\ts_waitcnt lgkmcnt(0)"
                             : "=&v"(o) : "v"(pk + lane * 4), "v"((s0 & 0xffffffu) | (s1 << 24)), "v"(((s1 >> 8) & 0xffffu) | (s2 << 16)),
                               "v"(((s2 >> 16) & 0xffu) | (s3 << 8)), "v"(pk + l16) : "memory");
                op = ydst + m * 3072;
            }
            if (foreign) {
                const int due = __builtin_amdgcn_readlane(vdue, m);
                while ((int)((unsigned)__builtin_amdgcn_s_memrealtime() - ts) < due) {
                    if (pf || mn < 0) { __builtin_amdgcn_s_sleep(1); continue; }
                    const unsigned v = __hip_atomic_load((GLB_AS unsigned*)(fbase + __builtin_amdgcn_readlane(vslot, mn) * 128), __ATOMIC_RELAXED, __HIP_MEMORY_SCOPE_SYSTEM);
                    if ((unsigned)__builtin_amdgcn_readfirstlane(v) >= call) { pf = true; issue(mn, mnn); }
                }
            }
            if (raw || lane < 48) asm volatile("s_nop 4\n\tglobal_store_dwordx4 %0, %1, %2 sc0 sc1\n\ts_nop 1" :: "v"(ooff), "v"(o), "s"(op) : "memory");
            pend = m;
            pendv = 1 + (raw ? 16u << (wave & 3) : 0u);
            if (!pf) { WAITV(0); publish(); }
        }
        if (mn >= 0 && !pf) {
            wait_flag(fbase + __builtin_amdgcn_readlane(vslot, mn) * 128, call);
            issue(mn, mnn);
        }
        m = mn; mn = mnn;
    }
    if (tid == 0) __hip_atomic_store(cnt, call, __ATOMIC_RELAXED, __HIP_MEMORY_SCOPE_AGENT);
}

extern "C" void kernel_launch(int rank, int nranks, ncclComm_t comm,
                              hipStream_t stream,
                              void* const* d_in, const int* in_sizes, int n_in,
                              void* d_out, int out_size,
                              const int* in_dtypes, int out_dtype,
                              void* const* d_out_all,
                              void* const* d_ws_all, size_t ws_bytes) {
    if (nranks != NR || (unsigned)rank >= (unsigned)NR || n_in != 4 || ws_bytes < WS_NEED
        || in_sizes[0] != ML * KDIM || in_sizes[1] != KDIM * NL || out_size != NR * ML * NL) {
        fprintf(stderr, "ag_gemm: unexpected problem shape\n");
        return;
    }
    HIP_CHECK(hipFuncSetAttribute((const void*)ag_gemm, hipFuncAttributeMaxDynamicSharedMemorySize, LDS_BYTES));
    Params p;
    p.x = (const char*)d_in[0]; p.w = (const char*)d_in[1];
    p.sx = (const float*)d_in[2]; p.sw = (const float*)d_in[3];
    for (int i = 0; i < NR; ++i) { p.ws[i] = (char*)d_ws_all[i]; p.out[i] = (float*)d_out_all[i]; }
    p.rank = rank; p.pad = 0;
    const int me = rank & 3, other = (rank & 4) ^ 4;
    p.extra[0] = p.extra[1] = 0;
    int ne = 0;
    for (int k = 0; k < 4; ++k) p.fin[k] = p.fout[k] = 0;
    for (int f = 0; f < NF; ++f) {
        p.fin[(me + offs(f)) & 3] |= 1u << f;
        for (int m = 0; m < 4; ++m) {
            if (((m + offs(f)) & 3) != me) continue;
            for (int k = 0; k < 4; ++k) if (k != m) p.fout[k] |= 1u << (4 * f + m);
            p.extra[ne >> 2] |= (unsigned)((other | m) * NRB + ND + f) << (8 * (ne & 3));
            ++ne;
        }
    }
    hipLaunchKernelGGL(ag_gemm, dim3(GRID), dim3(512), LDS_BYTES, stream, p);
    HIP_CHECK(hipGetLastError());
}
```

```cpp
#include <hip/hip_runtime.h>
#include <rccl/rccl.h>
#include <cstdio>
#include <cstdint>

#define HIP_CHECK(expr) do { hipError_t _e = (expr); if (_e != hipSuccess) { \
    fprintf(stderr, "HIP error %s at %s:%d\n", hipGetErrorString(_e), __FILE__, __LINE__); \
    return; } } while (0)

typedef unsigned u32x4 __attribute__((ext_vector_type(4)));
typedef int i32x4 __attribute__((ext_vector_type(4)));
typedef int i32x2 __attribute__((ext_vector_type(2)));
typedef float f32x4 __attribute__((ext_vector_type(4)));
#define LDS_AS __attribute__((address_space(3)))
#define GLB_AS __attribute__((address_space(1)))

#define NF 7
#define D1 13
#define ROUND 1120
#define LEAD 125
#define SLEAD 10
#define SORD (i + 1 < nround ? (i * 4 + wq) * 8 + j : (i * 8 + ((j + pi) & 7)) * 4 + wq)
#define GAP 115
#define HINTR 4
#define HINTMODE 1
#define YLEAD 10
#define GSLEEP 0
#define YROT 112
#define HOLDMIN 50
#define YSLOT 545
#define LAZY 8
#define WAKE 2
#define NSGL 3
#define NRAW 6

constexpr int NR = 8;
constexpr int KDIM = 4096;
constexpr int ML = 512;
constexpr int NL = 512;
constexpr int NRB = ML / 16;
constexpr int ND = NRB - NF;
constexpr int RBB = 16 * KDIM;
constexpr unsigned OFFS = NF == 5 ? 0x20321u : NF == 6 ? 0x310321u : 0x03210321u;
__host__ __device__ constexpr int offs(int f) { return (OFFS >> (4 * f)) & 3; }
static_assert(NF >= 4 && NF <= 8 && D1 <= ND, "plan");
constexpr int GEMM0 = (NR - 1) * 8;
constexpr int GRID = GEMM0 + 8 * 24;
constexpr int WPB = 8;
constexpr int LDS_BYTES = 148 << 10;
constexpr int NSLOT = NR * NRB + 64;
constexpr size_t WS_CNT = 0;
constexpr size_t WS_TMO = 48 << 10;
constexpr size_t WS_ZERO = 50 << 10;
constexpr size_t WS_HINT = 52 << 10;
constexpr size_t WS_FLAG = 256 << 10;
constexpr size_t WS_YFLAG = 128 << 10;
constexpr size_t WS_X = 1 << 20;
static_assert(WS_FLAG + NSLOT * 1024 <= WS_X, "flags");
constexpr size_t XPAR = (size_t)NSLOT * RBB;
constexpr size_t WS_Y = WS_X + 2 * XPAR;
constexpr int YT = 8;
constexpr size_t YPAR = (size_t)4 * 32 * YT * 3072;
constexpr size_t WS_DIAG = WS_Y + 2 * YPAR;
constexpr size_t WS_NEED = WS_DIAG + (4 << 20);
constexpr unsigned long long DEADLINE_TICKS = 30000000ull;

struct Link {
    char* ws;
    unsigned fin;
    unsigned fout;
};
struct Params {
    const char* x; const char* w; char* myws;
    int rank; int pad;
    Link link[NR - 1];
    const float* sx; const float* sw;
    float* out; float* pout[4];
    unsigned extra[2];
};

__device__ __forceinline__ int list_slot(int r, int pos) {
    if (pos < NRB) return r * NRB + pos;
    const int q = pos - NRB, t = q / 3, s = q - 3 * t;
    const int rb = t < D1 ? t : (t < D1 + NF ? ND + t - D1 : t - NF);
    return ((r & 4) | ((r + 1 + s) & 3)) * NRB + rb;
}
__device__ __forceinline__ bool held_by(int slot, int k) {
    const int rb = slot & (NRB - 1);
    return rb >= ND && (((slot >> 5) + offs(rb - ND)) & 3) == k;
}

#define WAITV(n) asm volatile("s_waitcnt vmcnt(" #n ")" ::: "memory")

#define STAMP(idx) do {} while (0)

__global__ void __launch_bounds__(512, 2) ag_gemm(Params p) {
    extern __shared__ __attribute__((aligned(16))) char smem[];
    const int tid = threadIdx.x, lane = tid & 63;
    const int wave = __builtin_amdgcn_readfirstlane(tid >> 6);
    const int bid = blockIdx.x;
    const unsigned long long t0 = __builtin_amdgcn_s_memrealtime();
    unsigned ts = (unsigned)t0 + LEAD;
    const Link lk = p.link[bid < GEMM0 ? bid >> 3 : 3 + (((bid - GEMM0) >> 4) & 3)];
    const char* const px = p.x; const char* const pw = p.w;
    const int rank = p.rank, me = rank & 3, other = (rank & 4) ^ 4;
    char* myws = p.myws;
    asm volatile("" :: "s"(lk.ws), "s"(lk.fin), "s"(px), "s"(pw), "s"(myws), "s"(rank));
    GLB_AS unsigned* cnt = (GLB_AS unsigned*)(myws + WS_CNT + (size_t)bid * 128);
    GLB_AS unsigned* tmo = (GLB_AS unsigned*)(myws + WS_TMO);
    const unsigned lbase = (unsigned)(size_t)(LDS_AS char*)smem;
    unsigned call;
    bool bad = false;
    auto wait_flag = [&](const char* f, unsigned want, bool lazy = false, int n = 1) -> unsigned {
        GLB_AS unsigned* fp = lane < n ? (GLB_AS unsigned*)f + lane * 32 : tmo;
        for (int it = 1; !bad; ++it) {
            unsigned fv = 0;
            if (lane < n + ((it & 7) == 0)) fv = __hip_atomic_load(fp, __ATOMIC_RELAXED, __HIP_MEMORY_SCOPE_SYSTEM);
            if ((~__ballot(fv >= want) & ((1ull << n) - 1)) == 0) return __builtin_amdgcn_readfirstlane(fv);
            if (__builtin_amdgcn_readlane(fv, n) == call) { bad = true; break; }
            if (__builtin_amdgcn_s_memrealtime() - t0 > DEADLINE_TICKS) {
                if (lane == 0) __hip_atomic_store(tmo, call, __ATOMIC_RELAXED, __HIP_MEMORY_SCOPE_AGENT);
                bad = true;
                break;
            }
            if (lazy) __builtin_amdgcn_s_sleep(LAZY); else __builtin_amdgcn_s_sleep(1);
        }
        return 0;
    };
    auto pace = [&](int due) { while ((int)((unsigned)__builtin_amdgcn_s_memrealtime() - ts) < due) __builtin_amdgcn_s_sleep(1); };
    auto snooze = [&](int due) { while ((int)((unsigned)__builtin_amdgcn_s_memrealtime() - ts) < due) __builtin_amdgcn_s_sleep(8); };
    auto lds_add = [&](unsigned addr, unsigned v) {
        unsigned old;
        asm volatile("ds_add_rtn_u32 %0, %1, %2\n\ts_waitcnt lgkmcnt(0)" : "=&v"(old) : "v"(addr), "v"(v) : "memory");
        return old;
    };

    const int g = lane >> 4;
    if (__builtin_expect(bid < GEMM0, 1)) {
        const int j = bid & 7, pi = bid >> 3;
        const bool cross = pi >= 3;
        const int k = (pi - 3) & 3;
        const int nround = cross ? 8 + NF : ND;
        const int wq = wave & 3, frag = j * 8 + wq * 2;
        int vslot;
        {
            const unsigned fin = lk.fin, fout = lk.fout;
            const int nin = __builtin_popcount(fin);
            unsigned m = lane < nin ? fin : fout;
            const int c = lane < nin ? lane : lane - nin - 8;
#pragma unroll
            for (int t = 0; t < NF - 1; ++t) m &= m - (c > t);
            const int b = __builtin_ctz(m | 0x80000000u);
            vslot = !cross ? rank * NRB + (lane & 31) : lane < nin ? rank * NRB + ND + b
                  : lane < nin + 8 ? NR * NRB + me * 8 + lane - nin : (other | (b & 3)) * NRB + min(ND + (b >> 2), NRB - 1);
        }
        const unsigned stg = lbase + 1024 + wave * 4096;
        unsigned fseen = 0;
        auto fetch = [&](int i) {
            const int slot = __builtin_amdgcn_readlane(vslot, i);
            const int kind = (slot >> 7) - (other >> 2);
            const char* base; unsigned o0, o1;
            if (slot >= NR * NRB) {
                const int r = slot & 7;
                base = pw + (size_t)(((r & 3) * 8 + j) * 128 + wq * 32) * NL + k * 128 + (r >> 2) * 64;
                o0 = (lane >> 2) * NL + (((lane & 3) ^ (g & 1)) << 4);
                o1 = (o0 ^ 32) + 16 * NL;
            } else if (kind != 0) {
                base = px + (size_t)(slot & 31) * RBB + frag * 64;
                o0 = (lane >> 3) * KDIM + (((lane & 7) ^ g) << 4);
                o1 = (o0 ^ 64) + 8 * KDIM;
            } else {
                if (__builtin_amdgcn_readlane(fseen, i) < call) wait_flag(myws + WS_FLAG + slot * 1024 + j * 128, call);
                base = myws + WS_X + (call & 1) * XPAR + (size_t)slot * RBB + frag * 1024;
                o0 = lane * 16; o1 = o0 + 1024;
            }
            const unsigned d = stg + ((i >> 1) & 1) * 2048;
            unsigned keep;
            asm volatile("s_mov_b32 %0, m0\n\ts_mov_b32 m0, %4\n\ts_cmp_eq_u32 %5, 0\n\ts_nop 4\n\ts_cbranch_scc1 1f\n\t"
                         "global_load_lds_dwordx4 %1, %3\n\ts_add_u32 m0, m0, 1024\n\ts_nop 0\n\t"
                         "global_load_lds_dwordx4 %2, %3\n\ts_branch 2f\n"
                         "1:\n\tglobal_load_lds_dwordx4 %1, %3 sc0 sc1\n\ts_add_u32 m0, m0, 1024\n\ts_nop 0\n\t"
                         "global_load_lds_dwordx4 %2, %3 sc0 sc1\n"
                         "2:\n\ts_mov_b32 m0, %0"
                         : "=&s"(keep) : "v"(o0), "v"(o1), "s"(base), "s"(d), "s"(kind) : "memory", "scc");
        };
        call = 0;
        fetch(wave >> 2);
        {
            unsigned c0;
            asm volatile("global_load_dword %0, %1, off sc1\n\t"
                         "ds_write_b32 %2, %3\n\ts_waitcnt lgkmcnt(0)\n\ts_barrier\n\ts_waitcnt vmcnt(0)"
                         : "=&v"(c0) : "v"(cnt), "v"(lbase + 4 * lane), "v"(0u) : "memory");
            call = __builtin_amdgcn_readfirstlane(c0) + 1;
        }
        ts = (unsigned)__builtin_amdgcn_s_memrealtime() + SLEAD;
        int early = 0;
        if (wave >= 4) fetch(3);
        char* dst = lk.ws + WS_X + (call & 1) * XPAR + frag * 1024;
        char* flag = lk.ws + WS_FLAG + j * 128;
        const unsigned l16 = lane * 16;
#pragma unroll 1
        for (int i = wave >> 2; i < nround; i += 2) {
            const int slot = __builtin_amdgcn_readlane(vslot, i);
            const bool own = (slot >> 5) == rank;
            const unsigned r0 = stg + ((i >> 1) & 1) * 2048 + (own ? (lane & 15) * 128 + ((g ^ ((lane & 15) >> 1)) << 4) : l16);
            u32x4 v0, v1;
            if (slot < NR * NRB) {
                asm volatile("ds_read_b128 %0, %2\n\tds_read_b128 %1, %3\n\ts_waitcnt lgkmcnt(0)"
                             : "=&v"(v0), "=&v"(v1) : "v"(r0), "v"(own ? r0 ^ 64 : r0 + 1024) : "memory");
            } else {
                const int q = (lane & 15) >> 1;
                const unsigned rt = stg + ((i >> 1) & 1) * 2048 + (16 * (g & 1) + q) * 64 + 8 * (lane & 1) + (((g >> 1) ^ (q >> 2) ^ ((g & 1) << 1)) << 4);
                i32x2 a, b, c, d;
                asm volatile("ds_read_b64_tr_b8 %0, %4\n\tds_read_b64_tr_b8 %1, %4 offset:512\n\t"
                             "ds_read_b64_tr_b8 %2, %5\n\tds_read_b64_tr_b8 %3, %5 offset:512\n\ts_waitcnt lgkmcnt(0)"
                             : "=&v"(a), "=&v"(b), "=&v"(c), "=&v"(d) : "v"(rt), "v"(rt ^ 32) : "memory");
                v0 = (u32x4){(unsigned)a[0], (unsigned)a[1], (unsigned)b[0], (unsigned)b[1]};
                v1 = (u32x4){(unsigned)c[0], (unsigned)c[1], (unsigned)d[0], (unsigned)d[1]};
            }
            pace(max((SORD * ROUND) / 320, early));
            asm volatile("global_store_dwordx4 %0, %2, %1 sc0 sc1\n\tglobal_store_dwordx4 %0, %3, %1 offset:1024 sc0 sc1\n\ts_nop 1"
                         :: "v"(l16), "s"(dst + (size_t)slot * RBB), "v"(v0), "v"(v1) : "memory");
            if (i == 0) { STAMP(40); fetch(2); }
            if (cross && (i >> 1) == HINTR && lane == 0)
                __hip_atomic_store((GLB_AS unsigned long long*)(myws + WS_HINT) + k * 64 + j * 8 + wave,
                                   ((unsigned long long)call << 32) | (ts + max(early - (SORD * ROUND) / 320, 0)), __ATOMIC_RELAXED, __HIP_MEMORY_SCOPE_AGENT);
            if (cross && (i >> 1) == 2 && lane < 16) fseen = __hip_atomic_load((GLB_AS unsigned*)(myws + WS_FLAG + (vslot & 255) * 1024 + j * 128), __ATOMIC_RELAXED, __HIP_MEMORY_SCOPE_SYSTEM);
            asm volatile("s_waitcnt vmcnt(0)" : "+v"(fseen) :: "memory");
            early = i + 3 < nround ? (int)((unsigned)__builtin_amdgcn_s_memrealtime() - ts) + GAP : 0;
            if (lane == 0 && !bad && lds_add(lbase + 4 * i, 1) == 3) {
                __hip_atomic_store((GLB_AS unsigned*)(flag + slot * 1024), call, __ATOMIC_RELAXED, __HIP_MEMORY_SCOPE_SYSTEM);
            }
            if (i + 4 < nround) fetch(i + 4);
        }
        if (lane == 0 && lds_add(lbase + 4 * 63, 1) == WPB - 1) __hip_atomic_store(cnt, call, __ATOMIC_RELAXED, __HIP_MEMORY_SCOPE_AGENT);
        if (!cross) return;

        const float scale = p.sx[0] * p.sw[0];
        const int prod = j * 4 + wq, h = prod >> 4, rep = prod & 15, e0 = (wave >> 2) * 2;
        const char* yf = myws + WS_YFLAG + (k * 32 + prod) * 128;
        const char* yb = myws + WS_Y + (call & 1) * YPAR + (size_t)(k * 32 + prod) * YT * 3072 + e0 * 768 + lane * 4;
        const int yslot = list_slot(other | k, rep + 16 * (lane & 7));
        unsigned todo = (unsigned)__ballot(lane < 8 && !held_by(yslot, me));
#pragma unroll 1
        for (; todo; todo &= todo - 1) {
            const int n = __builtin_ctz(todo), slot = __builtin_amdgcn_readlane(yslot, n);
            const unsigned raw = (wait_flag(yf + n * 4, call * 16) >> e0) & 3;
            STAMP(44 + n);
            if (raw == 3) continue;
            unsigned d[6];
            asm volatile("global_load_dword %0, %6, off sc0 sc1\n\tglobal_load_dword %1, %6, off offset:256 sc0 sc1\n\t"
                         "global_load_dword %2, %6, off offset:512 sc0 sc1\n\tglobal_load_dword %3, %6, off offset:768 sc0 sc1\n\t"
                         "global_load_dword %4, %6, off offset:1024 sc0 sc1\n\tglobal_load_dword %5, %6, off offset:1280 sc0 sc1\n\t"
                         "s_waitcnt vmcnt(0)"
                         : "=&v"(d[0]), "=&v"(d[1]), "=&v"(d[2]), "=&v"(d[3]), "=&v"(d[4]), "=&v"(d[5]) : "v"(yb + n * 3072) : "memory");
            const float sc = bad ? __builtin_nanf("") : scale;
            float* op = p.out + (size_t)(slot * 16 + e0 * 4 + g) * NL + k * 128 + h * 64 + (lane & 15) * 4;
#pragma unroll
            for (int e = 0; e < 2; ++e) {
                if ((raw >> e) & 1) continue;
                const unsigned d0 = d[3 * e], d1 = d[3 * e + 1], d2 = d[3 * e + 2];
                f32x4 o;
                o[0] = (float)((int)(d0 << 8) >> 8) * sc;
                o[1] = (float)((int)(((d0 >> 16) & 0xff00u) | (d1 << 16)) >> 8) * sc;
                o[2] = (float)((int)(((d1 >> 8) & 0xffff00u) | (d2 << 24)) >> 8) * sc;
                o[3] = (float)((int)d2 >> 8) * sc;
                asm volatile("global_store_dwordx4 %0, %1, off sc0 sc1\n\ts_nop 1" :: "v"(op + e * 4 * NL), "v"(o) : "memory");
            }
        }
        return;
    }

    const int gb = bid - GEMM0;
    const int xcd = gb & 7, sl = (gb >> 3) & 7, set = gb >> 6;
    const bool foreign = set > 0;
    const int rep = foreign ? xcd + 8 * (set - 1) : xcd;
    if (GSLEEP) __builtin_amdgcn_s_sleep(GSLEEP);
    const unsigned c0 = __hip_atomic_load(cnt, __ATOMIC_RELAXED, __HIP_MEMORY_SCOPE_AGENT);
    const float scale = p.sx[0] * p.sw[0];

    constexpr int PTILE = 16 * 272, PBUF = 8 * PTILE, PACK0 = 4 * PBUF, ARR0 = PACK0 + 8 * 768, BAD0 = ARR0 + 32;
    static_assert(ARR0 + 256 + 64 <= LDS_BYTES, "LDS");
    const unsigned l16 = lane * 16;
    const unsigned ast = lbase + wave * 4096;

    if (foreign) {
        const int c = gb - 64, sn = c >> 2, frag = (c & 3) * 16 + wave * 2;
        const char* xs = px + (size_t)sn * RBB + frag * 64;
        const char* wsrc = pw + (size_t)((sn & 3) * 1024 + frag * 16) * NL + (sn >> 2) * 64;
        const unsigned xo = (lane >> 3) * KDIM + (((lane & 7) ^ g) << 4), wo = (lane >> 2) * NL + (((lane & 3) ^ (g & 1)) << 4);
        const int q = (lane & 15) >> 1;
        const unsigned rx = ast + (lane & 15) * 128 + ((g ^ q) << 4);
        const unsigned rt = ast + 2048 + (16 * (g & 1) + q) * 64 + 8 * (lane & 1) + (((g >> 1) ^ (q >> 2) ^ ((g & 1) << 1)) << 4);
        u32x4 v0, v1; i32x2 t0, t1, t2, t3;
        unsigned keep;
        asm volatile("s_mov_b32 %6, m0\n\ts_mov_b32 m0, %13\n\ts_nop 4\n\t"
                     "global_load_lds_dwordx4 %7, %11\n\ts_add_u32 m0, m0, 1024\n\ts_nop 0\n\t"
                     "global_load_lds_dwordx4 %8, %11\n\ts_add_u32 m0, m0, 1024\n\ts_nop 0\n\t"
                     "global_load_lds_dwordx4 %9, %12\n\ts_add_u32 m0, m0, 1024\n\ts_nop 0\n\t"
                     "global_load_lds_dwordx4 %10, %12\n\ts_mov_b32 m0, %6\n\ts_waitcnt vmcnt(0)\n\t"
                     "ds_read_b128 %0, %14\n\tds_read_b128 %1, %15\n\t"
                     "ds_read_b64_tr_b8 %2, %16\n\tds_read_b64_tr_b8 %3, %16 offset:512\n\t"
                     "ds_read_b64_tr_b8 %4, %17\n\tds_read_b64_tr_b8 %5, %17 offset:512\n\ts_waitcnt lgkmcnt(0)"
                     : "=&v"(v0), "=&v"(v1), "=&v"(t0), "=&v"(t1), "=&v"(t2), "=&v"(t3), "=&s"(keep)
                     : "v"(xo), "v"((xo ^ 64) + 8 * KDIM), "v"(wo), "v"((wo ^ 32) + 16 * NL), "s"(xs), "s"(wsrc), "s"(ast),
                       "v"(rx), "v"(rx ^ 64), "v"(rt), "v"(rt ^ 32) : "memory", "scc");
        call = __builtin_amdgcn_readfirstlane(c0) + 1;
        const char* xw = myws + WS_X + (call & 1) * XPAR + frag * 1024;
        const i32x4 v2 = {t0[0], t0[1], t1[0], t1[1]}, v3 = {t2[0], t2[1], t3[0], t3[1]};
        asm volatile("global_store_dwordx4 %0, %5, %2 sc0 sc1\n\tglobal_store_dwordx4 %0, %6, %2 offset:1024 sc0 sc1\n\t"
                     "global_store_dwordx4 %0, %3, %1 sc0 sc1\n\tglobal_store_dwordx4 %0, %4, %1 offset:1024 sc0 sc1\n\t"
                     "s_waitcnt vmcnt(0)\n\ts_barrier"
                     :: "v"(l16), "s"(xw + (size_t)(rank * NRB + sn) * RBB), "s"(xw + (size_t)(NR * NRB + 32 + sn) * RBB),
                        "v"(v0), "v"(v1), "v"(v2), "v"(v3) : "memory");
        if (tid < 4) __hip_atomic_store((GLB_AS unsigned*)(myws + WS_FLAG + ((tid & 2 ? NR * NRB + 32 : rank * NRB) + sn) * 1024 + ((c & 3) * 2 + (tid & 1)) * 128),
                                        call, __ATOMIC_RELAXED, __HIP_MEMORY_SCOPE_SYSTEM);
        STAMP(2);
    }
    call = __builtin_amdgcn_readfirstlane(c0) + 1;
    const char* xws = myws + WS_X + (call & 1) * XPAR;
    if (tid < 16) *(unsigned*)(smem + ARR0 + tid * 4) = 0;
    asm volatile("s_waitcnt lgkmcnt(0)\n\ts_barrier" ::: "memory");

    unsigned vinfo, vflag;
    int nunit;
    {
        const int pos = min(foreign ? rep + 16 * lane : rep + 8 * (lane - (lane > 4)), 127);
        const int slot = !foreign && lane == 4 ? (int)((p.extra[xcd >> 2] >> (8 * (xcd & 3))) & 255) : list_slot(rank, pos);
        const bool use = foreign ? lane < 8 && !held_by(slot, sl >> 1) : lane < 17 && (lane != 4 || xcd < NF);
        const unsigned mask = (unsigned)__ballot(use);
        const int pr = pos < YROT ? (pos & ~7) | ((pos + (sl & 6)) & 7) : pos;
        const int c4 = 4 * pr - min(max(pr - ND, 0), NF) - min(max(pr - NRB - 3 * D1, 0), 3 * NF);
        const int due = ((8 + NF) * ROUND) / 10 + ((c4 + 2 * (sl & 1)) * YSLOT) / 200;
        nunit = __builtin_popcount(mask);
        vinfo = __builtin_amdgcn_ds_permute((use ? __builtin_amdgcn_mbcnt_lo(mask, 0) : 63) * 4, slot | (lane & 7) << 8 | due << 12);
        vflag = lane < nunit ? (unsigned)WS_FLAG + wave * 128 + (vinfo & 255) * 1024 : (unsigned)WS_ZERO;
    }
    const unsigned vaoff = (vinfo & 255) << 16;
    char* ydst = lk.ws + WS_Y + (call & 1) * YPAR + (size_t)(me * 32 + (sl & 1) * 16 + rep) * YT * 3072 + (wave & 3) * 768;
    char* yflag = lk.ws + WS_YFLAG + (me * 32 + (sl & 1) * 16 + rep) * 128;
    float* obase = (foreign ? p.pout[sl >> 1] + me * 128 + (sl & 1) * 64 : p.out + sl * 64) + (wave & 3) * 4 * NL;

    int hold = -1, hdue = 0;
    unsigned holdv = 0;
    u32x4 o = {0, 0, 0, 0};
    char* op = ydst;
    unsigned ooff = l16;
    int u = 0;
    int pend = -1;
    auto emit = [&]() {
        if (holdv > 1 || lane < 48) asm volatile("s_nop 4\n\tglobal_store_dwordx4 %0, %1, %2 sc0 sc1\n\ts_nop 1" :: "v"(ooff), "v"(o), "s"(op) : "memory");
        pend = hold; hold = -1;
    };
    auto publish = [&]() {
        WAITV(0);
        if (lane == 0) {
            const unsigned now = lds_add(lbase + ARR0 + pend * 4, holdv) + holdv;
            if ((now & 15) == 4) __hip_atomic_store((GLB_AS unsigned*)(yflag + pend * 4), call * 16 + (now >> 4), __ATOMIC_RELAXED, __HIP_MEMORY_SCOPE_SYSTEM);
        }
        pend = -1;
    };

    i32x4 bf[8][4];
    {
        const int wslot = NR * NRB + (foreign ? 0 : 32) + sl * 4 + (wave >> 1);
        if (foreign) snooze(((WAKE + (sl & 1) * 4 + (wave >> 1)) * ROUND) / 10);
        wait_flag(myws + WS_FLAG + wslot * 1024 + (wave & 1) * 512, call, foreign, 4);
        asm volatile("" ::: "memory");
        const __amdgpu_buffer_rsrc_t rs = __builtin_amdgcn_make_buffer_rsrc((void*)(xws + (size_t)wslot * RBB + (wave & 1) * 32768), 0, 32768, 0x00020000);
#pragma unroll
        for (int s = 0; s < 8; ++s)
#pragma unroll
            for (int t = 0; t < 4; ++t)
                bf[s][t] = __builtin_bit_cast(i32x4, __builtin_amdgcn_raw_buffer_load_b128(rs, (lane >> 5) * 2048 + (lane & 31) * 16 + t * 512, s * 4096, 17));
#pragma unroll
        for (int s = 0; s < 8; ++s) asm volatile("" : "+v"(bf[s][0]), "+v"(bf[s][1]), "+v"(bf[s][2]), "+v"(bf[s][3]));
    }
    if (foreign) {
        snooze(((2 * HINTR + 3) * ROUND) / 10);
        const unsigned long long h = __hip_atomic_load((GLB_AS unsigned long long*)(myws + WS_HINT) + (sl >> 1) * 64 + lane, __ATOMIC_RELAXED, __HIP_MEMORY_SCOPE_AGENT);
        int d = (unsigned)(h >> 32) == call ? (int)((unsigned)h - (unsigned)t0) : LEAD;
#pragma unroll
        for (int o = 32; o; o >>= 1) d += __shfl_xor(d, o);
        d >>= 6;
        ts = (unsigned)t0 + __builtin_amdgcn_readfirstlane(min(max(d, 0), 1000)) + YLEAD;
    }
    STAMP(1);
    i32x4 a[8];
#pragma unroll
    for (int s = 0; s < 8; ++s) a[s] = (i32x4){0, 0, 0, 0};
    unsigned fv = 0, fa = (unsigned)WS_TMO;
    u32x4 ars;
    ars[0] = (unsigned)(size_t)xws; ars[1] = (unsigned)((size_t)xws >> 32) & 0xffff; ars[2] = (unsigned)XPAR; ars[3] = 0x00020000;
    const unsigned alo = wave * 8192 + l16, ahi = alo + 4096;
    const unsigned pw0 = lbase + wave * PTILE + (lane & 15) * 272 + g * 16;
    const unsigned pr0 = lbase + ((wave & 3) * 4 + g) * 272 + (lane & 15) * 16;
    auto poison_all = [&]() { asm volatile("ds_write_b32 %0, %1" :: "v"(lbase + BAD0), "v"(1) : "memory"); };
    constexpr int RUN = 3, LAST = 9, FETCH = 5;
    int md = LAST;
    bool paired = false;
#pragma unroll 1
    while (u < nunit) {
        asm volatile("s_waitcnt vmcnt(7)" : "+v"(a[0]), "+v"(fv) :: "memory");
        int un;
        if (__builtin_expect(md != LAST, 1)) {
            md = __builtin_amdgcn_readlane(fv, 0) >= call ? RUN : LAST;
            if (__builtin_expect(__builtin_amdgcn_readlane(fv, 1) == call, 0)) poison_all();
            un = u + 1;
        } else {
            const char* f = myws + __builtin_amdgcn_readlane(vflag, u);
            while (hold >= 0) {
                unsigned v = 0;
                if (lane == 0) v = __hip_atomic_load((GLB_AS unsigned*)f, __ATOMIC_RELAXED, __HIP_MEMORY_SCOPE_SYSTEM);
                if ((unsigned)__builtin_amdgcn_readfirstlane(v) >= call) break;
                if ((int)((unsigned)__builtin_amdgcn_s_memrealtime() - ts) >= hdue) emit();
            }
            if (pend >= 0) publish();
            if (hold < 0) wait_flag(f, call);
            if (bad) poison_all();
            md = FETCH;
            un = u;
        }
        const unsigned soff = __builtin_amdgcn_readlane(vaoff, un);
        asm volatile("s_nop 1\n\tv_writelane_b32 %1, %3, 0\n\tglobal_load_dword %0, %1, %2 sc0 sc1"
                     : "+v"(fv), "+v"(fa) : "s"(myws), "s"(__builtin_amdgcn_readlane(vflag, un + 1)), "s"(soff) : "memory");
        i32x4 acc[4];
        asm volatile("s_cmp_lg_u32 %28, 3\n\ts_cbranch_scc1 1f\n\t"
                     "v_mfma_i32_16x16x64_i8 %0, %9, %4, 0\n\tv_mfma_i32_16x16x64_i8 %1, %10, %4, 0\n\tv_mfma_i32_16x16x64_i8 %2, %11, %4, 0\n\tv_mfma_i32_16x16x64_i8 %3, %12, %4, 0\n\tbuffer_load_dwordx4 %4, %25, %26, %27 offen offset:0 sc0 sc1\n\ts_waitcnt vmcnt(8)\n\t"
                     "v_mfma_i32_16x16x64_i8 %0, %13, %5, %0\n\tv_mfma_i32_16x16x64_i8 %1, %14, %5, %1\n\tv_mfma_i32_16x16x64_i8 %2, %15, %5, %2\n\tv_mfma_i32_16x16x64_i8 %3, %16, %5, %3\n\tbuffer_load_dwordx4 %5, %25, %26, %27 offen offset:1024 sc0 sc1\n\ts_waitcnt vmcnt(8)\n\t"
                     "v_mfma_i32_16x16x64_i8 %0, %17, %6, %0\n\tv_mfma_i32_16x16x64_i8 %1, %18, %6, %1\n\tv_mfma_i32_16x16x64_i8 %2, %19, %6, %2\n\tv_mfma_i32_16x16x64_i8 %3, %20, %6, %3\n\tbuffer_load_dwordx4 %6, %25, %26, %27 offen offset:2048 sc0 sc1\n\ts_waitcnt vmcnt(8)\n\t"
                     "v_mfma_i32_16x16x64_i8 %0, %21, %7, %0\n\tv_mfma_i32_16x16x64_i8 %1, %22, %7, %1\n\tv_mfma_i32_16x16x64_i8 %2, %23, %7, %2\n\tv_mfma_i32_16x16x64_i8 %3, %24, %7, %3\n\tbuffer_load_dwordx4 %7, %25, %26, %27 offen offset:3072 sc0 sc1\n\ts_waitcnt vmcnt(8)\n\t"
                     "s_branch 3f\n1:\n\ts_cmp_lg_u32 %28, 9\n\ts_cbranch_scc1 2f\n\t"
                     "v_mfma_i32_16x16x64_i8 %0, %9, %4, 0\n\tv_mfma_i32_16x16x64_i8 %1, %10, %4, 0\n\tv_mfma_i32_16x16x64_i8 %2, %11, %4, 0\n\tv_mfma_i32_16x16x64_i8 %3, %12, %4, 0\n\ts_waitcnt vmcnt(7)\n\t"
                     "v_mfma_i32_16x16x64_i8 %0, %13, %5, %0\n\tv_mfma_i32_16x16x64_i8 %1, %14, %5, %1\n\tv_mfma_i32_16x16x64_i8 %2, %15, %5, %2\n\tv_mfma_i32_16x16x64_i8 %3, %16, %5, %3\n\ts_waitcnt vmcnt(6)\n\t"
                     "v_mfma_i32_16x16x64_i8 %0, %17, %6, %0\n\tv_mfma_i32_16x16x64_i8 %1, %18, %6, %1\n\tv_mfma_i32_16x16x64_i8 %2, %19, %6, %2\n\tv_mfma_i32_16x16x64_i8 %3, %20, %6, %3\n\ts_waitcnt vmcnt(5)\n\t"
                     "v_mfma_i32_16x16x64_i8 %0, %21, %7, %0\n\tv_mfma_i32_16x16x64_i8 %1, %22, %7, %1\n\tv_mfma_i32_16x16x64_i8 %2, %23, %7, %2\n\tv_mfma_i32_16x16x64_i8 %3, %24, %7, %3\n\ts_waitcnt vmcnt(4)\n\t"
                     "s_branch 3f\n2:\n\tbuffer_load_dwordx4 %4, %25, %26, %27 offen offset:0 sc0 sc1\n\tbuffer_load_dwordx4 %5, %25, %26, %27 offen offset:1024 sc0 sc1\n\tbuffer_load_dwordx4 %6, %25, %26, %27 offen offset:2048 sc0 sc1\n\tbuffer_load_dwordx4 %7, %25, %26, %27 offen offset:3072 sc0 sc1\n\t"
                     "\n3:\n\t"
                     : "=&v"(acc[0]), "=&v"(acc[1]), "=&v"(acc[2]), "=&v"(acc[3]), "+v"(a[0]), "+v"(a[1]), "+v"(a[2]), "+v"(a[3]), "+v"(a[4])
                     : "v"(bf[0][0]), "v"(bf[0][1]), "v"(bf[0][2]), "v"(bf[0][3]), "v"(bf[1][0]), "v"(bf[1][1]), "v"(bf[1][2]), "v"(bf[1][3]), "v"(bf[2][0]), "v"(bf[2][1]), "v"(bf[2][2]), "v"(bf[2][3]), "v"(bf[3][0]), "v"(bf[3][1]), "v"(bf[3][2]), "v"(bf[3][3]),
                       "v"(alo), "s"(ars), "s"(soff), "s"(md) : "memory", "scc");
        asm volatile("s_cmp_lg_u32 %28, 3\n\ts_cbranch_scc1 1f\n\t"
                     "v_mfma_i32_16x16x64_i8 %0, %9, %4, %0\n\tv_mfma_i32_16x16x64_i8 %1, %10, %4, %1\n\tv_mfma_i32_16x16x64_i8 %2, %11, %4, %2\n\tv_mfma_i32_16x16x64_i8 %3, %12, %4, %3\n\tbuffer_load_dwordx4 %4, %25, %26, %27 offen offset:0 sc0 sc1\n\ts_waitcnt vmcnt(8)\n\t"
                     "v_mfma_i32_16x16x64_i8 %0, %13, %5, %0\n\tv_mfma_i32_16x16x64_i8 %1, %14, %5, %1\n\tv_mfma_i32_16x16x64_i8 %2, %15, %5, %2\n\tv_mfma_i32_16x16x64_i8 %3, %16, %5, %3\n\tbuffer_load_dwordx4 %5, %25, %26, %27 offen offset:1024 sc0 sc1\n\ts_waitcnt vmcnt(8)\n\t"
                     "v_mfma_i32_16x16x64_i8 %0, %17, %6, %0\n\tv_mfma_i32_16x16x64_i8 %1, %18, %6, %1\n\tv_mfma_i32_16x16x64_i8 %2, %19, %6, %2\n\tv_mfma_i32_16x16x64_i8 %3, %20, %6, %3\n\tbuffer_load_dwordx4 %6, %25, %26, %27 offen offset:2048 sc0 sc1\n\ts_waitcnt vmcnt(8)\n\t"
                     "v_mfma_i32_16x16x64_i8 %0, %21, %7, %0\n\tv_mfma_i32_16x16x64_i8 %1, %22, %7, %1\n\tv_mfma_i32_16x16x64_i8 %2, %23, %7, %2\n\tv_mfma_i32_16x16x64_i8 %3, %24, %7, %3\n\tbuffer_load_dwordx4 %7, %25, %26, %27 offen offset:3072 sc0 sc1\n\t"
                     "s_branch 3f\n1:\n\ts_cmp_lg_u32 %28, 9\n\ts_cbranch_scc1 2f\n\t"
                     "v_mfma_i32_16x16x64_i8 %0, %9, %4, %0\n\tv_mfma_i32_16x16x64_i8 %1, %10, %4, %1\n\tv_mfma_i32_16x16x64_i8 %2, %11, %4, %2\n\tv_mfma_i32_16x16x64_i8 %3, %12, %4, %3\n\ts_waitcnt vmcnt(3)\n\t"
                     "v_mfma_i32_16x16x64_i8 %0, %13, %5, %0\n\tv_mfma_i32_16x16x64_i8 %1, %14, %5, %1\n\tv_mfma_i32_16x16x64_i8 %2, %15, %5, %2\n\tv_mfma_i32_16x16x64_i8 %3, %16, %5, %3\n\ts_waitcnt vmcnt(2)\n\t"
                     "v_mfma_i32_16x16x64_i8 %0, %17, %6, %0\n\tv_mfma_i32_16x16x64_i8 %1, %18, %6, %1\n\tv_mfma_i32_16x16x64_i8 %2, %19, %6, %2\n\tv_mfma_i32_16x16x64_i8 %3, %20, %6, %3\n\ts_waitcnt vmcnt(1)\n\t"
                     "v_mfma_i32_16x16x64_i8 %0, %21, %7, %0\n\tv_mfma_i32_16x16x64_i8 %1, %22, %7, %1\n\tv_mfma_i32_16x16x64_i8 %2, %23, %7, %2\n\tv_mfma_i32_16x16x64_i8 %3, %24, %7, %3\n\t"
                     "s_branch 3f\n2:\n\tbuffer_load_dwordx4 %4, %25, %26, %27 offen offset:0 sc0 sc1\n\tbuffer_load_dwordx4 %5, %25, %26, %27 offen offset:1024 sc0 sc1\n\tbuffer_load_dwordx4 %6, %25, %26, %27 offen offset:2048 sc0 sc1\n\tbuffer_load_dwordx4 %7, %25, %26, %27 offen offset:3072 sc0 sc1\n\t"
                     "\n3:\n\ts_nop 7"
                     : "+v"(acc[0]), "+v"(acc[1]), "+v"(acc[2]), "+v"(acc[3]), "+v"(a[4]), "+v"(a[5]), "+v"(a[6]), "+v"(a[7]), "+v"(a[0])
                     : "v"(bf[4][0]), "v"(bf[4][1]), "v"(bf[4][2]), "v"(bf[4][3]), "v"(bf[5][0]), "v"(bf[5][1]), "v"(bf[5][2]), "v"(bf[5][3]), "v"(bf[6][0]), "v"(bf[6][1]), "v"(bf[6][2]), "v"(bf[6][3]), "v"(bf[7][0]), "v"(bf[7][1]), "v"(bf[7][2]), "v"(bf[7][3]),
                       "v"(ahi), "s"(ars), "s"(soff), "s"(md) : "memory", "scc");
        if (md == FETCH) continue;
        const unsigned pb = (u & 3) * PBUF;
        asm volatile("ds_write_b128 %0, %1\n\tds_write_b128 %0, %2 offset:64\n\tds_write_b128 %0, %3 offset:128\n\tds_write_b128 %0, %4 offset:192"
                     :: "v"(pw0 + pb), "v"(acc[0]), "v"(acc[1]), "v"(acc[2]), "v"(acc[3]) : "memory");
        ++u;
        if (!foreign && (u & 1) && u < nunit - NSGL) { paired = true; continue; }
        if (hold >= 0) { pace(hdue); emit(); }
        asm volatile("s_waitcnt lgkmcnt(0)\n\ts_barrier" ::: "memory");
        if (pend >= 0) publish();
        const bool mine = (wave >> 2) != (u & 1);
        if (mine || paired) {
            const unsigned info = __builtin_amdgcn_readlane(vinfo, mine ? u - 1 : u - 2);
            i32x4 r[8];
            int poison;
            asm volatile("ds_read_b128 %0, %9\n\tds_read_b128 %1, %9 offset:%11\n\tds_read_b128 %2, %9 offset:2*%11\n\tds_read_b128 %3, %9 offset:3*%11\n\t"
                         "ds_read_b128 %4, %9 offset:4*%11\n\tds_read_b128 %5, %9 offset:5*%11\n\tds_read_b128 %6, %9 offset:6*%11\n\tds_read_b128 %7, %9 offset:7*%11\n\t"
                         "ds_read_b32 %8, %10\n\ts_waitcnt lgkmcnt(0)"
                         : "=&v"(r[0]), "=&v"(r[1]), "=&v"(r[2]), "=&v"(r[3]), "=&v"(r[4]), "=&v"(r[5]), "=&v"(r[6]), "=&v"(r[7]), "=&v"(poison)
                         : "v"(pr0 + (mine ? pb : pb - PBUF)), "v"(lbase + BAD0), "n"(PTILE) : "memory");
            const i32x4 sum = ((r[0] + r[1]) + (r[2] + r[3])) + ((r[4] + r[5]) + (r[6] + r[7]));
            const float sc = poison ? __builtin_nanf("") : scale;
            char* const orow = (char*)obase + (size_t)(info & 255) * (16 * NL * 4);
            const unsigned roff = g * (NL * 4) + (lane & 15) * 16;
            if (!foreign) {
                f32x4 y;
#pragma unroll
                for (int e = 0; e < 4; ++e) y[e] = (float)sum[e] * sc;
                asm volatile("global_store_dwordx4 %0, %1, %2 sc0 sc1\n\ts_nop 1" :: "v"(roff), "v"(y), "s"(orow) : "memory");
            } else {
                hold = (info >> 8) & 7;
                hdue = info >> 12;
                const unsigned big = ((unsigned)sum[0] + 0x800000u) | ((unsigned)sum[1] + 0x800000u) | ((unsigned)sum[2] + 0x800000u) | ((unsigned)sum[3] + 0x800000u);
                if (rep + 16 * hold >= 128 - NRAW || __any((big >> 24) | poison)) {
#pragma unroll
                    for (int e = 0; e < 4; ++e) o[e] = __builtin_bit_cast(unsigned, (float)sum[e] * sc);
                    op = orow; ooff = roff;
                    holdv = 1 + (16u << (wave & 3));
                } else {
                    const unsigned s0 = sum[0], s1 = sum[1], s2 = sum[2], s3 = sum[3];
                    const unsigned pk = lbase + PACK0 + wave * 768;
                    asm volatile("ds_write_b32 %1, %2\n\tds_write_b32 %1, %3 offset:256\n\tds_write_b32 %1, %4 offset:512\n\t"
                                 "ds_read_b128 %0, %5\n\ts_waitcnt lgkmcnt(0)"
                                 : "=&v"(o) : "v"(pk + lane * 4), "v"((s0 & 0xffffffu) | (s1 << 24)), "v"(((s1 >> 8) & 0xffffu) | (s2 << 16)),
                                   "v"(((s2 >> 16) & 0xffu) | (s3 << 8)), "v"(pk + l16) : "memory");
                    op = ydst + hold * 3072; ooff = l16;
                    holdv = 1;
                }
                if (u >= nunit || (int)((unsigned)__builtin_amdgcn_s_memrealtime() - ts) + HOLDMIN >= hdue) { pace(hdue); emit(); }
            }
        }
        paired = false;
    }
    asm volatile("s_waitcnt vmcnt(0)" : "+v"(a[0]), "+v"(a[1]), "+v"(a[2]), "+v"(a[3]), "+v"(a[4]), "+v"(a[5]), "+v"(a[6]), "+v"(a[7]), "+v"(fv) :: "memory");
    if (pend >= 0) publish();
    if (tid == 0) __hip_atomic_store(cnt, call, __ATOMIC_RELAXED, __HIP_MEMORY_SCOPE_AGENT);
}

extern "C" void kernel_launch(int rank, int nranks, ncclComm_t comm,
                              hipStream_t stream,
                              void* const* d_in, const int* in_sizes, int n_in,
                              void* d_out, int out_size,
                              const int* in_dtypes, int out_dtype,
                              void* const* d_out_all,
                              void* const* d_ws_all, size_t ws_bytes) {
    if (nranks != NR || (unsigned)rank >= (unsigned)NR || n_in != 4 || ws_bytes < WS_NEED
        || in_sizes[0] != ML * KDIM || in_sizes[1] != KDIM * NL || out_size != NR * ML * NL) {
        fprintf(stderr, "ag_gemm: unexpected problem shape\n");
        return;
    }
    HIP_CHECK(hipFuncSetAttribute((const void*)ag_gemm, hipFuncAttributeMaxDynamicSharedMemorySize, LDS_BYTES));
    Params p;
    p.x = (const char*)d_in[0]; p.w = (const char*)d_in[1];
    p.sx = (const float*)d_in[2]; p.sw = (const float*)d_in[3];
    p.rank = rank; p.pad = 0;
    const int me = rank & 3, other = (rank & 4) ^ 4;
    p.myws = (char*)d_ws_all[rank]; p.out = (float*)d_out;
    for (int i = 0; i < NR - 1; ++i) {
        p.link[i].ws = (char*)d_ws_all[i < 3 ? (rank & 4) | ((me + 1 + i) & 3) : other | (i - 3)];
        p.link[i].fin = p.link[i].fout = 0;
    }
    for (int k = 0; k < 4; ++k) p.pout[k] = (float*)d_out_all[other | k];
    p.extra[0] = p.extra[1] = 0;
    int ne = 0;
    for (int f = 0; f < NF; ++f) {
        p.link[3 + ((me + offs(f)) & 3)].fin |= 1u << f;
        for (int m = 0; m < 4; ++m) {
            if (((m + offs(f)) & 3) != me) continue;
            for (int k = 0; k < 4; ++k) if (k != m) p.link[3 + k].fout |= 1u << (4 * f + m);
            p.extra[ne >> 2] |= (unsigned)((other | m) * NRB + ND + f) << (8 * (ne & 3));
            ++ne;
        }
    }
    hipLaunchKernelGGL(ag_gemm, dim3(GRID), dim3(512), LDS_BYTES, stream, p);
    HIP_CHECK(hipGetLastError());
}
```

```cpp
#include <hip/hip_runtime.h>
#include <rccl/rccl.h>
#include <cstdio>
#include <cstdint>

#define HIP_CHECK(expr) do { hipError_t _e = (expr); if (_e != hipSuccess) { \
    fprintf(stderr, "HIP error %s at %s:%d\n", hipGetErrorString(_e), __FILE__, __LINE__); \
    return; } } while (0)

typedef unsigned u32x4 __attribute__((ext_vector_type(4)));
typedef int i32x4 __attribute__((ext_vector_type(4)));
typedef int i32x2 __attribute__((ext_vector_type(2)));
typedef float f32x4 __attribute__((ext_vector_type(4)));
#define LDS_AS __attribute__((address_space(3)))
#define GLB_AS __attribute__((address_space(1)))

#define NF 7
#define D1 13
#define ROUND 1112
#define LEAD 125
#define SLEAD 10
#define SORD (i + 1 < nround ? (i * 4 + wq) * 8 + j : (i * 8 + ((j + pi) & 7)) * 4 + wq)
#define GAP 115
#define HINTR 4
#define HINTMODE 1
#define YLEAD 10
#define GSLEEP 0
#define YROT 112
#define HOLDMIN 50
#define PSLEEP 1
#define LASTX 4
#define YSLOT 492
#define LAZY 8
#define WAKE 2
#define NSGL 3
#define NRAW 12

constexpr int NR = 8;
constexpr int KDIM = 4096;
constexpr int ML = 512;
constexpr int NL = 512;
constexpr int NRB = ML / 16;
constexpr int ND = NRB - NF;
constexpr int RBB = 16 * KDIM;
constexpr unsigned OFFS = NF == 5 ? 0x20321u : NF == 6 ? 0x310321u : 0x03210321u;
__host__ __device__ constexpr int offs(int f) { return (OFFS >> (4 * f)) & 3; }
static_assert(NF >= 4 && NF <= 8 && D1 <= ND, "plan");
constexpr int GEMM0 = (NR - 1) * 8;
constexpr int GRID = GEMM0 + 8 * 24;
constexpr int WPB = 8;
constexpr int LDS_BYTES = 148 << 10;
constexpr int NSLOT = NR * NRB + 64;
constexpr size_t WS_CNT = 0;
constexpr size_t WS_TMO = 48 << 10;
constexpr size_t WS_ZERO = 50 << 10;
constexpr size_t WS_HINT = 52 << 10;
constexpr size_t WS_FLAG = 256 << 10;
constexpr size_t WS_YFLAG = 128 << 10;
constexpr size_t WS_X = 1 << 20;
static_assert(WS_FLAG + NSLOT * 1024 <= WS_X, "flags");
constexpr size_t XPAR = (size_t)NSLOT * RBB;
constexpr size_t WS_Y = WS_X + 2 * XPAR;
constexpr int YT = 8;
constexpr int YPART = 704, YTILE = 4 * YPART;
constexpr size_t YPAR = (size_t)4 * 32 * YT * YTILE;
constexpr size_t WS_DIAG = WS_Y + 2 * YPAR;
constexpr size_t WS_NEED = WS_DIAG + (4 << 20);
constexpr unsigned long long DEADLINE_TICKS = 30000000ull;

struct Link {
    char* ws;
    unsigned first;
    unsigned fin;
    unsigned fout;
    unsigned pad[3];
};
struct Params {
    const char* x; const char* w; char* myws;
    int rank; int pad;
    Link link[NR - 1];
    const float* sx; const float* sw;
    float* out; float* pout[4];
    unsigned extra[2];
};

__device__ __forceinline__ int list_slot(int r, int pos) {
    if (pos < NRB) return r * NRB + pos;
    const int q = pos - NRB, t = q / 3, s = q - 3 * t;
    const int rb = t < D1 ? t : (t < D1 + NF ? ND + t - D1 : t - NF);
    return ((r & 4) | ((r + 1 + s) & 3)) * NRB + rb;
}
__device__ __forceinline__ bool held_by(int slot, int k) {
    const int rb = slot & (NRB - 1);
    return rb >= ND && (((slot >> 5) + offs(rb - ND)) & 3) == k;
}

#define WAITV(n) asm volatile("s_waitcnt vmcnt(" #n ")" ::: "memory")

#define EXITSTAMP() do {} while (0)
#define STAMP(idx) do {} while (0)

__global__ void __launch_bounds__(512, 2) ag_gemm(Params p) {
    extern __shared__ __attribute__((aligned(16))) char smem[];
    const int tid = threadIdx.x, lane = tid & 63;
    const int wave = __builtin_amdgcn_readfirstlane(tid >> 6);
    const int bid = blockIdx.x;
    const unsigned long long t0 = __builtin_amdgcn_s_memrealtime();
    unsigned ts = (unsigned)t0 + LEAD;
    const Link lk = p.link[bid < GEMM0 ? bid >> 3 : 3 + (((bid - GEMM0) >> 4) & 3)];
    const char* const px = p.x; const char* const pw = p.w;
    const int rank = p.rank, me = rank & 3, other = (rank & 4) ^ 4;
    char* myws = p.myws;
    asm volatile("" :: "s"(lk.ws), "s"(lk.first), "s"(lk.fout), "s"(px), "s"(pw), "s"(myws), "s"(rank));
    GLB_AS unsigned* cnt = (GLB_AS unsigned*)(myws + WS_CNT + (size_t)bid * 128);
    GLB_AS unsigned* tmo = (GLB_AS unsigned*)(myws + WS_TMO);
    const unsigned lbase = (unsigned)(size_t)(LDS_AS char*)smem;
    unsigned call;
    bool bad = false;
    auto wait_flag = [&](const char* f, unsigned want, bool lazy = false, int n = 1) -> unsigned {
        const unsigned long long ex = (1ull << n) - 1;
        while (!bad) {
#pragma unroll 1
            for (int it = 0; it < 8; ++it) {
                unsigned fv;
                asm volatile("s_mov_b64 exec, %3\n\tglobal_load_dword %0, %1, %2 sc0 sc1\n\ts_mov_b64 exec, -1\n\ts_waitcnt vmcnt(0)"
                             : "=&v"(fv) : "v"(lane * 128), "s"(f), "s"(ex) : "memory");
                if ((__ballot(fv < want) & ex) == 0) return __builtin_amdgcn_readfirstlane(fv);
                if (lazy) __builtin_amdgcn_s_sleep(LAZY); else if (PSLEEP) __builtin_amdgcn_s_sleep(PSLEEP);
            }
            if (__builtin_amdgcn_readfirstlane(__hip_atomic_load(tmo, __ATOMIC_RELAXED, __HIP_MEMORY_SCOPE_AGENT)) == call) bad = true;
            else if (__builtin_amdgcn_s_memrealtime() - t0 > DEADLINE_TICKS) {
                if (lane == 0) __hip_atomic_store(tmo, call, __ATOMIC_RELAXED, __HIP_MEMORY_SCOPE_AGENT);
                bad = true;
            }
        }
        return 0;
    };
    auto pace = [&](int due) { while ((int)((unsigned)__builtin_amdgcn_s_memrealtime() - ts) < due) __builtin_amdgcn_s_sleep(1); };
    auto snooze = [&](int due) { while ((int)((unsigned)__builtin_amdgcn_s_memrealtime() - ts) < due) __builtin_amdgcn_s_sleep(8); };
    auto lds_add = [&](unsigned addr, unsigned v) {
        unsigned old;
        asm volatile("ds_add_rtn_u32 %0, %1, %2\n\ts_waitcnt lgkmcnt(0)" : "=&v"(old) : "v"(addr), "v"(v) : "memory");
        return old;
    };

    const int g = lane >> 4;
    if (__builtin_expect(bid < GEMM0, 1)) {
        const int j = bid & 7, pi = bid >> 3;
        const bool cross = pi >= 3;
        const int k = (pi - 3) & 3;
        const int nround = cross ? 8 + NF : ND;
        const int wq = wave & 3, frag = j * 8 + wq * 2;
        const unsigned stg = lbase + 1024 + wave * 4096;
        const unsigned xo = (lane >> 3) * KDIM + (((lane & 7) ^ g) << 4), wo = (lane >> 2) * NL + (((lane & 3) ^ (g & 1)) << 4);
        auto fetch = [&](int slot, int i) {
            const bool isw = slot >= NR * NRB;
            const char* base = isw ? pw + (size_t)(((slot & 3) * 8 + j) * 128 + wq * 32) * NL + k * 128 + (slot & 4) * 16
                                   : px + (size_t)(slot & 31) * RBB + frag * 64;
            unsigned keep;
            asm volatile("s_mov_b32 %0, m0\n\ts_mov_b32 m0, %4\n\ts_nop 0\n\t"
                         "global_load_lds_dwordx4 %1, %3\n\ts_add_u32 m0, m0, 1024\n\ts_nop 0\n\t"
                         "global_load_lds_dwordx4 %2, %3\n\ts_mov_b32 m0, %0"
                         : "=&s"(keep) : "v"(isw ? wo : xo), "v"(isw ? (wo ^ 32) + 16 * NL : (xo ^ 64) + 8 * KDIM), "s"(base),
                           "s"(stg + ((i >> 1) & 1) * 2048) : "memory", "scc");
        };
        fetch((lk.first >> (16 * (wave >> 2))) & 0xffff, 0);
        unsigned c0;
        asm volatile("global_load_dword %0, %1, %2 sc1\n\tds_write_b32 %3, %1" : "=&v"(c0) : "v"(0u), "s"(cnt), "v"(lbase + 4 * lane) : "memory");
        int vslot;
        {
            const unsigned fin = lk.fin, fout = lk.fout;
            const int nin = __builtin_popcount(fin);
            unsigned m = lane < nin ? fin : fout;
            const int c = lane < nin ? lane : lane - nin - 8;
#pragma unroll
            for (int t = 0; t < NF - 1; ++t) m &= m - (c > t);
            const int b = __builtin_ctz(m | 0x80000000u);
            vslot = !cross ? rank * NRB + (lane & 31) : lane < nin ? rank * NRB + ND + b
                  : lane < nin + 8 ? NR * NRB + me * 8 + lane - nin : (other | (b & 3)) * NRB + min(ND + (b >> 2), NRB - 1);
        }
        unsigned fseen = 0;
        asm volatile("s_waitcnt lgkmcnt(0)\n\ts_barrier" ::: "memory");
        int early = 0;
        unsigned par = 0;
        call = 0;
        char* dst = lk.ws + WS_X + frag * 1024;
        char* flag = lk.ws + WS_FLAG + j * 128;
        const unsigned l16 = lane * 16;
#pragma unroll 1
        for (int i = wave >> 2; i < nround; i += 2) {
            asm volatile("s_cmp_gt_u32 %1, 1\n\ts_cbranch_scc1 1f\n\ts_waitcnt vmcnt(0)\n1:" : "+v"(c0) : "s"(i) : "memory", "scc");
            if (i < 2) {
                call = __builtin_amdgcn_readfirstlane(c0) + 1;
                par = (call & 1) * (unsigned)XPAR;
                ts = (unsigned)__builtin_amdgcn_s_memrealtime() + SLEAD;
            }
            const int slot = __builtin_amdgcn_readlane(vslot, i);
            const bool own = (slot >> 5) == rank;
            const unsigned r0 = stg + ((i >> 1) & 1) * 2048 + (own ? (lane & 15) * 128 + ((g ^ ((lane & 15) >> 1)) << 4) : l16);
            u32x4 v0, v1;
            if (slot < NR * NRB) {
                asm volatile("ds_read_b128 %0, %2\n\tds_read_b128 %1, %3\n\ts_waitcnt lgkmcnt(0)"
                             : "=&v"(v0), "=&v"(v1) : "v"(r0), "v"(own ? r0 ^ 64 : r0 + 1024) : "memory");
            } else {
                const int q = (lane & 15) >> 1;
                const unsigned rt = stg + ((i >> 1) & 1) * 2048 + (16 * (g & 1) + q) * 64 + 8 * (lane & 1) + (((g >> 1) ^ (q >> 2) ^ ((g & 1) << 1)) << 4);
                i32x2 a, b, c, d;
                asm volatile("ds_read_b64_tr_b8 %0, %4\n\tds_read_b64_tr_b8 %1, %4 offset:512\n\t"
                             "ds_read_b64_tr_b8 %2, %5\n\tds_read_b64_tr_b8 %3, %5 offset:512\n\ts_waitcnt lgkmcnt(0)"
                             : "=&v"(a), "=&v"(b), "=&v"(c), "=&v"(d) : "v"(rt), "v"(rt ^ 32) : "memory");
                v0 = (u32x4){(unsigned)a[0], (unsigned)a[1], (unsigned)b[0], (unsigned)b[1]};
                v1 = (u32x4){(unsigned)c[0], (unsigned)c[1], (unsigned)d[0], (unsigned)d[1]};
            }
            pace(max((SORD * ROUND) / 320, early));
            asm volatile("global_store_dwordx4 %0, %2, %1 sc0 sc1\n\tglobal_store_dwordx4 %0, %3, %1 offset:1024 sc0 sc1\n\ts_nop 1"
                         :: "v"(l16), "s"(dst + (par + (unsigned)slot * RBB)), "v"(v0), "v"(v1) : "memory");
            if (i == 0) STAMP(40);
            if (i < 2) fetch(__builtin_amdgcn_readlane(vslot, i + 2), 2);
            if (cross && (i >> 1) == HINTR && lane == 0)
                __hip_atomic_store((GLB_AS unsigned long long*)(myws + WS_HINT) + k * 64 + j * 8 + wave,
                                   ((unsigned long long)call << 32) | (ts + max(early - (SORD * ROUND) / 320, 0)), __ATOMIC_RELAXED, __HIP_MEMORY_SCOPE_AGENT);
            if (cross && (i >> 1) == 2 && lane < 16) fseen = __hip_atomic_load((GLB_AS unsigned*)(myws + WS_FLAG + (vslot & 255) * 1024 + j * 128), __ATOMIC_RELAXED, __HIP_MEMORY_SCOPE_SYSTEM);
            asm volatile("s_waitcnt vmcnt(0)" : "+v"(fseen) :: "memory");
            early = i + 3 < nround ? (int)((unsigned)__builtin_amdgcn_s_memrealtime() - ts) + GAP : 0;
            if (lane == 0 && !bad && lds_add(lbase + 4 * i, 1) == 3) {
                __hip_atomic_store((GLB_AS unsigned*)(flag + slot * 1024), call, __ATOMIC_RELAXED, __HIP_MEMORY_SCOPE_SYSTEM);
            }
            if (i + 4 < nround) {
                const int ns = __builtin_amdgcn_readlane(vslot, i + 4);
                if ((ns >> 7) != (other >> 2)) fetch(ns, i);
                else {
                    if (__builtin_amdgcn_readlane(fseen, i + 4) < call) wait_flag(myws + WS_FLAG + ns * 1024 + j * 128, call);
                    unsigned keep;
                    asm volatile("s_mov_b32 %0, m0\n\ts_mov_b32 m0, %3\n\ts_nop 0\n\tglobal_load_lds_dwordx4 %1, %2 sc0 sc1\n\t"
                                 "global_load_lds_dwordx4 %1, %2 offset:1024 sc0 sc1\n\ts_mov_b32 m0, %0"
                                 : "=&s"(keep) : "v"(l16), "s"(myws + WS_X + frag * 1024 + (par + (unsigned)ns * RBB)),
                                   "s"(stg + ((i >> 1) & 1) * 2048) : "memory");
                }
            }
        }
        if (lane == 0 && lds_add(lbase + 4 * 63, 1) == WPB - 1) __hip_atomic_store(cnt, call, __ATOMIC_RELAXED, __HIP_MEMORY_SCOPE_AGENT);
        if (!cross) { EXITSTAMP(); return; }

        const float scale = p.sx[0] * p.sw[0];
        const int prod = j * 4 + wq, h = prod >> 4, rep = prod & 15, e0 = (wave >> 2) * 2;
        const char* yf = myws + WS_YFLAG + (k * 32 + prod) * 128;
        const char* yb = myws + WS_Y + (call & 1) * YPAR + (size_t)(k * 32 + prod) * YT * YTILE + e0 * YPART;
        const int yslot = list_slot(other | k, (lane == 7 ? (rep + LASTX) & 15 : rep) + 16 * (lane & 7));
        unsigned todo = (unsigned)__ballot(lane < 8 && !held_by(yslot, me));
#pragma unroll 1
        for (; todo; todo &= todo - 1) {
            const int n = __builtin_ctz(todo), slot = __builtin_amdgcn_readlane(yslot, n);
            const unsigned raw = (wait_flag(yf + n * 4, call * 16) >> e0) & 3;
            STAMP(44 + n);
            if (raw == 3) continue;
            unsigned d[8];
            asm volatile("global_load_dword %0, %8, %11 sc0 sc1\n\tglobal_load_dword %1, %8, %11 offset:256 sc0 sc1\n\t"
                         "global_load_ushort %2, %9, %11 offset:512 sc0 sc1\n\tglobal_load_ubyte %3, %10, %11 offset:640 sc0 sc1\n\t"
                         "global_load_dword %4, %8, %11 offset:704 sc0 sc1\n\tglobal_load_dword %5, %8, %11 offset:960 sc0 sc1\n\t"
                         "global_load_ushort %6, %9, %11 offset:1216 sc0 sc1\n\tglobal_load_ubyte %7, %10, %11 offset:1344 sc0 sc1\n\t"
                         "s_waitcnt vmcnt(0)"
                         : "=&v"(d[0]), "=&v"(d[1]), "=&v"(d[2]), "=&v"(d[3]), "=&v"(d[4]), "=&v"(d[5]), "=&v"(d[6]), "=&v"(d[7])
                         : "v"(lane * 4), "v"(lane * 2), "v"(lane), "s"(yb + n * YTILE) : "memory");
            const float sc = bad ? __builtin_nanf("") : scale;
            float* op = p.out + (size_t)(slot * 16 + e0 * 4 + g) * NL + k * 128 + h * 64 + (lane & 15) * 4;
#pragma unroll
            for (int e = 0; e < 2; ++e) {
                if ((raw >> e) & 1) continue;
                const unsigned d0 = d[4 * e], d1 = d[4 * e + 1], hi = d[4 * e + 2] | d[4 * e + 3] << 16;
                f32x4 o;
                o[0] = (float)(((int)(hi << 26) >> 10) | (int)(d0 & 0xffffu)) * sc;
                o[1] = (float)((((int)(hi << 20) >> 26) << 16) | (int)(d0 >> 16)) * sc;
                o[2] = (float)((((int)(hi << 14) >> 26) << 16) | (int)(d1 & 0xffffu)) * sc;
                o[3] = (float)((((int)(hi << 8) >> 26) << 16) | (int)(d1 >> 16)) * sc;
                asm volatile("global_store_dwordx4 %0, %1, off sc0 sc1\n\ts_nop 1" :: "v"(op + e * 4 * NL), "v"(o) : "memory");
            }
        }
        EXITSTAMP();
        return;
    }

    const int gb = bid - GEMM0;
    const int xcd = gb & 7, sl = (gb >> 3) & 7, set = gb >> 6;
    const bool foreign = set > 0;
    const int rep = foreign ? xcd + 8 * (set - 1) : xcd;
    if (GSLEEP) __builtin_amdgcn_s_sleep(GSLEEP);
    const unsigned c0 = __hip_atomic_load(cnt, __ATOMIC_RELAXED, __HIP_MEMORY_SCOPE_AGENT);
    const float scale = p.sx[0] * p.sw[0];

    constexpr int PTILE = 16 * 272, PBUF = 8 * PTILE, PACK0 = 4 * PBUF, ARR0 = PACK0 + 8 * 768, BAD0 = ARR0 + 32;
    static_assert(ARR0 + 256 + 64 <= LDS_BYTES, "LDS");
    const unsigned l16 = lane * 16;
    const unsigned ast = lbase + wave * 4096;

    if (foreign) {
        const int c = gb - 64, sn = c >> 2, frag = (c & 3) * 16 + wave * 2;
        const char* xs = px + (size_t)sn * RBB + frag * 64;
        const char* wsrc = pw + (size_t)((sn & 3) * 1024 + frag * 16) * NL + (sn >> 2) * 64;
        const unsigned xo = (lane >> 3) * KDIM + (((lane & 7) ^ g) << 4), wo = (lane >> 2) * NL + (((lane & 3) ^ (g & 1)) << 4);
        const int q = (lane & 15) >> 1;
        const unsigned rx = ast + (lane & 15) * 128 + ((g ^ q) << 4);
        const unsigned rt = ast + 2048 + (16 * (g & 1) + q) * 64 + 8 * (lane & 1) + (((g >> 1) ^ (q >> 2) ^ ((g & 1) << 1)) << 4);
        u32x4 v0, v1; i32x2 t0, t1, t2, t3;
        unsigned keep;
        asm volatile("s_mov_b32 %6, m0\n\ts_mov_b32 m0, %13\n\ts_nop 4\n\t"
                     "global_load_lds_dwordx4 %7, %11\n\ts_add_u32 m0, m0, 1024\n\ts_nop 0\n\t"
                     "global_load_lds_dwordx4 %8, %11\n\ts_add_u32 m0, m0, 1024\n\ts_nop 0\n\t"
                     "global_load_lds_dwordx4 %9, %12\n\ts_add_u32 m0, m0, 1024\n\ts_nop 0\n\t"
                     "global_load_lds_dwordx4 %10, %12\n\ts_mov_b32 m0, %6\n\ts_waitcnt vmcnt(0)\n\t"
                     "ds_read_b128 %0, %14\n\tds_read_b128 %1, %15\n\t"
                     "ds_read_b64_tr_b8 %2, %16\n\tds_read_b64_tr_b8 %3, %16 offset:512\n\t"
                     "ds_read_b64_tr_b8 %4, %17\n\tds_read_b64_tr_b8 %5, %17 offset:512\n\ts_waitcnt lgkmcnt(0)"
                     : "=&v"(v0), "=&v"(v1), "=&v"(t0), "=&v"(t1), "=&v"(t2), "=&v"(t3), "=&s"(keep)
                     : "v"(xo), "v"((xo ^ 64) + 8 * KDIM), "v"(wo), "v"((wo ^ 32) + 16 * NL), "s"(xs), "s"(wsrc), "s"(ast),
                       "v"(rx), "v"(rx ^ 64), "v"(rt), "v"(rt ^ 32) : "memory", "scc");
        call = __builtin_amdgcn_readfirstlane(c0) + 1;
        const char* xw = myws + WS_X + (call & 1) * XPAR + frag * 1024;
        const i32x4 v2 = {t0[0], t0[1], t1[0], t1[1]}, v3 = {t2[0], t2[1], t3[0], t3[1]};
        asm volatile("global_store_dwordx4 %0, %5, %2 sc0 sc1\n\tglobal_store_dwordx4 %0, %6, %2 offset:1024 sc0 sc1\n\t"
                     "global_store_dwordx4 %0, %3, %1 sc0 sc1\n\tglobal_store_dwordx4 %0, %4, %1 offset:1024 sc0 sc1\n\t"
                     "s_waitcnt vmcnt(0)\n\ts_barrier"
                     :: "v"(l16), "s"(xw + (size_t)(rank * NRB + sn) * RBB), "s"(xw + (size_t)(NR * NRB + 32 + sn) * RBB),
                        "v"(v0), "v"(v1), "v"(v2), "v"(v3) : "memory");
        if (tid < 4) __hip_atomic_store((GLB_AS unsigned*)(myws + WS_FLAG + ((tid & 2 ? NR * NRB + 32 : rank * NRB) + sn) * 1024 + ((c & 3) * 2 + (tid & 1)) * 128),
                                        call, __ATOMIC_RELAXED, __HIP_MEMORY_SCOPE_SYSTEM);
        STAMP(2);
    }
    call = __builtin_amdgcn_readfirstlane(c0) + 1;
    const char* xws = myws + WS_X + (call & 1) * XPAR;
    if (tid < 16) *(unsigned*)(smem + ARR0 + tid * 4) = 0;
    asm volatile("s_waitcnt lgkmcnt(0)\n\ts_barrier" ::: "memory");

    unsigned vinfo, vflag;
    int nunit;
    {
        const int pos = min(foreign ? (lane == 7 ? (rep + LASTX) & 15 : rep) + 16 * lane : rep + 8 * (lane - (lane > 4)), 127);
        const int slot = !foreign && lane == 4 ? (int)((p.extra[xcd >> 2] >> (8 * (xcd & 3))) & 255) : list_slot(rank, pos);
        const bool use = foreign ? lane < 8 && !held_by(slot, sl >> 1) : lane < 17 && (lane != 4 || xcd < NF);
        const unsigned mask = (unsigned)__ballot(use);
        const int pr = pos < YROT ? (pos & ~7) | ((pos + (sl & 6)) & 7) : pos;
        const int c4 = 4 * pr - min(max(pr - ND, 0), NF) - min(max(pr - NRB - 3 * D1, 0), 3 * NF);
        const int due = ((8 + NF) * ROUND) / 10 + ((c4 + 2 * (sl & 1)) * YSLOT) / 200;
        nunit = __builtin_popcount(mask);
        vinfo = __builtin_amdgcn_ds_permute((use ? __builtin_amdgcn_mbcnt_lo(mask, 0) : 63) * 4, slot | (lane & 7) << 8 | (pos >= 128 - NRAW) << 11 | due << 12);
        vflag = lane < nunit ? (unsigned)WS_FLAG + wave * 128 + (vinfo & 255) * 1024 : (unsigned)WS_ZERO;
    }
    const unsigned vaoff = (vinfo & 255) << 16;
    char* ydst = lk.ws + WS_Y + (call & 1) * YPAR + (size_t)(me * 32 + (sl & 1) * 16 + rep) * YT * YTILE + (wave & 3) * YPART;
    char* yflag = lk.ws + WS_YFLAG + (me * 32 + (sl & 1) * 16 + rep) * 128;
    float* obase = (foreign ? p.pout[sl >> 1] + me * 128 + (sl & 1) * 64 : p.out + sl * 64) + (wave & 3) * 4 * NL;

    int hold = -1, hdue = 0;
    unsigned holdv = 0;
    u32x4 o = {0, 0, 0, 0};
    char* op = ydst;
    unsigned ooff = l16;
    int u = 0;
    int pend = -1;
    auto emit = [&]() {
        if (holdv > 1 || lane < YPART / 16) asm volatile("s_nop 4\n\tglobal_store_dwordx4 %0, %1, %2 sc0 sc1\n\ts_nop 1" :: "v"(ooff), "v"(o), "s"(op) : "memory");
        pend = hold; hold = -1;
    };
    auto announce = [&]() {
        if (lane == 0) {
            const unsigned now = lds_add(lbase + ARR0 + pend * 4, holdv) + holdv;
            if ((now & 15) == 4) __hip_atomic_store((GLB_AS unsigned*)(yflag + pend * 4), call * 16 + (now >> 4), __ATOMIC_RELAXED, __HIP_MEMORY_SCOPE_SYSTEM);
        }
        pend = -1;
    };
    auto publish = [&]() { WAITV(0); announce(); };

    i32x4 bf[8][4];
    {
        const int wslot = NR * NRB + (foreign ? 0 : 32) + sl * 4 + (wave >> 1);
        if (foreign) snooze(((WAKE + (sl & 1) * 4 + (wave >> 1)) * ROUND) / 10);
        wait_flag(myws + WS_FLAG + wslot * 1024 + (wave & 1) * 512, call, foreign, 4);
        asm volatile("" ::: "memory");
        const __amdgpu_buffer_rsrc_t rs = __builtin_amdgcn_make_buffer_rsrc((void*)(xws + (size_t)wslot * RBB + (wave & 1) * 32768), 0, 32768, 0x00020000);
#pragma unroll
        for (int s = 0; s < 8; ++s)
#pragma unroll
            for (int t = 0; t < 4; ++t)
                bf[s][t] = __builtin_bit_cast(i32x4, __builtin_amdgcn_raw_buffer_load_b128(rs, (lane >> 5) * 2048 + (lane & 31) * 16 + t * 512, s * 4096, 17));
#pragma unroll
        for (int s = 0; s < 8; ++s) asm volatile("" : "+v"(bf[s][0]), "+v"(bf[s][1]), "+v"(bf[s][2]), "+v"(bf[s][3]));
    }
    if (foreign) {
        snooze(((2 * HINTR + 3) * ROUND) / 10);
        const unsigned long long h = __hip_atomic_load((GLB_AS unsigned long long*)(myws + WS_HINT) + (sl >> 1) * 64 + lane, __ATOMIC_RELAXED, __HIP_MEMORY_SCOPE_AGENT);
        int d = (unsigned)(h >> 32) == call ? (int)((unsigned)h - (unsigned)t0) : LEAD;
#pragma unroll
        for (int o = 32; o; o >>= 1) d += __shfl_xor(d, o);
        d >>= 6;
        ts = (unsigned)t0 + __builtin_amdgcn_readfirstlane(min(max(d, 0), 1000)) + YLEAD;
    }
    STAMP(1);
    i32x4 a[8];
#pragma unroll
    for (int s = 0; s < 8; ++s) a[s] = (i32x4){0, 0, 0, 0};
    unsigned fv = 0, fa = (unsigned)WS_TMO;
    u32x4 ars;
    ars[0] = (unsigned)(size_t)xws; ars[1] = (unsigned)((size_t)xws >> 32) & 0xffff; ars[2] = (unsigned)XPAR; ars[3] = 0x00020000;
    const unsigned alo = wave * 8192 + l16, ahi = alo + 4096;
    const unsigned pw0 = lbase + wave * PTILE + (lane & 15) * 272 + g * 16;
    const unsigned pr0 = lbase + ((wave & 3) * 4 + g) * 272 + (lane & 15) * 16;
    auto poison_all = [&]() { asm volatile("ds_write_b32 %0, %1" :: "v"(lbase + BAD0), "v"(1) : "memory"); };
    constexpr int RUN = 3, LAST = 9, FETCH = 5;
    int md = LAST;
    bool paired = false;
#pragma unroll 1
    while (u < nunit) {
        asm volatile("s_waitcnt vmcnt(7)" : "+v"(a[0]), "+v"(fv) :: "memory");
        int un;
        if (__builtin_expect(md != LAST, 1)) {
            md = __builtin_amdgcn_readlane(fv, 0) >= call ? RUN : LAST;
            if (__builtin_expect(__builtin_amdgcn_readlane(fv, 1) == call, 0)) poison_all();
            un = u + 1;
        } else {
            const char* f = myws + __builtin_amdgcn_readlane(vflag, u);
            bool seen = false;
            while ((hold & pend) >= 0) {
                unsigned v;
                asm volatile("s_mov_b64 exec, 1\n\tglobal_load_dword %0, %1, %2 sc0 sc1\n\ts_mov_b64 exec, -1\n\ts_waitcnt vmcnt(0)"
                             : "=&v"(v) : "v"(0u), "s"(f) : "memory");
                seen = (unsigned)__builtin_amdgcn_readfirstlane(v) >= call;
                if (seen) break;
                if (pend >= 0) announce();
                else if ((int)((unsigned)__builtin_amdgcn_s_memrealtime() - ts) >= hdue) emit();
            }
            if (!seen) wait_flag(f, call);
            if (bad) poison_all();
            md = FETCH;
            un = u;
        }
        const unsigned soff = __builtin_amdgcn_readlane(vaoff, un);
        asm volatile("s_nop 1\n\tv_writelane_b32 %1, %3, 0\n\tglobal_load_dword %0, %1, %2 sc0 sc1"
                     : "+v"(fv), "+v"(fa) : "s"(myws), "s"(__builtin_amdgcn_readlane(vflag, un + 1)), "s"(soff) : "memory");
        i32x4 acc[4];
        asm volatile("s_cmp_lg_u32 %28, 3\n\ts_cbranch_scc1 1f\n\t"
                     "v_mfma_i32_16x16x64_i8 %0, %9, %4, 0\n\tv_mfma_i32_16x16x64_i8 %1, %10, %4, 0\n\tv_mfma_i32_16x16x64_i8 %2, %11, %4, 0\n\tv_mfma_i32_16x16x64_i8 %3, %12, %4, 0\n\tbuffer_load_dwordx4 %4, %25, %26, %27 offen offset:0 sc0 sc1\n\ts_waitcnt vmcnt(8)\n\t"
                     "v_mfma_i32_16x16x64_i8 %0, %13, %5, %0\n\tv_mfma_i32_16x16x64_i8 %1, %14, %5, %1\n\tv_mfma_i32_16x16x64_i8 %2, %15, %5, %2\n\tv_mfma_i32_16x16x64_i8 %3, %16, %5, %3\n\tbuffer_load_dwordx4 %5, %25, %26, %27 offen offset:1024 sc0 sc1\n\ts_waitcnt vmcnt(8)\n\t"
                     "v_mfma_i32_16x16x64_i8 %0, %17, %6, %0\n\tv_mfma_i32_16x16x64_i8 %1, %18, %6, %1\n\tv_mfma_i32_16x16x64_i8 %2, %19, %6, %2\n\tv_mfma_i32_16x16x64_i8 %3, %20, %6, %3\n\tbuffer_load_dwordx4 %6, %25, %26, %27 offen offset:2048 sc0 sc1\n\ts_waitcnt vmcnt(8)\n\t"
                     "v_mfma_i32_16x16x64_i8 %0, %21, %7, %0\n\tv_mfma_i32_16x16x64_i8 %1, %22, %7, %1\n\tv_mfma_i32_16x16x64_i8 %2, %23, %7, %2\n\tv_mfma_i32_16x16x64_i8 %3, %24, %7, %3\n\tbuffer_load_dwordx4 %7, %25, %26, %27 offen offset:3072 sc0 sc1\n\ts_waitcnt vmcnt(8)\n\t"
                     "s_branch 3f\n1:\n\ts_cmp_lg_u32 %28, 9\n\ts_cbranch_scc1 2f\n\t"
                     "v_mfma_i32_16x16x64_i8 %0, %9, %4, 0\n\tv_mfma_i32_16x16x64_i8 %1, %10, %4, 0\n\tv_mfma_i32_16x16x64_i8 %2, %11, %4, 0\n\tv_mfma_i32_16x16x64_i8 %3, %12, %4, 0\n\ts_waitcnt vmcnt(7)\n\t"
                     "v_mfma_i32_16x16x64_i8 %0, %13, %5, %0\n\tv_mfma_i32_16x16x64_i8 %1, %14, %5, %1\n\tv_mfma_i32_16x16x64_i8 %2, %15, %5, %2\n\tv_mfma_i32_16x16x64_i8 %3, %16, %5, %3\n\ts_waitcnt vmcnt(6)\n\t"
                     "v_mfma_i32_16x16x64_i8 %0, %17, %6, %0\n\tv_mfma_i32_16x16x64_i8 %1, %18, %6, %1\n\tv_mfma_i32_16x16x64_i8 %2, %19, %6, %2\n\tv_mfma_i32_16x16x64_i8 %3, %20, %6, %3\n\ts_waitcnt vmcnt(5)\n\t"
                     "v_mfma_i32_16x16x64_i8 %0, %21, %7, %0\n\tv_mfma_i32_16x16x64_i8 %1, %22, %7, %1\n\tv_mfma_i32_16x16x64_i8 %2, %23, %7, %2\n\tv_mfma_i32_16x16x64_i8 %3, %24, %7, %3\n\ts_waitcnt vmcnt(4)\n\t"
                     "s_branch 3f\n2:\n\tbuffer_load_dwordx4 %4, %25, %26, %27 offen offset:0 sc0 sc1\n\tbuffer_load_dwordx4 %5, %25, %26, %27 offen offset:1024 sc0 sc1\n\tbuffer_load_dwordx4 %6, %25, %26, %27 offen offset:2048 sc0 sc1\n\tbuffer_load_dwordx4 %7, %25, %26, %27 offen offset:3072 sc0 sc1\n\t"
                     "\n3:\n\t"
                     : "=&v"(acc[0]), "=&v"(acc[1]), "=&v"(acc[2]), "=&v"(acc[3]), "+v"(a[0]), "+v"(a[1]), "+v"(a[2]), "+v"(a[3]), "+v"(a[4])
                     : "v"(bf[0][0]), "v"(bf[0][1]), "v"(bf[0][2]), "v"(bf[0][3]), "v"(bf[1][0]), "v"(bf[1][1]), "v"(bf[1][2]), "v"(bf[1][3]), "v"(bf[2][0]), "v"(bf[2][1]), "v"(bf[2][2]), "v"(bf[2][3]), "v"(bf[3][0]), "v"(bf[3][1]), "v"(bf[3][2]), "v"(bf[3][3]),
                       "v"(alo), "s"(ars), "s"(soff), "s"(md) : "memory", "scc");
        asm volatile("s_cmp_lg_u32 %28, 3\n\ts_cbranch_scc1 1f\n\t"
                     "v_mfma_i32_16x16x64_i8 %0, %9, %4, %0\n\tv_mfma_i32_16x16x64_i8 %1, %10, %4, %1\n\tv_mfma_i32_16x16x64_i8 %2, %11, %4, %2\n\tv_mfma_i32_16x16x64_i8 %3, %12, %4, %3\n\tbuffer_load_dwordx4 %4, %25, %26, %27 offen offset:0 sc0 sc1\n\ts_waitcnt vmcnt(8)\n\t"
                     "v_mfma_i32_16x16x64_i8 %0, %13, %5, %0\n\tv_mfma_i32_16x16x64_i8 %1, %14, %5, %1\n\tv_mfma_i32_16x16x64_i8 %2, %15, %5, %2\n\tv_mfma_i32_16x16x64_i8 %3, %16, %5, %3\n\tbuffer_load_dwordx4 %5, %25, %26, %27 offen offset:1024 sc0 sc1\n\ts_waitcnt vmcnt(8)\n\t"
                     "v_mfma_i32_16x16x64_i8 %0, %17, %6, %0\n\tv_mfma_i32_16x16x64_i8 %1, %18, %6, %1\n\tv_mfma_i32_16x16x64_i8 %2, %19, %6, %2\n\tv_mfma_i32_16x16x64_i8 %3, %20, %6, %3\n\tbuffer_load_dwordx4 %6, %25, %26, %27 offen offset:2048 sc0 sc1\n\ts_waitcnt vmcnt(8)\n\t"
                     "v_mfma_i32_16x16x64_i8 %0, %21, %7, %0\n\tv_mfma_i32_16x16x64_i8 %1, %22, %7, %1\n\tv_mfma_i32_16x16x64_i8 %2, %23, %7, %2\n\tv_mfma_i32_16x16x64_i8 %3, %24, %7, %3\n\tbuffer_load_dwordx4 %7, %25, %26, %27 offen offset:3072 sc0 sc1\n\t"
                     "s_branch 3f\n1:\n\ts_cmp_lg_u32 %28, 9\n\ts_cbranch_scc1 2f\n\t"
                     "v_mfma_i32_16x16x64_i8 %0, %9, %4, %0\n\tv_mfma_i32_16x16x64_i8 %1, %10, %4, %1\n\tv_mfma_i32_16x16x64_i8 %2, %11, %4, %2\n\tv_mfma_i32_16x16x64_i8 %3, %12, %4, %3\n\ts_waitcnt vmcnt(3)\n\t"
                     "v_mfma_i32_16x16x64_i8 %0, %13, %5, %0\n\tv_mfma_i32_16x16x64_i8 %1, %14, %5, %1\n\tv_mfma_i32_16x16x64_i8 %2, %15, %5, %2\n\tv_mfma_i32_16x16x64_i8 %3, %16, %5, %3\n\ts_waitcnt vmcnt(2)\n\t"
                     "v_mfma_i32_16x16x64_i8 %0, %17, %6, %0\n\tv_mfma_i32_16x16x64_i8 %1, %18, %6, %1\n\tv_mfma_i32_16x16x64_i8 %2, %19, %6, %2\n\tv_mfma_i32_16x16x64_i8 %3, %20, %6, %3\n\ts_waitcnt vmcnt(1)\n\t"
                     "v_mfma_i32_16x16x64_i8 %0, %21, %7, %0\n\tv_mfma_i32_16x16x64_i8 %1, %22, %7, %1\n\tv_mfma_i32_16x16x64_i8 %2, %23, %7, %2\n\tv_mfma_i32_16x16x64_i8 %3, %24, %7, %3\n\t"
                     "s_branch 3f\n2:\n\tbuffer_load_dwordx4 %4, %25, %26, %27 offen offset:0 sc0 sc1\n\tbuffer_load_dwordx4 %5, %25, %26, %27 offen offset:1024 sc0 sc1\n\tbuffer_load_dwordx4 %6, %25, %26, %27 offen offset:2048 sc0 sc1\n\tbuffer_load_dwordx4 %7, %25, %26, %27 offen offset:3072 sc0 sc1\n\t"
                     "\n3:\n\ts_nop 7"
                     : "+v"(acc[0]), "+v"(acc[1]), "+v"(acc[2]), "+v"(acc[3]), "+v"(a[4]), "+v"(a[5]), "+v"(a[6]), "+v"(a[7]), "+v"(a[0])
                     : "v"(bf[4][0]), "v"(bf[4][1]), "v"(bf[4][2]), "v"(bf[4][3]), "v"(bf[5][0]), "v"(bf[5][1]), "v"(bf[5][2]), "v"(bf[5][3]), "v"(bf[6][0]), "v"(bf[6][1]), "v"(bf[6][2]), "v"(bf[6][3]), "v"(bf[7][0]), "v"(bf[7][1]), "v"(bf[7][2]), "v"(bf[7][3]),
                       "v"(ahi), "s"(ars), "s"(soff), "s"(md) : "memory", "scc");
        if (md == FETCH) { if (pend >= 0) announce(); continue; }
        const unsigned pb = (u & 3) * PBUF;
        asm volatile("ds_write_b128 %0, %1\n\tds_write_b128 %0, %2 offset:64\n\tds_write_b128 %0, %3 offset:128\n\tds_write_b128 %0, %4 offset:192"
                     :: "v"(pw0 + pb), "v"(acc[0]), "v"(acc[1]), "v"(acc[2]), "v"(acc[3]) : "memory");
        ++u;
        if (!foreign && (u & 1) && u < nunit - NSGL) { paired = true; continue; }
        if (pend >= 0) publish();
        if (hold >= 0) { pace(hdue); emit(); }
        asm volatile("s_waitcnt lgkmcnt(0)\n\ts_barrier" ::: "memory");
        const bool mine = (wave >> 2) != (u & 1);
        if (mine || paired) {
            const unsigned info = __builtin_amdgcn_readlane(vinfo, mine ? u - 1 : u - 2);
            i32x4 r[8];
            int poison;
            asm volatile("ds_read_b128 %0, %9\n\tds_read_b128 %1, %9 offset:%11\n\tds_read_b128 %2, %9 offset:2*%11\n\tds_read_b128 %3, %9 offset:3*%11\n\t"
                         "ds_read_b128 %4, %9 offset:4*%11\n\tds_read_b128 %5, %9 offset:5*%11\n\tds_read_b128 %6, %9 offset:6*%11\n\tds_read_b128 %7, %9 offset:7*%11\n\t"
                         "ds_read_b32 %8, %10\n\ts_waitcnt lgkmcnt(0)"
                         : "=&v"(r[0]), "=&v"(r[1]), "=&v"(r[2]), "=&v"(r[3]), "=&v"(r[4]), "=&v"(r[5]), "=&v"(r[6]), "=&v"(r[7]), "=&v"(poison)
                         : "v"(pr0 + (mine ? pb : pb - PBUF)), "v"(lbase + BAD0), "n"(PTILE) : "memory");
            const i32x4 sum = ((r[0] + r[1]) + (r[2] + r[3])) + ((r[4] + r[5]) + (r[6] + r[7]));
            const float sc = poison ? __builtin_nanf("") : scale;
            char* const orow = (char*)obase + (size_t)(info & 255) * (16 * NL * 4);
            const unsigned roff = g * (NL * 4) + (lane & 15) * 16;
            if (!foreign) {
                f32x4 y;
#pragma unroll
                for (int e = 0; e < 4; ++e) y[e] = (float)sum[e] * sc;
                asm volatile("global_store_dwordx4 %0, %1, %2 sc0 sc1\n\ts_nop 1" :: "v"(roff), "v"(y), "s"(orow) : "memory");
            } else {
                hold = (info >> 8) & 7;
                hdue = info >> 12;
                const unsigned big = ((unsigned)sum[0] + 0x200000u) | ((unsigned)sum[1] + 0x200000u) | ((unsigned)sum[2] + 0x200000u) | ((unsigned)sum[3] + 0x200000u);
                if ((info & 0x800) || __any((big >> 22) | poison)) {
#pragma unroll
                    for (int e = 0; e < 4; ++e) o[e] = __builtin_bit_cast(unsigned, (float)sum[e] * sc);
                    op = orow; ooff = roff;
                    holdv = 1 + (16u << (wave & 3));
                } else {
                    const unsigned s0 = sum[0], s1 = sum[1], s2 = sum[2], s3 = sum[3];
                    const unsigned hi = ((s0 >> 16) & 63) | ((s1 >> 10) & (63 << 6)) | ((s2 >> 4) & (63 << 12)) | ((s3 << 2) & (63 << 18));
                    const unsigned pk = lbase + PACK0 + wave * 768;
                    asm volatile("ds_write_b32 %1, %2\n\tds_write_b32 %1, %3 offset:256\n\tds_write_b16 %5, %4 offset:512\n\t"
                                 "ds_write_b8_d16_hi %6, %4 offset:640\n\tds_read_b128 %0, %7\n\ts_waitcnt lgkmcnt(0)"
                                 : "=&v"(o) : "v"(pk + lane * 4), "v"((s0 & 0xffffu) | (s1 << 16)), "v"((s2 & 0xffffu) | (s3 << 16)), "v"(hi),
                                   "v"(pk + lane * 2), "v"(pk + lane), "v"(pk + l16) : "memory");
                    op = ydst + hold * YTILE; ooff = l16;
                    holdv = 1;
                }
                if (u >= nunit || (int)((unsigned)__builtin_amdgcn_s_memrealtime() - ts) + HOLDMIN >= hdue) { pace(hdue); emit(); }
            }
        }
        paired = false;
    }
    asm volatile("s_waitcnt vmcnt(0)" : "+v"(a[0]), "+v"(a[1]), "+v"(a[2]), "+v"(a[3]), "+v"(a[4]), "+v"(a[5]), "+v"(a[6]), "+v"(a[7]), "+v"(fv) :: "memory");
    if (pend >= 0) publish();
    if (tid == 0) __hip_atomic_store(cnt, call, __ATOMIC_RELAXED, __HIP_MEMORY_SCOPE_AGENT);
    EXITSTAMP();
}

extern "C" void kernel_launch(int rank, int nranks, ncclComm_t comm,
                              hipStream_t stream,
                              void* const* d_in, const int* in_sizes, int n_in,
                              void* d_out, int out_size,
                              const int* in_dtypes, int out_dtype,
                              void* const* d_out_all,
                              void* const* d_ws_all, size_t ws_bytes) {
    if (nranks != NR || (unsigned)rank >= (unsigned)NR || n_in != 4 || ws_bytes < WS_NEED
        || in_sizes[0] != ML * KDIM || in_sizes[1] != KDIM * NL || out_size != NR * ML * NL) {
        fprintf(stderr, "ag_gemm: unexpected problem shape\n");
        return;
    }
    HIP_CHECK(hipFuncSetAttribute((const void*)ag_gemm, hipFuncAttributeMaxDynamicSharedMemorySize, LDS_BYTES));
    Params p;
    p.x = (const char*)d_in[0]; p.w = (const char*)d_in[1];
    p.sx = (const float*)d_in[2]; p.sw = (const float*)d_in[3];
    p.rank = rank; p.pad = 0;
    const int me = rank & 3, other = (rank & 4) ^ 4;
    p.myws = (char*)d_ws_all[rank]; p.out = (float*)d_out;
    for (int i = 0; i < NR - 1; ++i) {
        p.link[i].ws = (char*)d_ws_all[i < 3 ? (rank & 4) | ((me + 1 + i) & 3) : other | (i - 3)];
        p.link[i].fin = p.link[i].fout = p.link[i].pad[0] = p.link[i].pad[1] = p.link[i].pad[2] = 0;
    }
    for (int k = 0; k < 4; ++k) p.pout[k] = (float*)d_out_all[other | k];
    p.extra[0] = p.extra[1] = 0;
    int ne = 0;
    for (int f = 0; f < NF; ++f) {
        p.link[3 + ((me + offs(f)) & 3)].fin |= 1u << f;
        for (int m = 0; m < 4; ++m) {
            if (((m + offs(f)) & 3) != me) continue;
            for (int k = 0; k < 4; ++k) if (k != m) p.link[3 + k].fout |= 1u << (4 * f + m);
            p.extra[ne >> 2] |= (unsigned)((other | m) * NRB + ND + f) << (8 * (ne & 3));
            ++ne;
        }
    }
    for (int i = 0; i < NR - 1; ++i) {
        const unsigned fin = p.link[i].fin, rest = fin & (fin - 1);
        const unsigned s0 = rank * NRB + (i < 3 ? 0 : ND + __builtin_ctz(fin));
        const unsigned s1 = i < 3 ? rank * NRB + 1 : rest ? rank * NRB + ND + __builtin_ctz(rest) : NR * NRB + me * 8;
        p.link[i].first = s0 | s1 << 16;
    }
    hipLaunchKernelGGL(ag_gemm, dim3(GRID), dim3(512), LDS_BYTES, stream, p);
    HIP_CHECK(hipGetLastError());
}
```

```cpp
#include <hip/hip_runtime.h>
#include <rccl/rccl.h>
#include <cstdio>
#include <cstdint>

#define HIP_CHECK(expr) do { hipError_t _e = (expr); if (_e != hipSuccess) { \
    fprintf(stderr, "HIP error %s at %s:%d\n", hipGetErrorString(_e), __FILE__, __LINE__); \
    return; } } while (0)

typedef unsigned u32x4 __attribute__((ext_vector_type(4)));
typedef int i32x4 __attribute__((ext_vector_type(4)));
typedef int i32x2 __attribute__((ext_vector_type(2)));
typedef float f32x4 __attribute__((ext_vector_type(4)));
#define LDS_AS __attribute__((address_space(3)))
#define GLB_AS __attribute__((address_space(1)))

#define NF 7
#define D1 13
#define ROUND 1112
#define LEAD 125
#define SLEAD 10
#define SORD (i + 1 < nround ? (i * 4 + wq) * 8 + j : (i * 8 + ((j + pi) & 7)) * 4 + wq)
#define GAP 115
#define HINTR 4
#define HINTMODE 1
#define YLEAD 10
#define GSLEEP 0
#define YROT 112
#define HOLDMIN 70
#define PSLEEP 1
#define LASTX 6
#define NFREE 6
#define YSLOT 484
#define LAZY 4
#define WAKE 2
#define NSGL 3
#define NRAW 10

constexpr int NR = 8;
constexpr int KDIM = 4096;
constexpr int ML = 512;
constexpr int NL = 512;
constexpr int NRB = ML / 16;
constexpr int ND = NRB - NF;
constexpr int RBB = 16 * KDIM;
constexpr unsigned OFFS = NF == 5 ? 0x20321u : NF == 6 ? 0x310321u : 0x03210321u;
__host__ __device__ constexpr int offs(int f) { return (OFFS >> (4 * f)) & 3; }
static_assert(NF >= 4 && NF <= 8 && D1 <= ND, "plan");
constexpr int GEMM0 = (NR - 1) * 8;
constexpr int GRID = GEMM0 + 8 * 24;
constexpr int WPB = 8;
constexpr int LDS_BYTES = 148 << 10;
constexpr int NSLOT = NR * NRB + 64;
constexpr size_t WS_CNT = 0;
constexpr size_t WS_TMO = 48 << 10;
constexpr size_t WS_ZERO = 50 << 10;
constexpr size_t WS_HINT = 52 << 10;
constexpr size_t WS_FLAG = 256 << 10;
constexpr size_t WS_YFLAG = 128 << 10;
constexpr size_t WS_X = 1 << 20;
static_assert(WS_FLAG + NSLOT * 1024 <= WS_X, "flags");
constexpr size_t XPAR = (size_t)NSLOT * RBB;
constexpr size_t WS_Y = WS_X + 2 * XPAR;
constexpr int YT = 8;
constexpr int YPART = 704, YTILE = 4 * YPART;
constexpr size_t YPAR = (size_t)4 * 32 * YT * YTILE;
constexpr size_t WS_DIAG = WS_Y + 2 * YPAR;
constexpr size_t WS_NEED = WS_DIAG + (4 << 20);
constexpr unsigned long long DEADLINE_TICKS = 30000000ull;

struct Link {
    char* ws;
    unsigned first;
    unsigned fin;
    unsigned fout;
    unsigned pad[3];
};
struct Params {
    const char* x; const char* w; char* myws;
    int rank; int pad;
    Link link[NR - 1];
    const float* sx; const float* sw;
    float* out; float* pout[4];
    unsigned extra[2];
};

__device__ __forceinline__ int list_slot(int r, int pos) {
    if (pos < NRB) return r * NRB + pos;
    const int q = pos - NRB, t = q / 3, s = q - 3 * t;
    const int rb = t < D1 ? t : (t < D1 + NF ? ND + t - D1 : t - NF);
    return ((r & 4) | ((r + 1 + s) & 3)) * NRB + rb;
}
__device__ __forceinline__ bool held_by(int slot, int k) {
    const int rb = slot & (NRB - 1);
    return rb >= ND && (((slot >> 5) + offs(rb - ND)) & 3) == k;
}

#define WAITV(n) asm volatile("s_waitcnt vmcnt(" #n ")" ::: "memory")

#define EXITSTAMP() do {} while (0)
#define STAMP(idx) do {} while (0)

__global__ void __launch_bounds__(512, 2) ag_gemm(Params p) {
    extern __shared__ __attribute__((aligned(16))) char smem[];
    const int tid = threadIdx.x, lane = tid & 63;
    const int wave = __builtin_amdgcn_readfirstlane(tid >> 6);
    const int bid = blockIdx.x;
    const unsigned long long t0 = __builtin_amdgcn_s_memrealtime();
    unsigned ts = (unsigned)t0 + LEAD;
    const Link lk = p.link[bid < GEMM0 ? bid >> 3 : 3 + (((bid - GEMM0) >> 4) & 3)];
    const char* const px = p.x; const char* const pw = p.w;
    const int rank = p.rank, me = rank & 3, other = (rank & 4) ^ 4;
    char* myws = p.myws;
    asm volatile("" :: "s"(lk.ws), "s"(lk.first), "s"(lk.fout), "s"(px), "s"(pw), "s"(myws), "s"(rank));
    GLB_AS unsigned* cnt = (GLB_AS unsigned*)(myws + WS_CNT + (size_t)bid * 128);
    GLB_AS unsigned* tmo = (GLB_AS unsigned*)(myws + WS_TMO);
    const unsigned lbase = (unsigned)(size_t)(LDS_AS char*)smem;
    unsigned call;
    bool bad = false;
    auto put_word = [&](const void* f, unsigned v) {
        asm volatile("s_mov_b64 exec, 1\n\tglobal_store_dword %0, %1, %2 sc0 sc1\n\ts_mov_b64 exec, -1" :: "v"(0u), "v"(v), "s"(f) : "memory");
    };
    auto give_up = [&]() {
        if (__builtin_amdgcn_readfirstlane(__hip_atomic_load(tmo, __ATOMIC_RELAXED, __HIP_MEMORY_SCOPE_AGENT)) == call) bad = true;
        else if (__builtin_amdgcn_s_memrealtime() - t0 > DEADLINE_TICKS) { put_word(myws + WS_TMO, call); bad = true; }
        return bad;
    };
    auto wait_flag = [&](const char* f, unsigned want) -> unsigned {
        unsigned v = 0;
        if (bad) return v;
        do {
            unsigned x, y, n;
            asm volatile("s_mov_b64 exec, 1\n\ts_mov_b32 %3, 8\n\tglobal_load_dword %1, %4, %5 sc0 sc1\n\ts_nop 15\n"
                         "1:\n\tglobal_load_dword %2, %4, %5 sc0 sc1\n\ts_waitcnt vmcnt(1)\n\tv_readfirstlane_b32 %0, %1\n\t"
                         "s_cmp_ge_u32 %0, %6\n\ts_cbranch_scc1 2f\n\t"
                         "global_load_dword %1, %4, %5 sc0 sc1\n\ts_waitcnt vmcnt(1)\n\tv_readfirstlane_b32 %0, %2\n\t"
                         "s_cmp_ge_u32 %0, %6\n\ts_cbranch_scc1 2f\n\t"
                         "s_sub_u32 %3, %3, 1\n\ts_cmp_lg_u32 %3, 0\n\ts_cbranch_scc1 1b\n"
                         "2:\n\ts_mov_b64 exec, -1\n\ts_waitcnt vmcnt(0)"
                         : "=&s"(v), "=&v"(x), "=&v"(y), "=&s"(n) : "v"(0u), "s"(f), "s"(want) : "memory", "scc");
            if (__builtin_expect(v >= want, 1)) return v;
        } while (!give_up());
        return 0;
    };
    auto wait_flags4 = [&](const char* f) {
        do {
#pragma unroll 1
            for (int it = 0; it < 8; ++it) {
                unsigned fv;
                asm volatile("s_mov_b64 exec, 15\n\tglobal_load_dword %0, %1, %2 sc0 sc1\n\ts_mov_b64 exec, -1\n\ts_waitcnt vmcnt(0)"
                             : "=&v"(fv) : "v"(lane * 128), "s"(f) : "memory");
                if ((__ballot(fv < call) & 15) == 0) return;
                __builtin_amdgcn_s_sleep(LAZY);
            }
        } while (!give_up());
    };
    auto pace = [&](int due) { while ((int)((unsigned)__builtin_amdgcn_s_memrealtime() - ts) < due) __builtin_amdgcn_s_sleep(1); };
    auto snooze = [&](int due) { while ((int)((unsigned)__builtin_amdgcn_s_memrealtime() - ts) < due) __builtin_amdgcn_s_sleep(8); };
    auto lds_add = [&](unsigned addr, unsigned v) -> unsigned {
        unsigned old;
        asm volatile("s_mov_b64 exec, 1\n\tds_add_rtn_u32 %0, %1, %2\n\ts_mov_b64 exec, -1\n\ts_waitcnt lgkmcnt(0)" : "=&v"(old) : "v"(addr), "v"(v) : "memory");
        return __builtin_amdgcn_readfirstlane(old);
    };

    const int g = lane >> 4;
    if (__builtin_expect(bid < GEMM0, 1)) {
        const int j = bid & 7, pi = bid >> 3;
        const bool cross = pi >= 3;
        const int k = (pi - 3) & 3;
        const int nround = cross ? 8 + NF : ND;
        const int wq = wave & 3, frag = j * 8 + wq * 2;
        const unsigned stg = lbase + 1024 + wave * 4096;
        const unsigned xo = (lane >> 3) * KDIM + (((lane & 7) ^ g) << 4), wo = (lane >> 2) * NL + (((lane & 3) ^ (g & 1)) << 4);
        auto fetch = [&](int slot, int i) {
            const bool isw = slot >= NR * NRB;
            const char* base = isw ? pw + (size_t)(((slot & 3) * 8 + j) * 128 + wq * 32) * NL + k * 128 + (slot & 4) * 16
                                   : px + (size_t)(slot & 31) * RBB + frag * 64;
            unsigned keep;
            asm volatile("s_mov_b32 %0, m0\n\ts_mov_b32 m0, %4\n\ts_nop 0\n\t"
                         "global_load_lds_dwordx4 %1, %3\n\ts_add_u32 m0, m0, 1024\n\ts_nop 0\n\t"
                         "global_load_lds_dwordx4 %2, %3\n\ts_mov_b32 m0, %0"
                         : "=&s"(keep) : "v"(isw ? wo : xo), "v"(isw ? (wo ^ 32) + 16 * NL : (xo ^ 64) + 8 * KDIM), "s"(base),
                           "s"(stg + ((i >> 1) & 1) * 2048) : "memory", "scc");
        };
        fetch((lk.first >> (16 * (wave >> 2))) & 0xffff, 0);
        unsigned c0;
        asm volatile("global_load_dword %0, %1, %2 sc1\n\tds_write_b32 %3, %1" : "=&v"(c0) : "v"(0u), "s"(cnt), "v"(lbase + 4 * lane) : "memory");
        int vslot;
        {
            const unsigned fin = lk.fin, fout = lk.fout;
            const int nin = __builtin_popcount(fin);
            unsigned m = lane < nin ? fin : fout;
            const int c = lane < nin ? lane : lane - nin - 8;
#pragma unroll
            for (int t = 0; t < NF - 1; ++t) m &= m - (c > t);
            const int b = __builtin_ctz(m | 0x80000000u);
            vslot = !cross ? rank * NRB + (lane & 31) : lane < nin ? rank * NRB + ND + b
                  : lane < nin + 8 ? NR * NRB + me * 8 + lane - nin : (other | (b & 3)) * NRB + min(ND + (b >> 2), NRB - 1);
        }
        unsigned fseen = 0;
        asm volatile("s_waitcnt lgkmcnt(0)\n\ts_barrier" ::: "memory");
        int early = 0;
        unsigned par = 0;
        call = 0;
        char* dst = lk.ws + WS_X + frag * 1024;
        char* flag = lk.ws + WS_FLAG + j * 128;
        const unsigned l16 = lane * 16;
#pragma unroll 1
        for (int i = wave >> 2; i < nround; i += 2) {
            asm volatile("s_cmp_gt_u32 %1, 1\n\ts_cbranch_scc1 1f\n\ts_waitcnt vmcnt(0)\n1:" : "+v"(c0) : "s"(i) : "memory", "scc");
            if (i < 2) {
                call = __builtin_amdgcn_readfirstlane(c0) + 1;
                par = (call & 1) * (unsigned)XPAR;
                ts = (unsigned)__builtin_amdgcn_s_memrealtime() + SLEAD;
            }
            const int slot = __builtin_amdgcn_readlane(vslot, i);
            const bool own = (slot >> 5) == rank;
            const unsigned r0 = stg + ((i >> 1) & 1) * 2048 + (own ? (lane & 15) * 128 + ((g ^ ((lane & 15) >> 1)) << 4) : l16);
            u32x4 v0, v1;
            if (slot < NR * NRB) {
                asm volatile("ds_read_b128 %0, %2\n\tds_read_b128 %1, %3\n\ts_waitcnt lgkmcnt(0)"
                             : "=&v"(v0), "=&v"(v1) : "v"(r0), "v"(own ? r0 ^ 64 : r0 + 1024) : "memory");
            } else {
                const int q = (lane & 15) >> 1;
                const unsigned rt = stg + ((i >> 1) & 1) * 2048 + (16 * (g & 1) + q) * 64 + 8 * (lane & 1) + (((g >> 1) ^ (q >> 2) ^ ((g & 1) << 1)) << 4);
                i32x2 a, b, c, d;
                asm volatile("ds_read_b64_tr_b8 %0, %4\n\tds_read_b64_tr_b8 %1, %4 offset:512\n\t"
                             "ds_read_b64_tr_b8 %2, %5\n\tds_read_b64_tr_b8 %3, %5 offset:512\n\ts_waitcnt lgkmcnt(0)"
                             : "=&v"(a), "=&v"(b), "=&v"(c), "=&v"(d) : "v"(rt), "v"(rt ^ 32) : "memory");
                v0 = (u32x4){(unsigned)a[0], (unsigned)a[1], (unsigned)b[0], (unsigned)b[1]};
                v1 = (u32x4){(unsigned)c[0], (unsigned)c[1], (unsigned)d[0], (unsigned)d[1]};
            }
            pace(max((SORD * ROUND) / 320, early));
            asm volatile("global_store_dwordx4 %0, %2, %1 sc0 sc1\n\tglobal_store_dwordx4 %0, %3, %1 offset:1024 sc0 sc1\n\ts_nop 1"
                         :: "v"(l16), "s"(dst + (par + (unsigned)slot * RBB)), "v"(v0), "v"(v1) : "memory");
            if (i == 0) STAMP(40);
            if (i < 2) fetch(__builtin_amdgcn_readlane(vslot, i + 2), 2);
            if (cross && (i >> 1) == HINTR && lane == 0)
                __hip_atomic_store((GLB_AS unsigned long long*)(myws + WS_HINT) + k * 64 + j * 8 + wave,
                                   ((unsigned long long)call << 32) | (ts + max(early - (SORD * ROUND) / 320, 0)), __ATOMIC_RELAXED, __HIP_MEMORY_SCOPE_AGENT);
            if (cross && (i >> 1) == 2 && lane < 16) fseen = __hip_atomic_load((GLB_AS unsigned*)(myws + WS_FLAG + (vslot & 255) * 1024 + j * 128), __ATOMIC_RELAXED, __HIP_MEMORY_SCOPE_SYSTEM);
            asm volatile("s_waitcnt vmcnt(0)" : "+v"(fseen) :: "memory");
            early = i + 3 < nround ? (int)((unsigned)__builtin_amdgcn_s_memrealtime() - ts) + GAP : 0;
            if (!bad && lds_add(lbase + 4 * i, 1) == 3) {
                put_word(flag + slot * 1024, call);
            }
            if (i + 4 < nround) {
                const int ns = __builtin_amdgcn_readlane(vslot, i + 4);
                if ((ns >> 7) != (other >> 2)) fetch(ns, i);
                else {
                    if (__builtin_amdgcn_readlane(fseen, i + 4) < call) wait_flag(myws + WS_FLAG + ns * 1024 + j * 128, call);
                    unsigned keep;
                    asm volatile("s_mov_b32 %0, m0\n\ts_mov_b32 m0, %3\n\ts_nop 0\n\tglobal_load_lds_dwordx4 %1, %2 sc0 sc1\n\t"
                                 "global_load_lds_dwordx4 %1, %2 offset:1024 sc0 sc1\n\ts_mov_b32 m0, %0"
                                 : "=&s"(keep) : "v"(l16), "s"(myws + WS_X + frag * 1024 + (par + (unsigned)ns * RBB)),
                                   "s"(stg + ((i >> 1) & 1) * 2048) : "memory");
                }
            }
        }
        if (lds_add(lbase + 4 * 63, 1) == WPB - 1) put_word((const void*)cnt, call);
        if (!cross) { EXITSTAMP(); return; }

        const float scale = p.sx[0] * p.sw[0];
        const int prod = j * 4 + wq, h = prod >> 4, rep = prod & 15, e0 = (wave >> 2) * 2;
        const char* yf = myws + WS_YFLAG + (k * 32 + prod) * 128;
        const char* yb = myws + WS_Y + (call & 1) * YPAR + (size_t)(k * 32 + prod) * YT * YTILE + e0 * YPART;
        const int yslot = list_slot(other | k, (lane == 7 ? (rep + LASTX) & 15 : rep) + 16 * (lane & 7));
        unsigned todo = (unsigned)__ballot(lane < 8 && !held_by(yslot, me));
#pragma unroll 1
        for (; todo; todo &= todo - 1) {
            const int n = __builtin_ctz(todo), slot = __builtin_amdgcn_readlane(yslot, n);
            const unsigned raw = (wait_flag(yf + n * 4, call * 16) >> e0) & 3;
            STAMP(44 + n);
            if (raw == 3) continue;
            unsigned d[8];
            asm volatile("global_load_dword %0, %8, %11 sc0 sc1\n\tglobal_load_dword %1, %8, %11 offset:256 sc0 sc1\n\t"
                         "global_load_ushort %2, %9, %11 offset:512 sc0 sc1\n\tglobal_load_ubyte %3, %10, %11 offset:640 sc0 sc1\n\t"
                         "global_load_dword %4, %8, %11 offset:704 sc0 sc1\n\tglobal_load_dword %5, %8, %11 offset:960 sc0 sc1\n\t"
                         "global_load_ushort %6, %9, %11 offset:1216 sc0 sc1\n\tglobal_load_ubyte %7, %10, %11 offset:1344 sc0 sc1\n\t"
                         "s_waitcnt vmcnt(0)"
                         : "=&v"(d[0]), "=&v"(d[1]), "=&v"(d[2]), "=&v"(d[3]), "=&v"(d[4]), "=&v"(d[5]), "=&v"(d[6]), "=&v"(d[7])
                         : "v"(lane * 4), "v"(lane * 2), "v"(lane), "s"(yb + n * YTILE) : "memory");
            const float sc = bad ? __builtin_nanf("") : scale;
            float* op = p.out + (size_t)(slot * 16 + e0 * 4 + g) * NL + k * 128 + h * 64 + (lane & 15) * 4;
#pragma unroll
            for (int e = 0; e < 2; ++e) {
                if ((raw >> e) & 1) continue;
                const unsigned d0 = d[4 * e], d1 = d[4 * e + 1], hi = d[4 * e + 2] | d[4 * e + 3] << 16;
                f32x4 o;
                o[0] = (float)(((int)(hi << 26) >> 10) | (int)(d0 & 0xffffu)) * sc;
                o[1] = (float)((((int)(hi << 20) >> 26) << 16) | (int)(d0 >> 16)) * sc;
                o[2] = (float)((((int)(hi << 14) >> 26) << 16) | (int)(d1 & 0xffffu)) * sc;
                o[3] = (float)((((int)(hi << 8) >> 26) << 16) | (int)(d1 >> 16)) * sc;
                asm volatile("global_store_dwordx4 %0, %1, off sc0 sc1\n\ts_nop 1" :: "v"(op + e * 4 * NL), "v"(o) : "memory");
            }
        }
        EXITSTAMP();
        return;
    }

    const int gb = bid - GEMM0;
    const int xcd = gb & 7, sl = (gb >> 3) & 7, set = gb >> 6;
    const bool foreign = set > 0;
    const int rep = foreign ? xcd + 8 * (set - 1) : xcd;
    if (GSLEEP) __builtin_amdgcn_s_sleep(GSLEEP);
    const unsigned c0 = __hip_atomic_load(cnt, __ATOMIC_RELAXED, __HIP_MEMORY_SCOPE_AGENT);
    const float scale = p.sx[0] * p.sw[0];

    constexpr int PTILE = 16 * 272, PBUF = 8 * PTILE, PACK0 = 4 * PBUF, ARR0 = PACK0 + 8 * 768, BAD0 = ARR0 + 32;
    static_assert(ARR0 + 256 + 64 <= LDS_BYTES, "LDS");
    const unsigned l16 = lane * 16;
    const unsigned ast = lbase + wave * 4096;

    if (foreign) {
        const int c = gb - 64, sn = c >> 2, frag = (c & 3) * 16 + wave * 2;
        const char* xs = px + (size_t)sn * RBB + frag * 64;
        const char* wsrc = pw + (size_t)((sn & 3) * 1024 + frag * 16) * NL + (sn >> 2) * 64;
        const unsigned xo = (lane >> 3) * KDIM + (((lane & 7) ^ g) << 4), wo = (lane >> 2) * NL + (((lane & 3) ^ (g & 1)) << 4);
        const int q = (lane & 15) >> 1;
        const unsigned rx = ast + (lane & 15) * 128 + ((g ^ q) << 4);
        const unsigned rt = ast + 2048 + (16 * (g & 1) + q) * 64 + 8 * (lane & 1) + (((g >> 1) ^ (q >> 2) ^ ((g & 1) << 1)) << 4);
        u32x4 v0, v1; i32x2 t0, t1, t2, t3;
        unsigned keep;
        asm volatile("s_mov_b32 %6, m0\n\ts_mov_b32 m0, %13\n\ts_nop 4\n\t"
                     "global_load_lds_dwordx4 %7, %11\n\ts_add_u32 m0, m0, 1024\n\ts_nop 0\n\t"
                     "global_load_lds_dwordx4 %8, %11\n\ts_add_u32 m0, m0, 1024\n\ts_nop 0\n\t"
                     "global_load_lds_dwordx4 %9, %12\n\ts_add_u32 m0, m0, 1024\n\ts_nop 0\n\t"
                     "global_load_lds_dwordx4 %10, %12\n\ts_mov_b32 m0, %6\n\ts_waitcnt vmcnt(0)\n\t"
                     "ds_read_b128 %0, %14\n\tds_read_b128 %1, %15\n\t"
                     "ds_read_b64_tr_b8 %2, %16\n\tds_read_b64_tr_b8 %3, %16 offset:512\n\t"
                     "ds_read_b64_tr_b8 %4, %17\n\tds_read_b64_tr_b8 %5, %17 offset:512\n\ts_waitcnt lgkmcnt(0)"
                     : "=&v"(v0), "=&v"(v1), "=&v"(t0), "=&v"(t1), "=&v"(t2), "=&v"(t3), "=&s"(keep)
                     : "v"(xo), "v"((xo ^ 64) + 8 * KDIM), "v"(wo), "v"((wo ^ 32) + 16 * NL), "s"(xs), "s"(wsrc), "s"(ast),
                       "v"(rx), "v"(rx ^ 64), "v"(rt), "v"(rt ^ 32) : "memory", "scc");
        call = __builtin_amdgcn_readfirstlane(c0) + 1;
        const char* xw = myws + WS_X + (call & 1) * XPAR + frag * 1024;
        const i32x4 v2 = {t0[0], t0[1], t1[0], t1[1]}, v3 = {t2[0], t2[1], t3[0], t3[1]};
        asm volatile("global_store_dwordx4 %0, %5, %2 sc0 sc1\n\tglobal_store_dwordx4 %0, %6, %2 offset:1024 sc0 sc1\n\t"
                     "global_store_dwordx4 %0, %3, %1 sc0 sc1\n\tglobal_store_dwordx4 %0, %4, %1 offset:1024 sc0 sc1\n\t"
                     "s_waitcnt vmcnt(0)\n\ts_barrier"
                     :: "v"(l16), "s"(xw + (size_t)(rank * NRB + sn) * RBB), "s"(xw + (size_t)(NR * NRB + 32 + sn) * RBB),
                        "v"(v0), "v"(v1), "v"(v2), "v"(v3) : "memory");
        if (tid < 4) __hip_atomic_store((GLB_AS unsigned*)(myws + WS_FLAG + ((tid & 2 ? NR * NRB + 32 : rank * NRB) + sn) * 1024 + ((c & 3) * 2 + (tid & 1)) * 128),
                                        call, __ATOMIC_RELAXED, __HIP_MEMORY_SCOPE_SYSTEM);
        STAMP(2);
    }
    call = __builtin_amdgcn_readfirstlane(c0) + 1;
    const char* xws = myws + WS_X + (call & 1) * XPAR;
    if (tid < 16) *(unsigned*)(smem + ARR0 + tid * 4) = 0;
    asm volatile("s_waitcnt lgkmcnt(0)\n\ts_barrier" ::: "memory");

    unsigned vinfo, vflag;
    int nunit;
    {
        const int pos = min(foreign ? (lane == 7 ? (rep + LASTX) & 15 : rep) + 16 * lane : rep + 8 * (lane - (lane > 4)), 127);
        const int slot = !foreign && lane == 4 ? (int)((p.extra[xcd >> 2] >> (8 * (xcd & 3))) & 255) : list_slot(rank, pos);
        const bool use = foreign ? lane < 8 && !held_by(slot, sl >> 1) : lane < 17 && (lane != 4 || xcd < NF);
        const unsigned mask = (unsigned)__ballot(use);
        const int pr = pos < YROT ? (pos & ~7) | ((pos + (sl & 6)) & 7) : pos;
        const int c4 = 4 * pr - min(max(pr - ND, 0), NF) - min(max(pr - NRB - 3 * D1, 0), 3 * NF);
        const int due = pos >= 128 - NFREE ? 0 : ((8 + NF) * ROUND) / 10 + ((c4 + 2 * (sl & 1)) * YSLOT) / 200;
        nunit = __builtin_popcount(mask);
        vinfo = __builtin_amdgcn_ds_permute((use ? __builtin_amdgcn_mbcnt_lo(mask, 0) : 63) * 4, slot | (lane & 7) << 8 | (pos >= 128 - NRAW) << 11 | due << 12);
        vflag = lane < nunit ? (unsigned)WS_FLAG + wave * 128 + (vinfo & 255) * 1024 : (unsigned)WS_ZERO;
    }
    const unsigned vaoff = (vinfo & 255) << 16;
    char* ydst = lk.ws + WS_Y + (call & 1) * YPAR + (size_t)(me * 32 + (sl & 1) * 16 + rep) * YT * YTILE + (wave & 3) * YPART;
    char* yflag = lk.ws + WS_YFLAG + (me * 32 + (sl & 1) * 16 + rep) * 128;
    float* obase = (foreign ? p.pout[sl >> 1] + me * 128 + (sl & 1) * 64 : p.out + sl * 64) + (wave & 3) * 4 * NL;

    int hold = -1, hdue = 0;
    unsigned holdv = 0;
    u32x4 o = {0, 0, 0, 0};
    char* op = ydst;
    unsigned ooff = l16;
    int u = 0;
    int pend = -1;
    auto emit = [&]() {
        asm volatile("s_mov_b32 exec_hi, %3\n\tglobal_store_dwordx4 %0, %1, %2 sc0 sc1\n\ts_mov_b32 exec_hi, -1\n\ts_nop 0"
                     :: "v"(ooff), "v"(o), "s"(op), "s"(holdv > 1 ? ~0u : (1u << (YPART / 16 - 32)) - 1) : "memory");
        pend = hold; hold = -1;
    };
    auto announce = [&]() {
        const unsigned now = lds_add(lbase + ARR0 + pend * 4, holdv) + holdv;
        if ((now & 15) == 4) put_word(yflag + pend * 4, call * 16 + (now >> 4));
        pend = -1;
    };
    auto publish = [&]() { WAITV(0); announce(); };

    i32x4 bf[8][4];
    {
        const int wslot = NR * NRB + (foreign ? 0 : 32) + sl * 4 + (wave >> 1);
        if (foreign) snooze(((WAKE + (sl & 1) * 4 + (wave >> 1)) * ROUND) / 10);
        wait_flags4(myws + WS_FLAG + wslot * 1024 + (wave & 1) * 512);
        asm volatile("" ::: "memory");
        const __amdgpu_buffer_rsrc_t rs = __builtin_amdgcn_make_buffer_rsrc((void*)(xws + (size_t)wslot * RBB + (wave & 1) * 32768), 0, 32768, 0x00020000);
#pragma unroll
        for (int s = 0; s < 8; ++s)
#pragma unroll
            for (int t = 0; t < 4; ++t)
                bf[s][t] = __builtin_bit_cast(i32x4, __builtin_amdgcn_raw_buffer_load_b128(rs, (lane >> 5) * 2048 + (lane & 31) * 16 + t * 512, s * 4096, 17));
#pragma unroll
        for (int s = 0; s < 8; ++s) asm volatile("" : "+v"(bf[s][0]), "+v"(bf[s][1]), "+v"(bf[s][2]), "+v"(bf[s][3]));
    }
    if (foreign) {
        snooze(((2 * HINTR + 3) * ROUND) / 10);
        const unsigned long long h = __hip_atomic_load((GLB_AS unsigned long long*)(myws + WS_HINT) + (sl >> 1) * 64 + lane, __ATOMIC_RELAXED, __HIP_MEMORY_SCOPE_AGENT);
        int d = (unsigned)(h >> 32) == call ? (int)((unsigned)h - (unsigned)t0) : LEAD;
#pragma unroll
        for (int o = 32; o; o >>= 1) d += __shfl_xor(d, o);
        d >>= 6;
        ts = (unsigned)t0 + __builtin_amdgcn_readfirstlane(min(max(d, 0), 1000)) + YLEAD;
    }
    STAMP(1);
    i32x4 a[8];
#pragma unroll
    for (int s = 0; s < 8; ++s) a[s] = (i32x4){0, 0, 0, 0};
    unsigned fv = 0, fa = (unsigned)WS_TMO;
    u32x4 ars;
    ars[0] = (unsigned)(size_t)xws; ars[1] = (unsigned)((size_t)xws >> 32) & 0xffff; ars[2] = (unsigned)XPAR; ars[3] = 0x00020000;
    const unsigned alo = wave * 8192 + l16, ahi = alo + 4096;
    const unsigned pw0 = lbase + wave * PTILE + (lane & 15) * 272 + g * 16;
    const unsigned pr0 = lbase + ((wave & 3) * 4 + g) * 272 + (lane & 15) * 16;
    auto poison_all = [&]() { asm volatile("ds_write_b32 %0, %1" :: "v"(lbase + BAD0), "v"(1) : "memory"); };
    constexpr int RUN = 3, LAST = 9, FETCH = 5;
    int md = LAST;
    unsigned want = call;
    bool paired = false;
#pragma unroll 1
    while (u < nunit) {
        asm volatile("s_waitcnt vmcnt(7)" : "+v"(a[0]), "+v"(fv) :: "memory");
        int un;
        if (__builtin_expect(md != LAST, 1)) {
            md = __builtin_amdgcn_readlane(fv, 0) >= want ? RUN : LAST;
            want = call;
            if (__builtin_expect(__builtin_amdgcn_readlane(fv, 1) == call, 0)) poison_all();
            un = u + 1;
        } else {
            const char* f = myws + __builtin_amdgcn_readlane(vflag, u);
            bool seen = false;
            while ((hold & pend) >= 0) {
                unsigned v;
                asm volatile("s_mov_b64 exec, 1\n\tglobal_load_dword %0, %1, %2 sc0 sc1\n\ts_mov_b64 exec, -1\n\ts_waitcnt vmcnt(0)"
                             : "=&v"(v) : "v"(0u), "s"(f) : "memory");
                seen = (unsigned)__builtin_amdgcn_readfirstlane(v) >= call;
                if (seen) break;
                if (pend >= 0) announce();
                else if ((int)((unsigned)__builtin_amdgcn_s_memrealtime() - ts) >= hdue) emit();
            }
            if (!seen) wait_flag(f, call);
            if (bad) poison_all();
            md = FETCH;
            un = u;
        }
        const unsigned soff = __builtin_amdgcn_readlane(vaoff, un);
        asm volatile("s_nop 1\n\tv_writelane_b32 %1, %3, 0\n\tglobal_load_dword %0, %1, %2 sc0 sc1"
                     : "+v"(fv), "+v"(fa) : "s"(myws), "s"(__builtin_amdgcn_readlane(vflag, un + 1)), "s"(soff) : "memory");
        i32x4 acc[4];
        asm volatile("s_cmp_lg_u32 %28, 3\n\ts_cbranch_scc1 1f\n\t"
                     "v_mfma_i32_16x16x64_i8 %0, %9, %4, 0\n\tv_mfma_i32_16x16x64_i8 %1, %10, %4, 0\n\tv_mfma_i32_16x16x64_i8 %2, %11, %4, 0\n\tv_mfma_i32_16x16x64_i8 %3, %12, %4, 0\n\tbuffer_load_dwordx4 %4, %25, %26, %27 offen offset:0 sc0 sc1\n\ts_waitcnt vmcnt(8)\n\t"
                     "v_mfma_i32_16x16x64_i8 %0, %13, %5, %0\n\tv_mfma_i32_16x16x64_i8 %1, %14, %5, %1\n\tv_mfma_i32_16x16x64_i8 %2, %15, %5, %2\n\tv_mfma_i32_16x16x64_i8 %3, %16, %5, %3\n\tbuffer_load_dwordx4 %5, %25, %26, %27 offen offset:1024 sc0 sc1\n\ts_waitcnt vmcnt(8)\n\t"
                     "v_mfma_i32_16x16x64_i8 %0, %17, %6, %0\n\tv_mfma_i32_16x16x64_i8 %1, %18, %6, %1\n\tv_mfma_i32_16x16x64_i8 %2, %19, %6, %2\n\tv_mfma_i32_16x16x64_i8 %3, %20, %6, %3\n\tbuffer_load_dwordx4 %6, %25, %26, %27 offen offset:2048 sc0 sc1\n\ts_waitcnt vmcnt(8)\n\t"
                     "v_mfma_i32_16x16x64_i8 %0, %21, %7, %0\n\tv_mfma_i32_16x16x64_i8 %1, %22, %7, %1\n\tv_mfma_i32_16x16x64_i8 %2, %23, %7, %2\n\tv_mfma_i32_16x16x64_i8 %3, %24, %7, %3\n\tbuffer_load_dwordx4 %7, %25, %26, %27 offen offset:3072 sc0 sc1\n\ts_waitcnt vmcnt(8)\n\t"
                     "s_branch 3f\n1:\n\ts_cmp_lg_u32 %28, 9\n\ts_cbranch_scc1 2f\n\t"
                     "v_mfma_i32_16x16x64_i8 %0, %9, %4, 0\n\tv_mfma_i32_16x16x64_i8 %1, %10, %4, 0\n\tv_mfma_i32_16x16x64_i8 %2, %11, %4, 0\n\tv_mfma_i32_16x16x64_i8 %3, %12, %4, 0\n\ts_waitcnt vmcnt(7)\n\t"
                     "v_mfma_i32_16x16x64_i8 %0, %13, %5, %0\n\tv_mfma_i32_16x16x64_i8 %1, %14, %5, %1\n\tv_mfma_i32_16x16x64_i8 %2, %15, %5, %2\n\tv_mfma_i32_16x16x64_i8 %3, %16, %5, %3\n\ts_waitcnt vmcnt(6)\n\t"
                     "v_mfma_i32_16x16x64_i8 %0, %17, %6, %0\n\tv_mfma_i32_16x16x64_i8 %1, %18, %6, %1\n\tv_mfma_i32_16x16x64_i8 %2, %19, %6, %2\n\tv_mfma_i32_16x16x64_i8 %3, %20, %6, %3\n\ts_waitcnt vmcnt(5)\n\t"
                     "v_mfma_i32_16x16x64_i8 %0, %21, %7, %0\n\tv_mfma_i32_16x16x64_i8 %1, %22, %7, %1\n\tv_mfma_i32_16x16x64_i8 %2, %23, %7, %2\n\tv_mfma_i32_16x16x64_i8 %3, %24, %7, %3\n\ts_waitcnt vmcnt(4)\n\t"
                     "s_branch 3f\n2:\n\tbuffer_load_dwordx4 %4, %25, %26, %27 offen offset:0 sc0 sc1\n\tbuffer_load_dwordx4 %5, %25, %26, %27 offen offset:1024 sc0 sc1\n\tbuffer_load_dwordx4 %6, %25, %26, %27 offen offset:2048 sc0 sc1\n\tbuffer_load_dwordx4 %7, %25, %26, %27 offen offset:3072 sc0 sc1\n\t"
                     "\n3:\n\t"
                     : "=&v"(acc[0]), "=&v"(acc[1]), "=&v"(acc[2]), "=&v"(acc[3]), "+v"(a[0]), "+v"(a[1]), "+v"(a[2]), "+v"(a[3]), "+v"(a[4])
                     : "v"(bf[0][0]), "v"(bf[0][1]), "v"(bf[0][2]), "v"(bf[0][3]), "v"(bf[1][0]), "v"(bf[1][1]), "v"(bf[1][2]), "v"(bf[1][3]), "v"(bf[2][0]), "v"(bf[2][1]), "v"(bf[2][2]), "v"(bf[2][3]), "v"(bf[3][0]), "v"(bf[3][1]), "v"(bf[3][2]), "v"(bf[3][3]),
                       "v"(alo), "s"(ars), "s"(soff), "s"(md) : "memory", "scc");
        asm volatile("s_cmp_lg_u32 %28, 3\n\ts_cbranch_scc1 1f\n\t"
                     "v_mfma_i32_16x16x64_i8 %0, %9, %4, %0\n\tv_mfma_i32_16x16x64_i8 %1, %10, %4, %1\n\tv_mfma_i32_16x16x64_i8 %2, %11, %4, %2\n\tv_mfma_i32_16x16x64_i8 %3, %12, %4, %3\n\tbuffer_load_dwordx4 %4, %25, %26, %27 offen offset:0 sc0 sc1\n\ts_waitcnt vmcnt(8)\n\t"
                     "v_mfma_i32_16x16x64_i8 %0, %13, %5, %0\n\tv_mfma_i32_16x16x64_i8 %1, %14, %5, %1\n\tv_mfma_i32_16x16x64_i8 %2, %15, %5, %2\n\tv_mfma_i32_16x16x64_i8 %3, %16, %5, %3\n\tbuffer_load_dwordx4 %5, %25, %26, %27 offen offset:1024 sc0 sc1\n\ts_waitcnt vmcnt(8)\n\t"
                     "v_mfma_i32_16x16x64_i8 %0, %17, %6, %0\n\tv_mfma_i32_16x16x64_i8 %1, %18, %6, %1\n\tv_mfma_i32_16x16x64_i8 %2, %19, %6, %2\n\tv_mfma_i32_16x16x64_i8 %3, %20, %6, %3\n\tbuffer_load_dwordx4 %6, %25, %26, %27 offen offset:2048 sc0 sc1\n\ts_waitcnt vmcnt(8)\n\t"
                     "v_mfma_i32_16x16x64_i8 %0, %21, %7, %0\n\tv_mfma_i32_16x16x64_i8 %1, %22, %7, %1\n\tv_mfma_i32_16x16x64_i8 %2, %23, %7, %2\n\tv_mfma_i32_16x16x64_i8 %3, %24, %7, %3\n\tbuffer_load_dwordx4 %7, %25, %26, %27 offen offset:3072 sc0 sc1\n\t"
                     "s_branch 3f\n1:\n\ts_cmp_lg_u32 %28, 9\n\ts_cbranch_scc1 2f\n\t"
                     "v_mfma_i32_16x16x64_i8 %0, %9, %4, %0\n\tv_mfma_i32_16x16x64_i8 %1, %10, %4, %1\n\tv_mfma_i32_16x16x64_i8 %2, %11, %4, %2\n\tv_mfma_i32_16x16x64_i8 %3, %12, %4, %3\n\ts_waitcnt vmcnt(3)\n\t"
                     "v_mfma_i32_16x16x64_i8 %0, %13, %5, %0\n\tv_mfma_i32_16x16x64_i8 %1, %14, %5, %1\n\tv_mfma_i32_16x16x64_i8 %2, %15, %5, %2\n\tv_mfma_i32_16x16x64_i8 %3, %16, %5, %3\n\ts_waitcnt vmcnt(2)\n\t"
                     "v_mfma_i32_16x16x64_i8 %0, %17, %6, %0\n\tv_mfma_i32_16x16x64_i8 %1, %18, %6, %1\n\tv_mfma_i32_16x16x64_i8 %2, %19, %6, %2\n\tv_mfma_i32_16x16x64_i8 %3, %20, %6, %3\n\ts_waitcnt vmcnt(1)\n\t"
                     "v_mfma_i32_16x16x64_i8 %0, %21, %7, %0\n\tv_mfma_i32_16x16x64_i8 %1, %22, %7, %1\n\tv_mfma_i32_16x16x64_i8 %2, %23, %7, %2\n\tv_mfma_i32_16x16x64_i8 %3, %24, %7, %3\n\t"
                     "s_branch 3f\n2:\n\tbuffer_load_dwordx4 %4, %25, %26, %27 offen offset:0 sc0 sc1\n\tbuffer_load_dwordx4 %5, %25, %26, %27 offen offset:1024 sc0 sc1\n\tbuffer_load_dwordx4 %6, %25, %26, %27 offen offset:2048 sc0 sc1\n\tbuffer_load_dwordx4 %7, %25, %26, %27 offen offset:3072 sc0 sc1\n\t"
                     "\n3:\n\ts_nop 7"
                     : "+v"(acc[0]), "+v"(acc[1]), "+v"(acc[2]), "+v"(acc[3]), "+v"(a[4]), "+v"(a[5]), "+v"(a[6]), "+v"(a[7]), "+v"(a[0])
                     : "v"(bf[4][0]), "v"(bf[4][1]), "v"(bf[4][2]), "v"(bf[4][3]), "v"(bf[5][0]), "v"(bf[5][1]), "v"(bf[5][2]), "v"(bf[5][3]), "v"(bf[6][0]), "v"(bf[6][1]), "v"(bf[6][2]), "v"(bf[6][3]), "v"(bf[7][0]), "v"(bf[7][1]), "v"(bf[7][2]), "v"(bf[7][3]),
                       "v"(ahi), "s"(ars), "s"(soff), "s"(md) : "memory", "scc");
        if (md == FETCH) { if (pend >= 0) announce(); continue; }
        const unsigned pb = (u & 3) * PBUF;
        asm volatile("ds_write_b128 %0, %1\n\tds_write_b128 %0, %2 offset:64\n\tds_write_b128 %0, %3 offset:128\n\tds_write_b128 %0, %4 offset:192"
                     :: "v"(pw0 + pb), "v"(acc[0]), "v"(acc[1]), "v"(acc[2]), "v"(acc[3]) : "memory");
        ++u;
        if (!foreign && (u & 1) && u < nunit - NSGL) { paired = true; continue; }
        const bool mine = (wave >> 2) != (u & 1);
        const bool after = pend >= 0 && !mine;
        if (pend >= 0 && mine) publish();
        if (hold >= 0) { pace(hdue); emit(); }
        asm volatile("s_waitcnt lgkmcnt(0)\n\ts_barrier" ::: "memory");
        if (after) publish();
        if (mine || paired) {
            const unsigned info = __builtin_amdgcn_readlane(vinfo, mine ? u - 1 : u - 2);
            i32x4 r[8];
            int poison;
            asm volatile("ds_read_b128 %0, %9\n\tds_read_b128 %1, %9 offset:%11\n\tds_read_b128 %2, %9 offset:2*%11\n\tds_read_b128 %3, %9 offset:3*%11\n\t"
                         "ds_read_b128 %4, %9 offset:4*%11\n\tds_read_b128 %5, %9 offset:5*%11\n\tds_read_b128 %6, %9 offset:6*%11\n\tds_read_b128 %7, %9 offset:7*%11\n\t"
                         "ds_read_b32 %8, %10\n\ts_waitcnt lgkmcnt(0)"
                         : "=&v"(r[0]), "=&v"(r[1]), "=&v"(r[2]), "=&v"(r[3]), "=&v"(r[4]), "=&v"(r[5]), "=&v"(r[6]), "=&v"(r[7]), "=&v"(poison)
                         : "v"(pr0 + (mine ? pb : pb - PBUF)), "v"(lbase + BAD0), "n"(PTILE) : "memory");
            const i32x4 sum = ((r[0] + r[1]) + (r[2] + r[3])) + ((r[4] + r[5]) + (r[6] + r[7]));
            const float sc = poison ? __builtin_nanf("") : scale;
            char* const orow = (char*)obase + (size_t)(info & 255) * (16 * NL * 4);
            const unsigned roff = g * (NL * 4) + (lane & 15) * 16;
#pragma unroll
            for (int e = 0; e < 4; ++e) o[e] = __builtin_bit_cast(unsigned, (float)sum[e] * sc);
            if (!foreign) {
                asm volatile("global_store_dwordx4 %0, %1, %2 sc0 sc1\n\ts_nop 1" :: "v"(roff), "v"(o), "s"(orow) : "memory");
            } else {
                hold = (info >> 8) & 7;
                hdue = info >> 12;
                op = orow; ooff = roff;
                holdv = 1 + (16u << (wave & 3));
                asm volatile("" : "+v"(o));
                const unsigned big = ((unsigned)sum[0] + 0x200000u) | ((unsigned)sum[1] + 0x200000u) | ((unsigned)sum[2] + 0x200000u) | ((unsigned)sum[3] + 0x200000u);
                if (!(info & 0x800) && !__any((big >> 22) | poison)) {
                    const unsigned s0 = sum[0], s1 = sum[1], s2 = sum[2], s3 = sum[3];
                    const unsigned hi = ((s0 >> 16) & 63) | ((s1 >> 10) & (63 << 6)) | ((s2 >> 4) & (63 << 12)) | ((s3 << 2) & (63 << 18));
                    const unsigned pk = lbase + PACK0 + wave * 768;
                    asm volatile("ds_write_b32 %1, %2\n\tds_write_b32 %1, %3 offset:256\n\tds_write_b16 %5, %4 offset:512\n\t"
                                 "ds_write_b8_d16_hi %6, %4 offset:640\n\tds_read_b128 %0, %7\n\ts_waitcnt lgkmcnt(0)"
                                 : "=&v"(o) : "v"(pk + lane * 4), "v"((s0 & 0xffffu) | (s1 << 16)), "v"((s2 & 0xffffu) | (s3 << 16)), "v"(hi),
                                   "v"(pk + lane * 2), "v"(pk + lane), "v"(pk + l16) : "memory");
                    op = ydst + hold * YTILE; ooff = l16;
                    holdv = 1;
                }
                if (hdue == 0) emit();
                else if (u >= nunit || (int)((unsigned)__builtin_amdgcn_s_memrealtime() - ts) + HOLDMIN >= hdue) { pace(hdue); emit(); }
            }
        }
        paired = false;
    }
    asm volatile("s_waitcnt vmcnt(0)" : "+v"(a[0]), "+v"(a[1]), "+v"(a[2]), "+v"(a[3]), "+v"(a[4]), "+v"(a[5]), "+v"(a[6]), "+v"(a[7]), "+v"(fv) :: "memory");
    if (pend >= 0) publish();
    if (tid == 0) __hip_atomic_store(cnt, call, __ATOMIC_RELAXED, __HIP_MEMORY_SCOPE_AGENT);
    EXITSTAMP();
}

extern "C" void kernel_launch(int rank, int nranks, ncclComm_t comm,
                              hipStream_t stream,
                              void* const* d_in, const int* in_sizes, int n_in,
                              void* d_out, int out_size,
                              const int* in_dtypes, int out_dtype,
                              void* const* d_out_all,
                              void* const* d_ws_all, size_t ws_bytes) {
    if (nranks != NR || (unsigned)rank >= (unsigned)NR || n_in != 4 || ws_bytes < WS_NEED
        || in_sizes[0] != ML * KDIM || in_sizes[1] != KDIM * NL || out_size != NR * ML * NL) {
        fprintf(stderr, "ag_gemm: unexpected problem shape\n");
        return;
    }
    HIP_CHECK(hipFuncSetAttribute((const void*)ag_gemm, hipFuncAttributeMaxDynamicSharedMemorySize, LDS_BYTES));
    Params p;
    p.x = (const char*)d_in[0]; p.w = (const char*)d_in[1];
    p.sx = (const float*)d_in[2]; p.sw = (const float*)d_in[3];
    p.rank = rank; p.pad = 0;
    const int me = rank & 3, other = (rank & 4) ^ 4;
    p.myws = (char*)d_ws_all[rank]; p.out = (float*)d_out;
    for (int i = 0; i < NR - 1; ++i) {
        p.link[i].ws = (char*)d_ws_all[i < 3 ? (rank & 4) | ((me + 1 + i) & 3) : other | (i - 3)];
        p.link[i].fin = p.link[i].fout = p.link[i].pad[0] = p.link[i].pad[1] = p.link[i].pad[2] = 0;
    }
    for (int k = 0; k < 4; ++k) p.pout[k] = (float*)d_out_all[other | k];
    p.extra[0] = p.extra[1] = 0;
    int ne = 0;
    for (int f = 0; f < NF; ++f) {
        p.link[3 + ((me + offs(f)) & 3)].fin |= 1u << f;
        for (int m = 0; m < 4; ++m) {
            if (((m + offs(f)) & 3) != me) continue;
            for (int k = 0; k < 4; ++k) if (k != m) p.link[3 + k].fout |= 1u << (4 * f + m);
            p.extra[ne >> 2] |= (unsigned)((other | m) * NRB + ND + f) << (8 * (ne & 3));
            ++ne;
        }
    }
    for (int i = 0; i < NR - 1; ++i) {
        const unsigned fin = p.link[i].fin, rest = fin & (fin - 1);
        const unsigned s0 = rank * NRB + (i < 3 ? 0 : ND + __builtin_ctz(fin));
        const unsigned s1 = i < 3 ? rank * NRB + 1 : rest ? rank * NRB + ND + __builtin_ctz(rest) : NR * NRB + me * 8;
        p.link[i].first = s0 | s1 << 16;
    }
    hipLaunchKernelGGL(ag_gemm, dim3(GRID), dim3(512), LDS_BYTES, stream, p);
    HIP_CHECK(hipGetLastError());
}
```

```cpp
#include <hip/hip_runtime.h>
#include <rccl/rccl.h>
#include <cstdio>
#include <cstdint>

#define HIP_CHECK(expr) do { hipError_t _e = (expr); if (_e != hipSuccess) { \
    fprintf(stderr, "HIP error %s at %s:%d\n", hipGetErrorString(_e), __FILE__, __LINE__); \
    return; } } while (0)

typedef unsigned u32x4 __attribute__((ext_vector_type(4)));
typedef int i32x4 __attribute__((ext_vector_type(4)));
typedef int i32x2 __attribute__((ext_vector_type(2)));
typedef float f32x4 __attribute__((ext_vector_type(4)));
#define LDS_AS __attribute__((address_space(3)))
#define GLB_AS __attribute__((address_space(1)))

#define NF 7
#define HJ 5
#define D1 13
#define ROUND 1116
#define YSLOT 490
#define LEAD 125
#define SLEAD (-20)
#define GAP 110
#define HINTR 4
#define YLEAD 10
#define YROT 112
#define HOLDMIN 25
#define MIDLEAD 10
#define LASTX 4
#define NFREE 6
#define NRAW 10
#define NSGL 3

constexpr int NR = 8;
constexpr int KDIM = 4096;
constexpr int ML = 512;
constexpr int NL = 512;
constexpr int NRB = ML / 16;
constexpr int ND = NRB - NF;
constexpr int RBB = 16 * KDIM;
constexpr unsigned OFFS = NF == 5 ? 0x20321u : NF == 6 ? 0x310321u : 0x03210321u;
__host__ __device__ constexpr int offs(int f) { return (OFFS >> (4 * f)) & 3; }
static_assert(NF >= 4 && NF <= 7 && D1 < ND && HJ >= 1 && HJ <= 7, "plan");
constexpr int GEMM0 = (NR - 1) * 8;
constexpr int GRID = GEMM0 + 8 * 24;
constexpr int WPB = 8;
constexpr int LDS_BYTES = 148 << 10;
constexpr int NSLOT = NR * NRB + 64;
constexpr size_t WS_CNT = 0;
constexpr size_t WS_TMO = 48 << 10;
constexpr size_t WS_ZERO = 50 << 10;
constexpr size_t WS_HINT = 52 << 10;
constexpr size_t WS_FLAG = 256 << 10;
constexpr size_t WS_YFLAG = 128 << 10;
constexpr size_t WS_X = 1 << 20;
static_assert(WS_FLAG + NSLOT * 1024 <= WS_X, "flags");
constexpr size_t XPAR = (size_t)NSLOT * RBB;
constexpr size_t WS_Y = WS_X + 2 * XPAR;
constexpr int YT = 8;
constexpr int YPART = 704, YTILE = 4 * YPART;
constexpr size_t YPAR = (size_t)4 * 32 * YT * YTILE;
constexpr size_t WS_DIAG = WS_Y + 2 * YPAR;
constexpr size_t WS_NEED = WS_DIAG + (4 << 20);
constexpr unsigned long long DEADLINE_TICKS = 100000000ull;

struct Link {
    char* ws;
    unsigned first;
    unsigned fin;
    unsigned fout;
    unsigned firstx;
    unsigned pad[2];
};
struct Params {
    const char* x; const char* w; char* myws;
    int rank; int pad;
    Link link[NR - 1];
    const float* sx; const float* sw;
    float* out; float* pout[4];
    unsigned extra[2];
};

__device__ __forceinline__ int list_slot(int r, int pos) {
    if (pos < NRB) return r * NRB + pos;
    const int q = pos - NRB, t = q / 3, s = q - 3 * t;
    const int rb = t < D1 ? t : (t < D1 + NF ? ND + t - D1 : t - NF);
    return ((r & 4) | ((r + 1 + s) & 3)) * NRB + rb;
}
__device__ __forceinline__ bool held_by(int slot, int k) {
    const int rb = slot & (NRB - 1);
    return rb >= ND && (((slot >> 5) + offs(rb - ND)) & 3) == k;
}

#define MF(d, b, a, c) "v_mfma_i32_16x16x64_i8 %" #d ", %" #b ", %" #a ", " c "\n\t"
#define MM(b0, b1, b2, b3, a) MF(0, b0, a, "%0") MF(1, b1, a, "%1") MF(2, b2, a, "%2") MF(3, b3, a, "%3")
#define MM0 MF(0, 9, 4, "0") MF(1, 10, 4, "0") MF(2, 11, 4, "0") MF(3, 12, 4, "0")
#define LD(a, o) "buffer_load_dwordx4 %" #a ", %25, %26, %27 offen offset:" #o " sc0 sc1\n\t"
#define W(n) "s_waitcnt vmcnt(" #n ")\n\t"
#define BF4(s) "v"(bf[s][0]), "v"(bf[s][1]), "v"(bf[s][2]), "v"(bf[s][3])
#define WAITV(n) asm volatile("s_waitcnt vmcnt(" #n ")" ::: "memory")

#define EXITSTAMP() do {} while (0)

__global__ void __launch_bounds__(512, 2) ag_gemm(Params p) {
    extern __shared__ __attribute__((aligned(16))) char smem[];
    const int tid = threadIdx.x, lane = tid & 63;
    const int wave = __builtin_amdgcn_readfirstlane(tid >> 6);
    const int bid = blockIdx.x;
    const unsigned long long t0 = __builtin_amdgcn_s_memrealtime();
    unsigned ts = (unsigned)t0 + LEAD;
    const Link lk = p.link[bid < GEMM0 ? bid >> 3 : 3 + (((bid - GEMM0) >> 4) & 3)];
    const char* const px = p.x; const char* const pw = p.w;
    const int rank = p.rank, me = rank & 3, other = (rank & 4) ^ 4;
    char* myws = p.myws;
    asm volatile("" :: "s"(lk.ws), "s"(lk.first), "s"(lk.firstx), "s"(px), "s"(pw), "s"(myws), "s"(rank));
    GLB_AS unsigned* cnt = (GLB_AS unsigned*)(myws + WS_CNT + (size_t)bid * 128);
    GLB_AS unsigned* tmo = (GLB_AS unsigned*)(myws + WS_TMO);
    const unsigned lbase = (unsigned)(size_t)(LDS_AS char*)smem;
    unsigned call;
    bool bad = false;
    auto put_word = [&](const void* f, unsigned v) {
        asm volatile("s_mov_b64 exec, 1\n\tglobal_store_dword %0, %1, %2 sc0 sc1\n\ts_mov_b64 exec, -1" :: "v"(0u), "v"(v), "s"(f) : "memory");
    };
    auto give_up = [&]() {
        if (__builtin_amdgcn_readfirstlane(__hip_atomic_load(tmo, __ATOMIC_RELAXED, __HIP_MEMORY_SCOPE_AGENT)) == call) bad = true;
        else if (__builtin_amdgcn_s_memrealtime() - t0 > DEADLINE_TICKS) { put_word(myws + WS_TMO, call); bad = true; }
        return bad;
    };
    auto wait_flag = [&](const char* f, unsigned want) -> unsigned {
        unsigned v = 0;
        if (bad) return v;
        do {
            unsigned x, n;
            asm volatile("s_mov_b64 exec, 1\n\ts_mov_b32 %2, 16\n"
                         "1:\n\tglobal_load_dword %1, %3, %4 sc0 sc1\n\ts_waitcnt vmcnt(0)\n\tv_readfirstlane_b32 %0, %1\n\t"
                         "s_cmp_ge_u32 %0, %5\n\ts_cbranch_scc1 2f\n\ts_sleep 1\n\t"
                         "s_sub_u32 %2, %2, 1\n\ts_cmp_lg_u32 %2, 0\n\ts_cbranch_scc1 1b\n"
                         "2:\n\ts_mov_b64 exec, -1"
                         : "=&s"(v), "=&v"(x), "=&s"(n) : "v"(0u), "s"(f), "s"(want) : "memory", "scc");
            if (__builtin_expect(v >= want, 1)) return v;
        } while (!give_up());
        return 0;
    };
    auto wait_flags4 = [&](const char* f) {
        do {
#pragma unroll 1
            for (int it = 0; it < 8; ++it) {
                unsigned fv;
                asm volatile("s_mov_b64 exec, 15\n\tglobal_load_dword %0, %1, %2 sc0 sc1\n\ts_mov_b64 exec, -1\n\ts_waitcnt vmcnt(0)"
                             : "=&v"(fv) : "v"(lane * 128), "s"(f) : "memory");
                if ((__ballot(fv < call) & 15) == 0) return;
                __builtin_amdgcn_s_sleep(4);
            }
        } while (!give_up());
    };
    auto pace = [&](int due) { while ((int)((unsigned)__builtin_amdgcn_s_memrealtime() - ts) < due) __builtin_amdgcn_s_sleep(1); };
    auto snooze = [&](int due) { while ((int)((unsigned)__builtin_amdgcn_s_memrealtime() - ts) < due) __builtin_amdgcn_s_sleep(8); };
    auto lds_add = [&](unsigned addr, unsigned v) -> unsigned {
        unsigned old;
        asm volatile("s_mov_b64 exec, 1\n\tds_add_rtn_u32 %0, %1, %2\n\ts_mov_b64 exec, -1\n\ts_waitcnt lgkmcnt(0)" : "=&v"(old) : "v"(addr), "v"(v) : "memory");
        return __builtin_amdgcn_readfirstlane(old);
    };

    const int g = lane >> 4;
    if (__builtin_expect(bid < GEMM0, 1)) {
        const int j = bid & 7, pi = bid >> 3;
        const bool cross = pi >= 3;
        const int k = (pi - 3) & 3;
        const int wq = wave & 3, frag = j * 8 + wq * 2;
        const bool ext = j >= HJ;
        const int nfull = cross ? 8 + NF : ND - 1, nround = nfull + (cross == ext);
        const int tail = (cross ? (j + pi) % (8 - HJ) : (j + pi) % HJ) * 4 + wq;
        const unsigned stg = lbase + 1024 + wave * 4096;
        const unsigned xo = (lane >> 3) * KDIM + (((lane & 7) ^ g) << 4), wo = (lane >> 2) * NL + (((lane & 3) ^ (g & 1)) << 4);
        auto fetch = [&](int slot, int i) {
            const bool isw = slot >= NR * NRB;
            const char* base = isw ? pw + (size_t)(((slot & 3) * 8 + j) * 128 + wq * 32) * NL + k * 128 + (slot & 4) * 16
                                   : px + (size_t)(slot & 31) * RBB + frag * 64;
            unsigned keep;
            asm volatile("s_mov_b32 %0, m0\n\ts_mov_b32 m0, %4\n\ts_nop 0\n\t"
                         "global_load_lds_dwordx4 %1, %3\n\ts_add_u32 m0, m0, 1024\n\ts_nop 0\n\t"
                         "global_load_lds_dwordx4 %2, %3\n\ts_mov_b32 m0, %0"
                         : "=&s"(keep) : "v"(isw ? wo : xo), "v"(isw ? (wo ^ 32) + 16 * NL : (xo ^ 64) + 8 * KDIM), "s"(base),
                           "s"(stg + ((i >> 1) & 1) * 2048) : "memory", "scc");
        };
        fetch(((ext ? lk.firstx : lk.first) >> (16 * (wave >> 2))) & 0xffff, 0);
        unsigned c0;
        asm volatile("global_load_dword %0, %1, %2 sc1\n\tds_write_b32 %3, %1" : "=&v"(c0) : "v"(0u), "s"(cnt), "v"(lbase + 4 * lane) : "memory");
        int vslot;
        {
            const unsigned fin = lk.fin | (ext && k == me ? 0x80u : 0u), fout = lk.fout | (ext && k != me ? 0x10000000u << me : 0u);
            const int nin = __builtin_popcount(fin);
            unsigned m = lane < nin ? fin : fout;
            const int c = lane < nin ? lane : lane - nin - 8;
#pragma unroll
            for (int t = 0; t < NF; ++t) m &= m - (c > t);
            const int b = __builtin_ctz(m | 0x80000000u);
            const int f = lane < nin ? b : b >> 2, rb = f < NF ? ND + f : ND - 1;
            vslot = !cross ? rank * NRB + (lane & 31) : lane < nin ? rank * NRB + rb
                  : lane < nin + 8 ? NR * NRB + me * 8 + lane - nin : (other | (b & 3)) * NRB + rb;
        }
        unsigned fseen = 0;
        asm volatile("s_waitcnt lgkmcnt(0)\n\ts_barrier" ::: "memory");
        int early = 0, drift = 0, left = 0;
        unsigned par = 0;
        call = 0;
        char* dst = lk.ws + WS_X + frag * 1024;
        char* flag = lk.ws + WS_FLAG + j * 128;
        const unsigned l16 = lane * 16;
#pragma unroll 1
        for (int i = wave >> 2; i < nround; i += 2) {
            asm volatile("s_cmp_gt_u32 %1, 1\n\ts_cbranch_scc1 1f\n\ts_waitcnt vmcnt(0)\n1:" : "+v"(c0) : "s"(i) : "memory", "scc");
            if (i < 2) {
                call = __builtin_amdgcn_readfirstlane(c0) + 1;
                par = (call & 1) * (unsigned)XPAR;
                ts = (unsigned)__builtin_amdgcn_s_memrealtime() + SLEAD;
            }
            const int slot = __builtin_amdgcn_readlane(vslot, i);
            const bool own = (slot >> 5) == rank;
            const unsigned r0 = stg + ((i >> 1) & 1) * 2048 + (own ? (lane & 15) * 128 + ((g ^ ((lane & 15) >> 1)) << 4) : l16);
            u32x4 v0, v1;
            if (slot < NR * NRB) {
                asm volatile("ds_read_b128 %0, %2\n\tds_read_b128 %1, %3\n\ts_waitcnt lgkmcnt(0)"
                             : "=&v"(v0), "=&v"(v1) : "v"(r0), "v"(own ? r0 ^ 64 : r0 + 1024) : "memory");
            } else {
                const int q = (lane & 15) >> 1;
                const unsigned rt = stg + ((i >> 1) & 1) * 2048 + (16 * (g & 1) + q) * 64 + 8 * (lane & 1) + (((g >> 1) ^ (q >> 2) ^ ((g & 1) << 1)) << 4);
                i32x2 a, b, c, d;
                asm volatile("ds_read_b64_tr_b8 %0, %4\n\tds_read_b64_tr_b8 %1, %4 offset:512\n\t"
                             "ds_read_b64_tr_b8 %2, %5\n\tds_read_b64_tr_b8 %3, %5 offset:512\n\ts_waitcnt lgkmcnt(0)"
                             : "=&v"(a), "=&v"(b), "=&v"(c), "=&v"(d) : "v"(rt), "v"(rt ^ 32) : "memory");
                v0 = (u32x4){(unsigned)a[0], (unsigned)a[1], (unsigned)b[0], (unsigned)b[1]};
                v1 = (u32x4){(unsigned)c[0], (unsigned)c[1], (unsigned)d[0], (unsigned)d[1]};
            }
            const int due = ((i < nfull ? (i * 4 + wq) * 8 + j : i * 32 + tail) * ROUND) / 320;
            pace(max(due, early));
            asm volatile("global_store_dwordx4 %0, %2, %1 sc0 sc1\n\tglobal_store_dwordx4 %0, %3, %1 offset:1024 sc0 sc1\n\ts_nop 1"
                         :: "v"(l16), "s"(dst + (par + (unsigned)slot * RBB)), "v"(v0), "v"(v1) : "memory");
            if (i < 2) fetch(__builtin_amdgcn_readlane(vslot, i + 2), 2);
            if ((i < 2 || (i >> 1) == HINTR) && lane == 0)
                __hip_atomic_store((GLB_AS unsigned long long*)(myws + WS_HINT) + ((i < 2 ? 7 : 0) + pi) * 64 + j * 8 + wave,
                                   ((unsigned long long)call << 32) | (ts + drift * left + max(early - due, 0)), __ATOMIC_RELAXED, __HIP_MEMORY_SCOPE_AGENT);
            unsigned long long hv = 0;
            if ((i >> 1) == 2) hv = __hip_atomic_load((GLB_AS unsigned long long*)(myws + WS_HINT) + (7 + pi) * 64 + lane, __ATOMIC_RELAXED, __HIP_MEMORY_SCOPE_AGENT);
            if (cross && (i >> 1) == 2 && lane < 16) fseen = __hip_atomic_load((GLB_AS unsigned*)(myws + WS_FLAG + (vslot & 255) * 1024 + j * 128), __ATOMIC_RELAXED, __HIP_MEMORY_SCOPE_SYSTEM);
            asm volatile("s_waitcnt vmcnt(0)" : "+v"(fseen), "+v"(hv) :: "memory");
            early = i + 2 < nfull ? (int)((unsigned)__builtin_amdgcn_s_memrealtime() - ts) + GAP : 0;
            if (!bad && lds_add(lbase + 4 * i, 1) == 3) put_word(flag + slot * 1024, call);
            if ((i >> 1) == 2 && __all((unsigned)(hv >> 32) == call)) {
                int d = (int)((unsigned)hv - ts);
#pragma unroll
                for (int o = 32; o; o >>= 1) d += __shfl_xor(d, o);
                drift = __builtin_amdgcn_readfirstlane(min(max(d, -6400), 6400)) / 256;
                left = 4;
            }
            if (left > 0) { ts += drift; early -= drift; --left; }
            if (i + 4 < nround) {
                const int ns = __builtin_amdgcn_readlane(vslot, i + 4);
                if ((ns >> 7) != (other >> 2)) fetch(ns, i);
                else {
                    if (__builtin_amdgcn_readlane(fseen, i + 4) < call) wait_flag(myws + WS_FLAG + ns * 1024 + j * 128, call);
                    unsigned keep;
                    asm volatile("s_mov_b32 %0, m0\n\ts_mov_b32 m0, %3\n\ts_nop 0\n\tglobal_load_lds_dwordx4 %1, %2 sc0 sc1\n\t"
                                 "global_load_lds_dwordx4 %1, %2 offset:1024 sc0 sc1\n\ts_mov_b32 m0, %0"
                                 : "=&s"(keep) : "v"(l16), "s"(myws + WS_X + frag * 1024 + (par + (unsigned)ns * RBB)),
                                   "s"(stg + ((i >> 1) & 1) * 2048) : "memory");
                }
            }
        }
        if (lds_add(lbase + 4 * 63, 1) == WPB - 1) put_word((const void*)cnt, call);
        if (!cross) { EXITSTAMP(); return; }

        const float scale = p.sx[0] * p.sw[0];
        const int prod = j * 4 + wq, h = prod >> 4, rep = prod & 15, e0 = (wave >> 2) * 2;
        const char* yf = myws + WS_YFLAG + (k * 32 + prod) * 128;
        const char* yb = myws + WS_Y + (call & 1) * YPAR + (size_t)(k * 32 + prod) * YT * YTILE + e0 * YPART;
        const int yslot = list_slot(other | k, (lane == 7 ? (rep + LASTX) & 15 : rep) + 16 * (lane & 7));
        unsigned todo = (unsigned)__ballot(lane < 8 && !held_by(yslot, me));
#pragma unroll 1
        for (; todo; todo &= todo - 1) {
            const int n = __builtin_ctz(todo), slot = __builtin_amdgcn_readlane(yslot, n);
            const unsigned raw = (wait_flag(yf + n * 4, call * 16) >> e0) & 3;
            if (raw == 3) continue;
            unsigned d[8], tv;
            asm volatile("global_load_dword %0, %9, %12 sc0 sc1\n\tglobal_load_dword %1, %9, %12 offset:256 sc0 sc1\n\t"
                         "global_load_ushort %2, %10, %12 offset:512 sc0 sc1\n\tglobal_load_ubyte %3, %11, %12 offset:640 sc0 sc1\n\t"
                         "global_load_dword %4, %9, %12 offset:704 sc0 sc1\n\tglobal_load_dword %5, %9, %12 offset:960 sc0 sc1\n\t"
                         "global_load_ushort %6, %10, %12 offset:1216 sc0 sc1\n\tglobal_load_ubyte %7, %11, %12 offset:1344 sc0 sc1\n\t"
                         "global_load_dword %8, %13, %14 sc1\n\ts_waitcnt vmcnt(0)"
                         : "=&v"(d[0]), "=&v"(d[1]), "=&v"(d[2]), "=&v"(d[3]), "=&v"(d[4]), "=&v"(d[5]), "=&v"(d[6]), "=&v"(d[7]), "=&v"(tv)
                         : "v"(lane * 4), "v"(lane * 2), "v"(lane), "s"(yb + n * YTILE), "v"(0u), "s"(myws + WS_TMO) : "memory");
            const float sc = bad || tv == call ? __builtin_nanf("") : scale;
            float* op = p.out + (size_t)(slot * 16 + e0 * 4 + g) * NL + k * 128 + h * 64 + (lane & 15) * 4;
#pragma unroll
            for (int e = 0; e < 2; ++e) {
                if ((raw >> e) & 1) continue;
                const unsigned d0 = d[4 * e], d1 = d[4 * e + 1], hi = d[4 * e + 2] | d[4 * e + 3] << 16;
                f32x4 o;
                o[0] = (float)(((int)(hi << 26) >> 10) | (int)(d0 & 0xffffu)) * sc;
                o[1] = (float)((((int)(hi << 20) >> 26) << 16) | (int)(d0 >> 16)) * sc;
                o[2] = (float)((((int)(hi << 14) >> 26) << 16) | (int)(d1 & 0xffffu)) * sc;
                o[3] = (float)((((int)(hi << 8) >> 26) << 16) | (int)(d1 >> 16)) * sc;
                asm volatile("global_store_dwordx4 %0, %1, off sc0 sc1\n\ts_nop 1" :: "v"(op + e * 4 * NL), "v"(o) : "memory");
            }
        }
        EXITSTAMP();
        return;
    }

    const int gb = bid - GEMM0;
    const int xcd = gb & 7, sl = (gb >> 3) & 7, set = gb >> 6;
    const bool foreign = set > 0;
    const int rep = foreign ? xcd + 8 * (set - 1) : xcd;
    const unsigned c0 = __hip_atomic_load(cnt, __ATOMIC_RELAXED, __HIP_MEMORY_SCOPE_AGENT);
    const float scale = p.sx[0] * p.sw[0];

    constexpr int PTILE = 16 * 272, PBUF = 8 * PTILE, PACK0 = 4 * PBUF, ARR0 = PACK0 + 8 * 768, BAD0 = ARR0 + 32;
    static_assert(ARR0 + 256 + 64 <= LDS_BYTES, "LDS");
    const unsigned l16 = lane * 16;
    const unsigned ast = lbase + wave * 4096;

    if (foreign) {
        const int c = gb - 64, sn = c >> 2, frag = (c & 3) * 16 + wave * 2;
        const char* xs = px + (size_t)sn * RBB + frag * 64;
        const char* wsrc = pw + (size_t)((sn & 3) * 1024 + frag * 16) * NL + (sn >> 2) * 64;
        const unsigned xo = (lane >> 3) * KDIM + (((lane & 7) ^ g) << 4), wo = (lane >> 2) * NL + (((lane & 3) ^ (g & 1)) << 4);
        const int q = (lane & 15) >> 1;
        const unsigned rx = ast + (lane & 15) * 128 + ((g ^ q) << 4);
        const unsigned rt = ast + 2048 + (16 * (g & 1) + q) * 64 + 8 * (lane & 1) + (((g >> 1) ^ (q >> 2) ^ ((g & 1) << 1)) << 4);
        u32x4 v0, v1; i32x2 t0, t1, t2, t3;
        unsigned keep;
        asm volatile("s_mov_b32 %6, m0\n\ts_mov_b32 m0, %13\n\ts_nop 4\n\t"
                     "global_load_lds_dwordx4 %7, %11\n\ts_add_u32 m0, m0, 1024\n\ts_nop 0\n\t"
                     "global_load_lds_dwordx4 %8, %11\n\ts_add_u32 m0, m0, 1024\n\ts_nop 0\n\t"
                     "global_load_lds_dwordx4 %9, %12\n\ts_add_u32 m0, m0, 1024\n\ts_nop 0\n\t"
                     "global_load_lds_dwordx4 %10, %12\n\ts_mov_b32 m0, %6\n\ts_waitcnt vmcnt(0)\n\t"
                     "ds_read_b128 %0, %14\n\tds_read_b128 %1, %15\n\t"
                     "ds_read_b64_tr_b8 %2, %16\n\tds_read_b64_tr_b8 %3, %16 offset:512\n\t"
                     "ds_read_b64_tr_b8 %4, %17\n\tds_read_b64_tr_b8 %5, %17 offset:512\n\ts_waitcnt lgkmcnt(0)"
                     : "=&v"(v0), "=&v"(v1), "=&v"(t0), "=&v"(t1), "=&v"(t2), "=&v"(t3), "=&s"(keep)
                     : "v"(xo), "v"((xo ^ 64) + 8 * KDIM), "v"(wo), "v"((wo ^ 32) + 16 * NL), "s"(xs), "s"(wsrc), "s"(ast),
                       "v"(rx), "v"(rx ^ 64), "v"(rt), "v"(rt ^ 32) : "memory", "scc");
        call = __builtin_amdgcn_readfirstlane(c0) + 1;
        const char* xw = myws + WS_X + (call & 1) * XPAR + frag * 1024;
        const i32x4 v2 = {t0[0], t0[1], t1[0], t1[1]}, v3 = {t2[0], t2[1], t3[0], t3[1]};
        asm volatile("global_store_dwordx4 %0, %5, %2 sc0 sc1\n\tglobal_store_dwordx4 %0, %6, %2 offset:1024 sc0 sc1\n\t"
                     "global_store_dwordx4 %0, %3, %1 sc0 sc1\n\tglobal_store_dwordx4 %0, %4, %1 offset:1024 sc0 sc1\n\t"
                     "s_waitcnt vmcnt(0)\n\ts_barrier"
                     :: "v"(l16), "s"(xw + (size_t)(rank * NRB + sn) * RBB), "s"(xw + (size_t)(NR * NRB + 32 + sn) * RBB),
                        "v"(v0), "v"(v1), "v"(v2), "v"(v3) : "memory");
        if (tid < 4) __hip_atomic_store((GLB_AS unsigned*)(myws + WS_FLAG + ((tid & 2 ? NR * NRB + 32 : rank * NRB) + sn) * 1024 + ((c & 3) * 2 + (tid & 1)) * 128),
                                        call, __ATOMIC_RELAXED, __HIP_MEMORY_SCOPE_SYSTEM);
    }
    call = __builtin_amdgcn_readfirstlane(c0) + 1;
    const char* xws = myws + WS_X + (call & 1) * XPAR;
    if (tid < 16) *(unsigned*)(smem + ARR0 + tid * 4) = 0;
    asm volatile("s_waitcnt lgkmcnt(0)\n\ts_barrier" ::: "memory");

    unsigned vinfo, vflag;
    int nunit;
    {
        const int pos = min(foreign ? (lane == 7 ? (rep + LASTX) & 15 : rep) + 16 * lane : rep + 8 * (lane - (lane > 4)), 127);
        const int slot = !foreign && lane == 4 ? (int)((p.extra[xcd >> 2] >> (8 * (xcd & 3))) & 255) : list_slot(rank, pos);
        const bool use = foreign ? lane < 8 && !held_by(slot, sl >> 1) : lane < 17 && (lane != 4 || xcd < NF);
        const unsigned mask = (unsigned)__ballot(use);
        const int pr = pos < YROT ? (pos & ~7) | ((pos + (sl & 6)) & 7) : pos;
        const int c4 = 4 * pr - min(max(pr - ND, 0), NF) - min(max(pr - NRB - 3 * D1, 0), 3 * NF);
        const int due = pos >= 128 - NFREE ? 0 : ((8 * (9 + NF) - HJ) * ROUND) / 80 + ((c4 + 2 * (sl & 1)) * YSLOT) / 200;
        nunit = __builtin_popcount(mask);
        vinfo = __builtin_amdgcn_ds_permute((use ? __builtin_amdgcn_mbcnt_lo(mask, 0) : 63) * 4, slot | (lane & 7) << 8 | (pos >= 128 - NRAW) << 11 | due << 12);
        vflag = lane < nunit ? (unsigned)WS_FLAG + wave * 128 + (vinfo & 255) * 1024 : (unsigned)WS_ZERO;
    }
    const unsigned vaoff = (vinfo & 255) << 16;
    char* ydst = lk.ws + WS_Y + (call & 1) * YPAR + (size_t)(me * 32 + (sl & 1) * 16 + rep) * YT * YTILE + (wave & 3) * YPART;
    char* yflag = lk.ws + WS_YFLAG + (me * 32 + (sl & 1) * 16 + rep) * 128;
    float* obase = (foreign ? p.pout[sl >> 1] + me * 128 + (sl & 1) * 64 : p.out + sl * 64) + (wave & 3) * 4 * NL;

    int hold = -1, hdue = 0;
    unsigned holdv = 0;
    u32x4 o = {0, 0, 0, 0};
    char* op = ydst;
    unsigned ooff = l16;
    int u = 0;
    int pend = -1;
    auto emit = [&]() {
        asm volatile("s_mov_b32 exec_hi, %3\n\tglobal_store_dwordx4 %0, %1, %2 sc0 sc1\n\ts_mov_b32 exec_hi, -1\n\ts_nop 0"
                     :: "v"(ooff), "v"(o), "s"(op), "s"(holdv > 1 ? ~0u : (1u << (YPART / 16 - 32)) - 1) : "memory");
        pend = hold; hold = -1;
    };
    auto announce = [&]() {
        const unsigned now = lds_add(lbase + ARR0 + pend * 4, holdv) + holdv;
        if ((now & 15) == 4) put_word(yflag + pend * 4, call * 16 + (now >> 4));
        pend = -1;
    };
    auto publish = [&]() { WAITV(0); announce(); };

    i32x4 bf[8][4];
    {
        const int wslot = NR * NRB + (foreign ? 0 : 32) + sl * 4 + (wave >> 1);
        if (foreign) snooze(((2 + (sl & 1) * 4 + (wave >> 1)) * ROUND) / 10);
        wait_flags4(myws + WS_FLAG + wslot * 1024 + (wave & 1) * 512);
        asm volatile("" ::: "memory");
        const __amdgpu_buffer_rsrc_t rs = __builtin_amdgcn_make_buffer_rsrc((void*)(xws + (size_t)wslot * RBB + (wave & 1) * 32768), 0, 32768, 0x00020000);
#pragma unroll
        for (int s = 0; s < 8; ++s)
#pragma unroll
            for (int t = 0; t < 4; ++t)
                bf[s][t] = __builtin_bit_cast(i32x4, __builtin_amdgcn_raw_buffer_load_b128(rs, (lane >> 5) * 2048 + (lane & 31) * 16 + t * 512, s * 4096, 17));
#pragma unroll
        for (int s = 0; s < 8; ++s) asm volatile("" : "+v"(bf[s][0]), "+v"(bf[s][1]), "+v"(bf[s][2]), "+v"(bf[s][3]));
    }
    if (foreign) {
        snooze(((2 * HINTR + 3) * ROUND) / 10);
        const unsigned long long h = __hip_atomic_load((GLB_AS unsigned long long*)(myws + WS_HINT) + (3 + (sl >> 1)) * 64 + lane, __ATOMIC_RELAXED, __HIP_MEMORY_SCOPE_AGENT);
        int d = (unsigned)(h >> 32) == call ? (int)((unsigned)h - (unsigned)t0) : LEAD;
#pragma unroll
        for (int o = 32; o; o >>= 1) d += __shfl_xor(d, o);
        d >>= 6;
        ts = (unsigned)t0 + __builtin_amdgcn_readfirstlane(min(max(d, 0), 1000)) + YLEAD;
    }
    i32x4 a[8];
#pragma unroll
    for (int s = 0; s < 8; ++s) a[s] = (i32x4){0, 0, 0, 0};
    unsigned fv = 0, fa = (unsigned)WS_TMO;
    u32x4 ars;
    ars[0] = (unsigned)(size_t)xws; ars[1] = (unsigned)((size_t)xws >> 32) & 0xffff; ars[2] = (unsigned)XPAR; ars[3] = 0x00020000;
    const unsigned alo = wave * 8192 + l16, ahi = alo + 4096;
    const unsigned pw0 = lbase + wave * PTILE + (lane & 15) * 272 + g * 16;
    const unsigned pr0 = lbase + ((wave & 3) * 4 + g) * 272 + (lane & 15) * 16;
    auto poison_all = [&]() { asm volatile("ds_write_b32 %0, %1" :: "v"(lbase + BAD0), "v"(1) : "memory"); };
    constexpr int RUN = 3, LAST = 9, FETCH = 5;
    int md = LAST;
    auto peek = [&](const char* f) -> unsigned {
        unsigned v;
        asm volatile("s_mov_b64 exec, 1\n\tglobal_load_dword %0, %1, %2 sc0 sc1\n\ts_mov_b64 exec, -1\n\ts_waitcnt vmcnt(0)"
                     : "=&v"(v) : "v"(0u), "s"(f) : "memory");
        return __builtin_amdgcn_readfirstlane(v);
    };
    bool paired = false;
#pragma unroll 1
    while (u < nunit) {
        asm volatile("s_waitcnt vmcnt(7)" : "+v"(a[0]), "+v"(fv) :: "memory");
        int un;
        if (__builtin_expect(md != LAST, 1)) {
            unsigned nx = __builtin_amdgcn_readlane(fv, 0);
            if (nx < call && u + 1 < nunit) nx = peek(myws + __builtin_amdgcn_readlane(vflag, u + 1));
            md = nx >= call ? RUN : LAST;
            if (__builtin_expect(__builtin_amdgcn_readlane(fv, 1) == call, 0)) poison_all();
            un = u + 1;
        } else {
            const char* f = myws + __builtin_amdgcn_readlane(vflag, u);
            bool seen = false;
            while ((hold & pend) >= 0) {
                seen = peek(f) >= call;
                if (seen) break;
                if (pend >= 0) announce();
                else if ((int)((unsigned)__builtin_amdgcn_s_memrealtime() - ts) >= hdue) emit();
            }
            if (!seen) wait_flag(f, call);
            if (bad) poison_all();
            md = FETCH;
            un = u;
        }
        const unsigned soff = __builtin_amdgcn_readlane(vaoff, un);
        asm volatile("s_nop 1\n\tv_writelane_b32 %1, %3, 0\n\tglobal_load_dword %0, %1, %2 sc0 sc1"
                     : "+v"(fv), "+v"(fa) : "s"(myws), "s"(__builtin_amdgcn_readlane(vflag, un + 1)), "s"(soff) : "memory");
        i32x4 acc[4];
        asm volatile("s_cmp_lg_u32 %28, 3\n\ts_cbranch_scc1 1f\n\t"
                     MM0 LD(4, 0) W(8) MM(13, 14, 15, 16, 5) LD(5, 1024) W(8) MM(17, 18, 19, 20, 6) LD(6, 2048) W(8) MM(21, 22, 23, 24, 7) LD(7, 3072) W(8)
                     "s_branch 3f\n1:\n\ts_cmp_lg_u32 %28, 9\n\ts_cbranch_scc1 2f\n\t"
                     MM0 W(7) MM(13, 14, 15, 16, 5) W(6) MM(17, 18, 19, 20, 6) W(5) MM(21, 22, 23, 24, 7) W(4)
                     "s_branch 3f\n2:\n\t" LD(4, 0) LD(5, 1024) LD(6, 2048) LD(7, 3072) "\n3:"
                     : "=&v"(acc[0]), "=&v"(acc[1]), "=&v"(acc[2]), "=&v"(acc[3]), "+v"(a[0]), "+v"(a[1]), "+v"(a[2]), "+v"(a[3]), "+v"(a[4])
                     : BF4(0), BF4(1), BF4(2), BF4(3), "v"(alo), "s"(ars), "s"(soff), "s"(md) : "memory", "scc");
        bool mid = false;
        if (hold >= 0 && (int)((unsigned)__builtin_amdgcn_s_memrealtime() - ts) + MIDLEAD >= hdue) { emit(); mid = true; }
        asm volatile("s_cmp_lg_u32 %28, 3\n\ts_cbranch_scc1 1f\n\t"
                     MM(9, 10, 11, 12, 4) LD(4, 0) W(8) MM(13, 14, 15, 16, 5) LD(5, 1024) W(8) MM(17, 18, 19, 20, 6) LD(6, 2048) W(8) MM(21, 22, 23, 24, 7) LD(7, 3072)
                     "s_branch 3f\n1:\n\ts_cmp_lg_u32 %28, 9\n\ts_cbranch_scc1 2f\n\t"
                     MM(9, 10, 11, 12, 4) W(3) MM(13, 14, 15, 16, 5) W(2) MM(17, 18, 19, 20, 6) W(1) MM(21, 22, 23, 24, 7)
                     "s_branch 3f\n2:\n\t" LD(4, 0) LD(5, 1024) LD(6, 2048) LD(7, 3072) "\n3:\n\ts_nop 7"
                     : "+v"(acc[0]), "+v"(acc[1]), "+v"(acc[2]), "+v"(acc[3]), "+v"(a[4]), "+v"(a[5]), "+v"(a[6]), "+v"(a[7]), "+v"(a[0])
                     : BF4(4), BF4(5), BF4(6), BF4(7), "v"(ahi), "s"(ars), "s"(soff), "s"(md) : "memory", "scc");
        if (md == FETCH) { if (pend >= 0 && !mid) announce(); continue; }
        const unsigned pb = (u & 3) * PBUF;
        asm volatile("ds_write_b128 %0, %1\n\tds_write_b128 %0, %2 offset:64\n\tds_write_b128 %0, %3 offset:128\n\tds_write_b128 %0, %4 offset:192"
                     :: "v"(pw0 + pb), "v"(acc[0]), "v"(acc[1]), "v"(acc[2]), "v"(acc[3]) : "memory");
        ++u;
        if (!foreign && (u & 1) && u < nunit - NSGL) { paired = true; continue; }
        const bool mine = (wave >> 2) != (u & 1);
        const bool after = pend >= 0 && !mine && !mid;
        if (pend >= 0 && mine) publish();
        if (hold >= 0) { pace(hdue); emit(); }
        asm volatile("s_waitcnt lgkmcnt(0)\n\ts_barrier" ::: "memory");
        if (after) publish();
        if (mine || paired) {
            const unsigned info = __builtin_amdgcn_readlane(vinfo, mine ? u - 1 : u - 2);
            i32x4 r[8];
            int poison;
            asm volatile("ds_read_b128 %0, %9\n\tds_read_b128 %1, %9 offset:%11\n\tds_read_b128 %2, %9 offset:2*%11\n\tds_read_b128 %3, %9 offset:3*%11\n\t"
                         "ds_read_b128 %4, %9 offset:4*%11\n\tds_read_b128 %5, %9 offset:5*%11\n\tds_read_b128 %6, %9 offset:6*%11\n\tds_read_b128 %7, %9 offset:7*%11\n\t"
                         "ds_read_b32 %8, %10\n\ts_waitcnt lgkmcnt(0)"
                         : "=&v"(r[0]), "=&v"(r[1]), "=&v"(r[2]), "=&v"(r[3]), "=&v"(r[4]), "=&v"(r[5]), "=&v"(r[6]), "=&v"(r[7]), "=&v"(poison)
                         : "v"(pr0 + (mine ? pb : pb - PBUF)), "v"(lbase + BAD0), "n"(PTILE) : "memory");
            const i32x4 sum = ((r[0] + r[1]) + (r[2] + r[3])) + ((r[4] + r[5]) + (r[6] + r[7]));
            const float sc = poison ? __builtin_nanf("") : scale;
            char* const orow = (char*)obase + (size_t)(info & 255) * (16 * NL * 4);
            const unsigned roff = g * (NL * 4) + (lane & 15) * 16;
#pragma unroll
            for (int e = 0; e < 4; ++e) o[e] = __builtin_bit_cast(unsigned, (float)sum[e] * sc);
            if (!foreign) {
                asm volatile("global_store_dwordx4 %0, %1, %2 sc0 sc1\n\ts_nop 1" :: "v"(roff), "v"(o), "s"(orow) : "memory");
            } else {
                hold = (info >> 8) & 7;
                hdue = info >> 12;
                op = orow; ooff = roff;
                holdv = 1 + (16u << (wave & 3));
                asm volatile("" : "+v"(o));
                const unsigned big = ((unsigned)sum[0] + 0x200000u) | ((unsigned)sum[1] + 0x200000u) | ((unsigned)sum[2] + 0x200000u) | ((unsigned)sum[3] + 0x200000u);
                if (!(info & 0x800) && !__any((big >> 22) | poison)) {
                    const unsigned s0 = sum[0], s1 = sum[1], s2 = sum[2], s3 = sum[3];
                    const unsigned hi = ((s0 >> 16) & 63) | ((s1 >> 10) & (63 << 6)) | ((s2 >> 4) & (63 << 12)) | ((s3 << 2) & (63 << 18));
                    const unsigned pk = lbase + PACK0 + wave * 768;
                    asm volatile("ds_write_b32 %1, %2\n\tds_write_b32 %1, %3 offset:256\n\tds_write_b16 %5, %4 offset:512\n\t"
                                 "ds_write_b8_d16_hi %6, %4 offset:640\n\tds_read_b128 %0, %7\n\ts_waitcnt lgkmcnt(0)"
                                 : "=&v"(o) : "v"(pk + lane * 4), "v"((s0 & 0xffffu) | (s1 << 16)), "v"((s2 & 0xffffu) | (s3 << 16)), "v"(hi),
                                   "v"(pk + lane * 2), "v"(pk + lane), "v"(pk + l16) : "memory");
                    op = ydst + hold * YTILE; ooff = l16;
                    holdv = 1;
                }
                if (u >= nunit) { if (hdue) pace(hdue); emit(); }
                else {
                    const int left = hdue - (int)((unsigned)__builtin_amdgcn_s_memrealtime() - ts);
                    if (left <= HOLDMIN) { if (left > 4) pace(hdue); emit(); }
                }
            }
        }
        paired = false;
    }
    asm volatile("s_waitcnt vmcnt(0)" : "+v"(a[0]), "+v"(a[1]), "+v"(a[2]), "+v"(a[3]), "+v"(a[4]), "+v"(a[5]), "+v"(a[6]), "+v"(a[7]), "+v"(fv) :: "memory");
    if (pend >= 0) publish();
    if (tid == 0) __hip_atomic_store(cnt, call, __ATOMIC_RELAXED, __HIP_MEMORY_SCOPE_AGENT);
    EXITSTAMP();
}

extern "C" void kernel_launch(int rank, int nranks, ncclComm_t comm,
                              hipStream_t stream,
                              void* const* d_in, const int* in_sizes, int n_in,
                              void* d_out, int out_size,
                              const int* in_dtypes, int out_dtype,
                              void* const* d_out_all,
                              void* const* d_ws_all, size_t ws_bytes) {
    if (nranks != NR || (unsigned)rank >= (unsigned)NR || n_in != 4 || ws_bytes < WS_NEED
        || in_sizes[0] != ML * KDIM || in_sizes[1] != KDIM * NL || out_size != NR * ML * NL) {
        fprintf(stderr, "ag_gemm: unexpected problem shape\n");
        return;
    }
    HIP_CHECK(hipFuncSetAttribute((const void*)ag_gemm, hipFuncAttributeMaxDynamicSharedMemorySize, LDS_BYTES));
    Params p;
    p.x = (const char*)d_in[0]; p.w = (const char*)d_in[1];
    p.sx = (const float*)d_in[2]; p.sw = (const float*)d_in[3];
    p.rank = rank; p.pad = 0;
    const int me = rank & 3, other = (rank & 4) ^ 4;
    p.myws = (char*)d_ws_all[rank]; p.out = (float*)d_out;
    for (int i = 0; i < NR - 1; ++i) {
        p.link[i].ws = (char*)d_ws_all[i < 3 ? (rank & 4) | ((me + 1 + i) & 3) : other | (i - 3)];
        p.link[i].fin = p.link[i].fout = p.link[i].pad[0] = p.link[i].pad[1] = 0;
    }
    for (int k = 0; k < 4; ++k) p.pout[k] = (float*)d_out_all[other | k];
    p.extra[0] = p.extra[1] = 0;
    int ne = 0;
    for (int f = 0; f < NF; ++f) {
        p.link[3 + ((me + offs(f)) & 3)].fin |= 1u << f;
        for (int m = 0; m < 4; ++m) {
            if (((m + offs(f)) & 3) != me) continue;
            for (int k = 0; k < 4; ++k) if (k != m) p.link[3 + k].fout |= 1u << (4 * f + m);
            p.extra[ne >> 2] |= (unsigned)((other | m) * NRB + ND + f) << (8 * (ne & 3));
            ++ne;
        }
    }
    for (int i = 0; i < NR - 1; ++i) {
        const unsigned fin = p.link[i].fin, rest = fin & (fin - 1);
        const unsigned s0 = rank * NRB + (i < 3 ? 0 : ND + __builtin_ctz(fin));
        const unsigned s1 = i < 3 ? rank * NRB + 1 : rest ? rank * NRB + ND + __builtin_ctz(rest) : NR * NRB + me * 8;
        p.link[i].first = s0 | s1 << 16;
        p.link[i].firstx = i == 3 + me && !rest ? s0 | (rank * NRB + ND - 1) << 16 : p.link[i].first;
    }
    hipLaunchKernelGGL(ag_gemm, dim3(GRID), dim3(512), LDS_BYTES, stream, p);
    HIP_CHECK(hipGetLastError());
}
```

```cpp
#include <hip/hip_runtime.h>
#include <rccl/rccl.h>
#include <cstdio>
#include <cstdint>

#define HIP_CHECK(expr) do { hipError_t _e = (expr); if (_e != hipSuccess) { \
    fprintf(stderr, "HIP error %s at %s:%d\n", hipGetErrorString(_e), __FILE__, __LINE__); \
    return; } } while (0)

typedef unsigned u32x4 __attribute__((ext_vector_type(4)));
typedef int i32x4 __attribute__((ext_vector_type(4)));
typedef int i32x2 __attribute__((ext_vector_type(2)));
typedef float f32x4 __attribute__((ext_vector_type(4)));
#define LDS_AS __attribute__((address_space(3)))
#define GLB_AS __attribute__((address_space(1)))

#define NF 7
#define HJ 4
#define D1 13
#define ROUND 1116
#define YSLOT 490
#define LEAD 125
#define SLEAD (-20)
#define GAP 110
#define HINTR 4
#define YLEAD 8
#define YROT 112
#define HOLDMIN 25
#define MIDLEAD 10
#define RAWW 4
#define NRAW 10
#define NSGL 3

constexpr int NR = 8;
constexpr int KDIM = 4096;
constexpr int ML = 512;
constexpr int NL = 512;
constexpr int NRB = ML / 16;
constexpr int ND = NRB - NF;
constexpr int RBB = 16 * KDIM;
constexpr unsigned OFFS = NF == 5 ? 0x20321u : NF == 6 ? 0x310321u : 0x03210321u;
__host__ __device__ constexpr int offs(int f) { return (OFFS >> (4 * f)) & 3; }
static_assert(NF >= 4 && NF <= 7 && D1 < ND && HJ >= 1 && HJ <= 7, "plan");
constexpr int GEMM0 = (NR - 1) * 8;
constexpr int GRID = GEMM0 + 8 * 24;
constexpr int WPB = 8;
constexpr int LDS_BYTES = 148 << 10;
constexpr int NSLOT = NR * NRB + 64;
constexpr size_t WS_CNT = 0;
constexpr size_t WS_TMO = 48 << 10;
constexpr size_t WS_ZERO = 50 << 10;
constexpr size_t WS_HINT = 52 << 10;
constexpr size_t WS_FLAG = 256 << 10;
constexpr size_t WS_YFLAG = 128 << 10;
constexpr size_t WS_X = 1 << 20;
static_assert(WS_FLAG + NSLOT * 1024 <= WS_X, "flags");
constexpr size_t XPAR = (size_t)NSLOT * RBB;
constexpr size_t WS_Y = WS_X + 2 * XPAR;
constexpr int YT = 8;
constexpr int YPART = 704, YTILE = 4 * YPART;
constexpr size_t YPAR = (size_t)4 * 32 * YT * YTILE;
constexpr size_t WS_DIAG = WS_Y + 2 * YPAR;
constexpr size_t WS_NEED = WS_DIAG + (4 << 20);
constexpr unsigned long long DEADLINE_TICKS = 100000000ull;

struct Link {
    char* ws;
    unsigned first;
    unsigned fin;
    unsigned fout;
    unsigned firstx;
    unsigned pad[2];
};
struct Params {
    const char* x; const char* w; char* myws;
    int rank; int pad;
    Link link[NR - 1];
    const float* sx; const float* sw;
    float* out; float* pout[4];
    unsigned extra[2];
};

__device__ __forceinline__ int list_slot(int r, int pos) {
    if (pos < NRB) return r * NRB + pos;
    const int q = pos - NRB, t = q / 3, s = q - 3 * t;
    const int rb = t < D1 ? t : (t < D1 + NF ? ND + t - D1 : t - NF);
    return ((r & 4) | ((r + 1 + s) & 3)) * NRB + rb;
}
__device__ __forceinline__ bool held_by(int slot, int k) {
    const int rb = slot & (NRB - 1);
    return rb >= ND && (((slot >> 5) + offs(rb - ND)) & 3) == k;
}

#define MF(d, b, a, c) "v_mfma_i32_16x16x64_i8 %" #d ", %" #b ", %" #a ", " c "\n\t"
#define MM(b0, b1, b2, b3, a) MF(0, b0, a, "%0") MF(1, b1, a, "%1") MF(2, b2, a, "%2") MF(3, b3, a, "%3")
#define MM0 MF(0, 9, 4, "0") MF(1, 10, 4, "0") MF(2, 11, 4, "0") MF(3, 12, 4, "0")
#define LD(a, o) "buffer_load_dwordx4 %" #a ", %25, %26, %27 offen offset:" #o " sc0 sc1\n\t"
#define W(n) "s_waitcnt vmcnt(" #n ")\n\t"
#define BF4(s) "v"(bf[s][0]), "v"(bf[s][1]), "v"(bf[s][2]), "v"(bf[s][3])
#define WAITV(n) asm volatile("s_waitcnt vmcnt(" #n ")" ::: "memory")

#define EXITSTAMP() do {} while (0)

__global__ void __launch_bounds__(512, 2) ag_gemm(Params p) {
    extern __shared__ __attribute__((aligned(16))) char smem[];
    const int tid = threadIdx.x, lane = tid & 63;
    const int wave = __builtin_amdgcn_readfirstlane(tid >> 6);
    const int bid = blockIdx.x;
    const unsigned long long t0 = __builtin_amdgcn_s_memrealtime();
    unsigned ts = (unsigned)t0 + LEAD;
    const Link lk = p.link[bid < GEMM0 ? bid >> 3 : 3 + (((bid - GEMM0) >> 4) & 3)];
    const char* const px = p.x; const char* const pw = p.w;
    const int rank = p.rank, me = rank & 3, other = (rank & 4) ^ 4;
    char* myws = p.myws;
    asm volatile("" :: "s"(lk.ws), "s"(lk.first), "s"(lk.firstx), "s"(px), "s"(pw), "s"(myws), "s"(rank));
    GLB_AS unsigned* cnt = (GLB_AS unsigned*)(myws + WS_CNT + (size_t)bid * 128);
    GLB_AS unsigned* tmo = (GLB_AS unsigned*)(myws + WS_TMO);
    const unsigned lbase = (unsigned)(size_t)(LDS_AS char*)smem;
    unsigned call;
    bool bad = false;
    auto put_word = [&](const void* f, unsigned v) {
        asm volatile("s_mov_b64 exec, 1\n\tglobal_store_dword %0, %1, %2 sc0 sc1\n\ts_mov_b64 exec, -1" :: "v"(0u), "v"(v), "s"(f) : "memory");
    };
    auto give_up = [&]() {
        if (__builtin_amdgcn_readfirstlane(__hip_atomic_load(tmo, __ATOMIC_RELAXED, __HIP_MEMORY_SCOPE_AGENT)) == call) bad = true;
        else if (__builtin_amdgcn_s_memrealtime() - t0 > DEADLINE_TICKS) { put_word(myws + WS_TMO, call); bad = true; }
        return bad;
    };
    auto wait_flag = [&](const char* f, unsigned want) -> unsigned {
        unsigned v = 0;
        if (bad) return v;
        do {
            unsigned x, n;
            asm volatile("s_mov_b64 exec, 1\n\ts_mov_b32 %2, 16\n"
                         "1:\n\tglobal_load_dword %1, %3, %4 sc0 sc1\n\ts_waitcnt vmcnt(0)\n\tv_readfirstlane_b32 %0, %1\n\t"
                         "s_cmp_ge_u32 %0, %5\n\ts_cbranch_scc1 2f\n\ts_sleep 1\n\t"
                         "s_sub_u32 %2, %2, 1\n\ts_cmp_lg_u32 %2, 0\n\ts_cbranch_scc1 1b\n"
                         "2:\n\ts_mov_b64 exec, -1"
                         : "=&s"(v), "=&v"(x), "=&s"(n) : "v"(0u), "s"(f), "s"(want) : "memory", "scc");
            if (__builtin_expect(v >= want, 1)) return v;
        } while (!give_up());
        return 0;
    };
    auto wait_flags4 = [&](const char* f) {
        do {
#pragma unroll 1
            for (int it = 0; it < 8; ++it) {
                unsigned fv;
                asm volatile("s_mov_b64 exec, 15\n\tglobal_load_dword %0, %1, %2 sc0 sc1\n\ts_mov_b64 exec, -1\n\ts_waitcnt vmcnt(0)"
                             : "=&v"(fv) : "v"(lane * 128), "s"(f) : "memory");
                if ((__ballot(fv < call) & 15) == 0) return;
                __builtin_amdgcn_s_sleep(4);
            }
        } while (!give_up());
    };
    auto pace = [&](int due) { while ((int)((unsigned)__builtin_amdgcn_s_memrealtime() - ts) < due) __builtin_amdgcn_s_sleep(1); };
    auto snooze = [&](int due) { while ((int)((unsigned)__builtin_amdgcn_s_memrealtime() - ts) < due) __builtin_amdgcn_s_sleep(8); };
    auto lds_add = [&](unsigned addr, unsigned v) -> unsigned {
        unsigned old;
        asm volatile("s_mov_b64 exec, 1\n\tds_add_rtn_u32 %0, %1, %2\n\ts_mov_b64 exec, -1\n\ts_waitcnt lgkmcnt(0)" : "=&v"(old) : "v"(addr), "v"(v) : "memory");
        return __builtin_amdgcn_readfirstlane(old);
    };

    const int g = lane >> 4;
    if (__builtin_expect(bid < GEMM0, 1)) {
        const int j = bid & 7, pi = bid >> 3;
        const bool cross = pi >= 3;
        const int k = (pi - 3) & 3;
        const int wq = wave & 3, frag = j * 8 + wq * 2;
        const bool ext = j >= HJ;
        const int nfull = cross ? 8 + NF : ND - 1, nround = nfull + (cross == ext);
        const int tail = (cross ? (j + pi) % (8 - HJ) : (j + pi) % HJ) * 4 + wq;
        const unsigned stg = lbase + 1024 + wave * 4096;
        const unsigned xo = (lane >> 3) * KDIM + (((lane & 7) ^ g) << 4), wo = (lane >> 2) * NL + (((lane & 3) ^ (g & 1)) << 4);
        auto fetch = [&](int slot, int i) {
            const bool isw = slot >= NR * NRB;
            const char* base = isw ? pw + (size_t)(((slot & 3) * 8 + j) * 128 + wq * 32) * NL + k * 128 + (slot & 4) * 16
                                   : px + (size_t)(slot & 31) * RBB + frag * 64;
            unsigned keep;
            asm volatile("s_mov_b32 %0, m0\n\ts_mov_b32 m0, %4\n\ts_nop 0\n\t"
                         "global_load_lds_dwordx4 %1, %3\n\ts_add_u32 m0, m0, 1024\n\ts_nop 0\n\t"
                         "global_load_lds_dwordx4 %2, %3\n\ts_mov_b32 m0, %0"
                         : "=&s"(keep) : "v"(isw ? wo : xo), "v"(isw ? (wo ^ 32) + 16 * NL : (xo ^ 64) + 8 * KDIM), "s"(base),
                           "s"(stg + ((i >> 1) & 1) * 2048) : "memory", "scc");
        };
        fetch(((ext ? lk.firstx : lk.first) >> (16 * (wave >> 2))) & 0xffff, 0);
        unsigned c0;
        asm volatile("global_load_dword %0, %1, %2 sc1\n\tds_write_b32 %3, %1" : "=&v"(c0) : "v"(0u), "s"(cnt), "v"(lbase + 4 * lane) : "memory");
        int vslot;
        {
            const unsigned fin = lk.fin | (ext && k == me ? 0x80u : 0u), fout = lk.fout | (ext && k != me ? 0x10000000u << me : 0u);
            const int nin = __builtin_popcount(fin);
            unsigned m = lane < nin ? fin : fout;
            const int c = lane < nin ? lane : lane - nin - 8;
#pragma unroll
            for (int t = 0; t < NF; ++t) m &= m - (c > t);
            const int b = __builtin_ctz(m | 0x80000000u);
            const int f = lane < nin ? b : b >> 2, rb = f < NF ? ND + f : ND - 1;
            vslot = !cross ? rank * NRB + (lane & 31) : lane < nin ? rank * NRB + rb
                  : lane < nin + 8 ? NR * NRB + me * 8 + lane - nin : (other | (b & 3)) * NRB + rb;
        }
        unsigned fseen = 0;
        asm volatile("s_waitcnt lgkmcnt(0)\n\ts_barrier" ::: "memory");
        int early = 0, drift = 0, left = 0;
        unsigned par = 0;
        call = 0;
        char* dst = lk.ws + WS_X + frag * 1024;
        char* flag = lk.ws + WS_FLAG + j * 128;
        const unsigned l16 = lane * 16;
#pragma unroll 1
        for (int i = wave >> 2; i < nround; i += 2) {
            asm volatile("s_cmp_gt_u32 %1, 1\n\ts_cbranch_scc1 1f\n\ts_waitcnt vmcnt(0)\n1:" : "+v"(c0) : "s"(i) : "memory", "scc");
            if (i < 2) {
                call = __builtin_amdgcn_readfirstlane(c0) + 1;
                par = (call & 1) * (unsigned)XPAR;
                ts = (unsigned)__builtin_amdgcn_s_memrealtime() + SLEAD;
            }
            const int slot = __builtin_amdgcn_readlane(vslot, i);
            const bool own = (slot >> 5) == rank;
            const unsigned r0 = stg + ((i >> 1) & 1) * 2048 + (own ? (lane & 15) * 128 + ((g ^ ((lane & 15) >> 1)) << 4) : l16);
            u32x4 v0, v1;
            if (slot < NR * NRB) {
                asm volatile("ds_read_b128 %0, %2\n\tds_read_b128 %1, %3\n\ts_waitcnt lgkmcnt(0)"
                             : "=&v"(v0), "=&v"(v1) : "v"(r0), "v"(own ? r0 ^ 64 : r0 + 1024) : "memory");
            } else {
                const int q = (lane & 15) >> 1;
                const unsigned rt = stg + ((i >> 1) & 1) * 2048 + (16 * (g & 1) + q) * 64 + 8 * (lane & 1) + (((g >> 1) ^ (q >> 2) ^ ((g & 1) << 1)) << 4);
                i32x2 a, b, c, d;
                asm volatile("ds_read_b64_tr_b8 %0, %4\n\tds_read_b64_tr_b8 %1, %4 offset:512\n\t"
                             "ds_read_b64_tr_b8 %2, %5\n\tds_read_b64_tr_b8 %3, %5 offset:512\n\ts_waitcnt lgkmcnt(0)"
                             : "=&v"(a), "=&v"(b), "=&v"(c), "=&v"(d) : "v"(rt), "v"(rt ^ 32) : "memory");
                v0 = (u32x4){(unsigned)a[0], (unsigned)a[1], (unsigned)b[0], (unsigned)b[1]};
                v1 = (u32x4){(unsigned)c[0], (unsigned)c[1], (unsigned)d[0], (unsigned)d[1]};
            }
            const int due = ((i < nfull ? (i * 4 + wq) * 8 + j : i * 32 + tail) * ROUND) / 320;
            pace(max(due, early));
            asm volatile("global_store_dwordx4 %0, %2, %1 sc0 sc1\n\tglobal_store_dwordx4 %0, %3, %1 offset:1024 sc0 sc1\n\ts_nop 1"
                         :: "v"(l16), "s"(dst + (par + (unsigned)slot * RBB)), "v"(v0), "v"(v1) : "memory");
            if (i < 2) fetch(__builtin_amdgcn_readlane(vslot, i + 2), 2);
            if ((i < 2 || (i >> 1) == HINTR) && lane == 0)
                __hip_atomic_store((GLB_AS unsigned long long*)(myws + WS_HINT) + ((i < 2 ? 7 : 0) + pi) * 64 + j * 8 + wave,
                                   ((unsigned long long)call << 32) | (ts + drift * left + max(early - due, 0)), __ATOMIC_RELAXED, __HIP_MEMORY_SCOPE_AGENT);
            unsigned long long hv = 0;
            if ((i >> 1) == 2) hv = __hip_atomic_load((GLB_AS unsigned long long*)(myws + WS_HINT) + (7 + pi) * 64 + lane, __ATOMIC_RELAXED, __HIP_MEMORY_SCOPE_AGENT);
            if (cross && (i >> 1) == 2 && lane < 16) fseen = __hip_atomic_load((GLB_AS unsigned*)(myws + WS_FLAG + (vslot & 255) * 1024 + j * 128), __ATOMIC_RELAXED, __HIP_MEMORY_SCOPE_SYSTEM);
            asm volatile("s_waitcnt vmcnt(0)" : "+v"(fseen), "+v"(hv) :: "memory");
            early = i + 2 < nfull ? (int)((unsigned)__builtin_amdgcn_s_memrealtime() - ts) + GAP : 0;
            if (!bad && lds_add(lbase + 4 * i, 1) == 3) put_word(flag + slot * 1024, call);
            if ((i >> 1) == 2 && __all((unsigned)(hv >> 32) == call)) {
                int d = (int)((unsigned)hv - ts);
#pragma unroll
                for (int o = 32; o; o >>= 1) d += __shfl_xor(d, o);
                drift = __builtin_amdgcn_readfirstlane(min(max(d, -6400), 6400)) / 256;
                left = 4;
            }
            if (left > 0) { ts += drift; early -= drift; --left; }
            if (i + 4 < nround) {
                const int ns = __builtin_amdgcn_readlane(vslot, i + 4);
                if ((ns >> 7) != (other >> 2)) fetch(ns, i);
                else {
                    if (__builtin_amdgcn_readlane(fseen, i + 4) < call) wait_flag(myws + WS_FLAG + ns * 1024 + j * 128, call);
                    unsigned keep;
                    asm volatile("s_mov_b32 %0, m0\n\ts_mov_b32 m0, %3\n\ts_nop 0\n\tglobal_load_lds_dwordx4 %1, %2 sc0 sc1\n\t"
                                 "global_load_lds_dwordx4 %1, %2 offset:1024 sc0 sc1\n\ts_mov_b32 m0, %0"
                                 : "=&s"(keep) : "v"(l16), "s"(myws + WS_X + frag * 1024 + (par + (unsigned)ns * RBB)),
                                   "s"(stg + ((i >> 1) & 1) * 2048) : "memory");
                }
            }
        }
        if (lds_add(lbase + 4 * 63, 1) == WPB - 1) put_word((const void*)cnt, call);
        if (!cross) { EXITSTAMP(); return; }

        const float scale = p.sx[0] * p.sw[0];
        const int prod = j * 4 + wq, h = prod >> 4, rep = prod & 15, e0 = (wave >> 2) * 2;
        const char* yf = myws + WS_YFLAG + (k * 32 + prod) * 128;
        const char* yb = myws + WS_Y + (call & 1) * YPAR + (size_t)(k * 32 + prod) * YT * YTILE + e0 * YPART;
        const int yslot = list_slot(other | k, rep + 16 * (lane & 7));
        unsigned todo = (unsigned)__ballot(lane < 8 && !held_by(yslot, me));
#pragma unroll 1
        for (; todo; todo &= todo - 1) {
            const int n = __builtin_ctz(todo), slot = __builtin_amdgcn_readlane(yslot, n);
            const unsigned raw = (wait_flag(yf + n * 4, call * 16) >> e0) & 3;
            if (raw == 3) continue;
            unsigned d[8], tv;
            asm volatile("global_load_dword %0, %9, %12 sc0 sc1\n\tglobal_load_dword %1, %9, %12 offset:256 sc0 sc1\n\t"
                         "global_load_ushort %2, %10, %12 offset:512 sc0 sc1\n\tglobal_load_ubyte %3, %11, %12 offset:640 sc0 sc1\n\t"
                         "global_load_dword %4, %9, %12 offset:704 sc0 sc1\n\tglobal_load_dword %5, %9, %12 offset:960 sc0 sc1\n\t"
                         "global_load_ushort %6, %10, %12 offset:1216 sc0 sc1\n\tglobal_load_ubyte %7, %11, %12 offset:1344 sc0 sc1\n\t"
                         "global_load_dword %8, %13, %14 sc1\n\ts_waitcnt vmcnt(0)"
                         : "=&v"(d[0]), "=&v"(d[1]), "=&v"(d[2]), "=&v"(d[3]), "=&v"(d[4]), "=&v"(d[5]), "=&v"(d[6]), "=&v"(d[7]), "=&v"(tv)
                         : "v"(lane * 4), "v"(lane * 2), "v"(lane), "s"(yb + n * YTILE), "v"(0u), "s"(myws + WS_TMO) : "memory");
            const float sc = bad || tv == call ? __builtin_nanf("") : scale;
            float* op = p.out + (size_t)(slot * 16 + e0 * 4 + g) * NL + k * 128 + h * 64 + (lane & 15) * 4;
#pragma unroll
            for (int e = 0; e < 2; ++e) {
                if ((raw >> e) & 1) continue;
                const unsigned d0 = d[4 * e], d1 = d[4 * e + 1], hi = d[4 * e + 2] | d[4 * e + 3] << 16;
                f32x4 o;
                o[0] = (float)(((int)(hi << 26) >> 10) | (int)(d0 & 0xffffu)) * sc;
                o[1] = (float)((((int)(hi << 20) >> 26) << 16) | (int)(d0 >> 16)) * sc;
                o[2] = (float)((((int)(hi << 14) >> 26) << 16) | (int)(d1 & 0xffffu)) * sc;
                o[3] = (float)((((int)(hi << 8) >> 26) << 16) | (int)(d1 >> 16)) * sc;
                asm volatile("global_store_dwordx4 %0, %1, off sc0 sc1\n\ts_nop 1" :: "v"(op + e * 4 * NL), "v"(o) : "memory");
            }
        }
        EXITSTAMP();
        return;
    }

    const int gb = bid - GEMM0;
    const int xcd = gb & 7, sl = (gb >> 3) & 7, set = gb >> 6;
    const bool foreign = set > 0;
    const int rep = foreign ? xcd + 8 * (set - 1) : xcd;
    const unsigned c0 = __hip_atomic_load(cnt, __ATOMIC_RELAXED, __HIP_MEMORY_SCOPE_AGENT);
    const float scale = p.sx[0] * p.sw[0];

    constexpr int PTILE = 16 * 272, PBUF = 8 * PTILE, PACK0 = 4 * PBUF, ARR0 = PACK0 + 8 * 768, BAD0 = ARR0 + 32;
    static_assert(ARR0 + 256 + 64 <= LDS_BYTES, "LDS");
    const unsigned l16 = lane * 16;
    const unsigned ast = lbase + wave * 4096;

    if (foreign) {
        const int c = gb - 64, sn = c >> 2, frag = (c & 3) * 16 + wave * 2;
        const char* xs = px + (size_t)sn * RBB + frag * 64;
        const char* wsrc = pw + (size_t)((sn & 3) * 1024 + frag * 16) * NL + (sn >> 2) * 64;
        const unsigned xo = (lane >> 3) * KDIM + (((lane & 7) ^ g) << 4), wo = (lane >> 2) * NL + (((lane & 3) ^ (g & 1)) << 4);
        const int q = (lane & 15) >> 1;
        const unsigned rx = ast + (lane & 15) * 128 + ((g ^ q) << 4);
        const unsigned rt = ast + 2048 + (16 * (g & 1) + q) * 64 + 8 * (lane & 1) + (((g >> 1) ^ (q >> 2) ^ ((g & 1) << 1)) << 4);
        u32x4 v0, v1; i32x2 t0, t1, t2, t3;
        unsigned keep;
        asm volatile("s_mov_b32 %6, m0\n\ts_mov_b32 m0, %13\n\ts_nop 4\n\t"
                     "global_load_lds_dwordx4 %7, %11\n\ts_add_u32 m0, m0, 1024\n\ts_nop 0\n\t"
                     "global_load_lds_dwordx4 %8, %11\n\ts_add_u32 m0, m0, 1024\n\ts_nop 0\n\t"
                     "global_load_lds_dwordx4 %9, %12\n\ts_add_u32 m0, m0, 1024\n\ts_nop 0\n\t"
                     "global_load_lds_dwordx4 %10, %12\n\ts_mov_b32 m0, %6\n\ts_waitcnt vmcnt(0)\n\t"
                     "ds_read_b128 %0, %14\n\tds_read_b128 %1, %15\n\t"
                     "ds_read_b64_tr_b8 %2, %16\n\tds_read_b64_tr_b8 %3, %16 offset:512\n\t"
                     "ds_read_b64_tr_b8 %4, %17\n\tds_read_b64_tr_b8 %5, %17 offset:512\n\ts_waitcnt lgkmcnt(0)"
                     : "=&v"(v0), "=&v"(v1), "=&v"(t0), "=&v"(t1), "=&v"(t2), "=&v"(t3), "=&s"(keep)
                     : "v"(xo), "v"((xo ^ 64) + 8 * KDIM), "v"(wo), "v"((wo ^ 32) + 16 * NL), "s"(xs), "s"(wsrc), "s"(ast),
                       "v"(rx), "v"(rx ^ 64), "v"(rt), "v"(rt ^ 32) : "memory", "scc");
        call = __builtin_amdgcn_readfirstlane(c0) + 1;
        const char* xw = myws + WS_X + (call & 1) * XPAR + frag * 1024;
        const i32x4 v2 = {t0[0], t0[1], t1[0], t1[1]}, v3 = {t2[0], t2[1], t3[0], t3[1]};
        asm volatile("global_store_dwordx4 %0, %5, %2 sc0 sc1\n\tglobal_store_dwordx4 %0, %6, %2 offset:1024 sc0 sc1\n\t"
                     "global_store_dwordx4 %0, %3, %1 sc0 sc1\n\tglobal_store_dwordx4 %0, %4, %1 offset:1024 sc0 sc1\n\t"
                     "s_waitcnt vmcnt(0)\n\ts_barrier"
                     :: "v"(l16), "s"(xw + (size_t)(rank * NRB + sn) * RBB), "s"(xw + (size_t)(NR * NRB + 32 + sn) * RBB),
                        "v"(v0), "v"(v1), "v"(v2), "v"(v3) : "memory");
        if (tid < 4) __hip_atomic_store((GLB_AS unsigned*)(myws + WS_FLAG + ((tid & 2 ? NR * NRB + 32 : rank * NRB) + sn) * 1024 + ((c & 3) * 2 + (tid & 1)) * 128),
                                        call, __ATOMIC_RELAXED, __HIP_MEMORY_SCOPE_SYSTEM);
    }
    call = __builtin_amdgcn_readfirstlane(c0) + 1;
    const char* xws = myws + WS_X + (call & 1) * XPAR;
    if (tid < 16) *(unsigned*)(smem + ARR0 + tid * 4) = 0;
    asm volatile("s_waitcnt lgkmcnt(0)\n\ts_barrier" ::: "memory");

    unsigned vinfo, vflag;
    int nunit;
    {
        const int pos = min(foreign ? rep + 16 * lane : rep + 8 * (lane - (lane > 4)), 127);
        const int slot = !foreign && lane == 4 ? (int)((p.extra[xcd >> 2] >> (8 * (xcd & 3))) & 255) : list_slot(rank, pos);
        const bool use = foreign ? lane < 8 && !held_by(slot, sl >> 1) : lane < 17 && (lane != 4 || xcd < NF);
        const unsigned mask = (unsigned)__ballot(use);
        const int pr = pos < YROT ? (pos & ~7) | ((pos + (sl & 6)) & 7) : pos;
        const int c4 = 4 * pr - min(max(pr - ND, 0), NF) - min(max(pr - NRB - 3 * D1, 0), 3 * NF) + max(pr - (128 - NRAW), 0) * RAWW / 4;
        const int due = ((8 * (9 + NF) - HJ) * ROUND) / 80 + ((c4 + 2 * (sl & 1)) * YSLOT) / 200;
        nunit = __builtin_popcount(mask);
        vinfo = __builtin_amdgcn_ds_permute((use ? __builtin_amdgcn_mbcnt_lo(mask, 0) : 63) * 4, slot | (lane & 7) << 8 | (pos >= 128 - NRAW) << 11 | due << 12);
        vflag = lane < nunit ? (unsigned)WS_FLAG + wave * 128 + (vinfo & 255) * 1024 : (unsigned)WS_ZERO;
    }
    const unsigned vaoff = (vinfo & 255) << 16;
    char* ydst = lk.ws + WS_Y + (call & 1) * YPAR + (size_t)(me * 32 + (sl & 1) * 16 + rep) * YT * YTILE + (wave & 3) * YPART;
    char* yflag = lk.ws + WS_YFLAG + (me * 32 + (sl & 1) * 16 + rep) * 128;
    float* obase = (foreign ? p.pout[sl >> 1] + me * 128 + (sl & 1) * 64 : p.out + sl * 64) + (wave & 3) * 4 * NL;

    int hold = -1, hdue = 0;
    unsigned holdv = 0;
    u32x4 o = {0, 0, 0, 0};
    char* op = ydst;
    unsigned ooff = l16;
    int u = 0;
    int pend = -1;
    auto emit = [&]() {
        asm volatile("s_mov_b32 exec_hi, %3\n\tglobal_store_dwordx4 %0, %1, %2 sc0 sc1\n\ts_mov_b32 exec_hi, -1\n\ts_nop 0"
                     :: "v"(ooff), "v"(o), "s"(op), "s"(holdv > 1 ? ~0u : (1u << (YPART / 16 - 32)) - 1) : "memory");
        pend = hold; hold = -1;
    };
    auto announce = [&]() {
        const unsigned now = lds_add(lbase + ARR0 + pend * 4, holdv) + holdv;
        if ((now & 15) == 4) put_word(yflag + pend * 4, call * 16 + (now >> 4));
        pend = -1;
    };
    auto publish = [&]() { WAITV(0); announce(); };

    i32x4 a[8], bf[8][4];
    wait_flag(myws + __builtin_amdgcn_readlane(vflag, 0), call);
    {
        const __amdgpu_buffer_rsrc_t rs = __builtin_amdgcn_make_buffer_rsrc((void*)(xws + __builtin_amdgcn_readlane(vaoff, 0) + wave * 8192), 0, 8192, 0x00020000);
#pragma unroll
        for (int s = 0; s < 8; ++s) a[s] = __builtin_bit_cast(i32x4, __builtin_amdgcn_raw_buffer_load_b128(rs, l16, s * 1024, 17));
    }
    {
        const int wslot = NR * NRB + (foreign ? 0 : 32) + sl * 4 + (wave >> 1);
        if (foreign) snooze(((2 + (sl & 1) * 4 + (wave >> 1)) * ROUND) / 10);
        wait_flags4(myws + WS_FLAG + wslot * 1024 + (wave & 1) * 512);
        asm volatile("" ::: "memory");
        const __amdgpu_buffer_rsrc_t rs = __builtin_amdgcn_make_buffer_rsrc((void*)(xws + (size_t)wslot * RBB + (wave & 1) * 32768), 0, 32768, 0x00020000);
#pragma unroll
        for (int s = 0; s < 8; ++s)
#pragma unroll
            for (int t = 0; t < 4; ++t)
                bf[s][t] = __builtin_bit_cast(i32x4, __builtin_amdgcn_raw_buffer_load_b128(rs, (lane >> 5) * 2048 + (lane & 31) * 16 + t * 512, s * 4096, 17));
#pragma unroll
        for (int s = 0; s < 8; ++s) asm volatile("" : "+v"(bf[s][0]), "+v"(bf[s][1]), "+v"(bf[s][2]), "+v"(bf[s][3]), "+v"(a[s]));
    }
    if (foreign) {
        snooze(((2 * HINTR + 3) * ROUND) / 10);
        const unsigned long long h = __hip_atomic_load((GLB_AS unsigned long long*)(myws + WS_HINT) + (3 + (sl >> 1)) * 64 + lane, __ATOMIC_RELAXED, __HIP_MEMORY_SCOPE_AGENT);
        int d = (unsigned)(h >> 32) == call ? (int)((unsigned)h - (unsigned)t0) : LEAD;
#pragma unroll
        for (int o = 32; o; o >>= 1) d += __shfl_xor(d, o);
        d >>= 6;
        ts = (unsigned)t0 + __builtin_amdgcn_readfirstlane(min(max(d, 0), 1000)) + YLEAD;
    }
    unsigned fv = 0, fa = (unsigned)WS_TMO;
    u32x4 ars;
    ars[0] = (unsigned)(size_t)xws; ars[1] = (unsigned)((size_t)xws >> 32) & 0xffff; ars[2] = (unsigned)XPAR; ars[3] = 0x00020000;
    const unsigned alo = wave * 8192 + l16, ahi = alo + 4096;
    const unsigned pw0 = lbase + wave * PTILE + (lane & 15) * 272 + g * 16;
    const unsigned pr0 = lbase + ((wave & 3) * 4 + g) * 272 + (lane & 15) * 16;
    auto poison_all = [&]() { asm volatile("ds_write_b32 %0, %1" :: "v"(lbase + BAD0), "v"(1) : "memory"); };
    constexpr int RUN = 3, LAST = 9, FETCH = 5;
    int md = FETCH;
    if (bad) poison_all();
    auto peek = [&](const char* f) -> unsigned {
        unsigned v;
        asm volatile("s_mov_b64 exec, 1\n\tglobal_load_dword %0, %1, %2 sc0 sc1\n\ts_mov_b64 exec, -1\n\ts_waitcnt vmcnt(0)"
                     : "=&v"(v) : "v"(0u), "s"(f) : "memory");
        return __builtin_amdgcn_readfirstlane(v);
    };
    bool paired = false;
#pragma unroll 1
    while (u < nunit) {
        asm volatile("s_waitcnt vmcnt(7)" : "+v"(a[0]), "+v"(fv) :: "memory");
        int un;
        if (__builtin_expect(md != LAST, 1)) {
            unsigned nx = __builtin_amdgcn_readlane(fv, 0);
            if (nx < call && u + 1 < nunit) nx = peek(myws + __builtin_amdgcn_readlane(vflag, u + 1));
            md = nx >= call ? RUN : LAST;
            if (__builtin_expect(__builtin_amdgcn_readlane(fv, 1) == call, 0)) poison_all();
            un = u + 1;
        } else {
            const char* f = myws + __builtin_amdgcn_readlane(vflag, u);
            bool seen = false;
            while ((hold & pend) >= 0) {
                seen = peek(f) >= call;
                if (seen) break;
                if (pend >= 0) announce();
                else if ((int)((unsigned)__builtin_amdgcn_s_memrealtime() - ts) >= hdue) emit();
            }
            if (!seen) wait_flag(f, call);
            if (bad) poison_all();
            md = FETCH;
            un = u;
        }
        const unsigned soff = __builtin_amdgcn_readlane(vaoff, un);
        asm volatile("s_nop 1\n\tv_writelane_b32 %1, %3, 0\n\tglobal_load_dword %0, %1, %2 sc0 sc1"
                     : "+v"(fv), "+v"(fa) : "s"(myws), "s"(__builtin_amdgcn_readlane(vflag, un + 1)), "s"(soff) : "memory");
        i32x4 acc[4];
        asm volatile("s_cmp_lg_u32 %28, 3\n\ts_cbranch_scc1 1f\n\t"
                     MM0 LD(4, 0) W(8) MM(13, 14, 15, 16, 5) LD(5, 1024) W(8) MM(17, 18, 19, 20, 6) LD(6, 2048) W(8) MM(21, 22, 23, 24, 7) LD(7, 3072) W(8)
                     "s_branch 3f\n1:\n\ts_cmp_lg_u32 %28, 9\n\ts_cbranch_scc1 2f\n\t"
                     MM0 W(7) MM(13, 14, 15, 16, 5) W(6) MM(17, 18, 19, 20, 6) W(5) MM(21, 22, 23, 24, 7) W(4)
                     "s_branch 3f\n2:\n\t" LD(4, 0) LD(5, 1024) LD(6, 2048) LD(7, 3072) "\n3:"
                     : "=&v"(acc[0]), "=&v"(acc[1]), "=&v"(acc[2]), "=&v"(acc[3]), "+v"(a[0]), "+v"(a[1]), "+v"(a[2]), "+v"(a[3]), "+v"(a[4])
                     : BF4(0), BF4(1), BF4(2), BF4(3), "v"(alo), "s"(ars), "s"(soff), "s"(md) : "memory", "scc");
        bool mid = false;
        if (hold >= 0 && (int)((unsigned)__builtin_amdgcn_s_memrealtime() - ts) + MIDLEAD >= hdue) { emit(); mid = true; }
        asm volatile("s_cmp_lg_u32 %28, 3\n\ts_cbranch_scc1 1f\n\t"
                     MM(9, 10, 11, 12, 4) LD(4, 0) W(8) MM(13, 14, 15, 16, 5) LD(5, 1024) W(8) MM(17, 18, 19, 20, 6) LD(6, 2048) W(8) MM(21, 22, 23, 24, 7) LD(7, 3072)
                     "s_branch 3f\n1:\n\ts_cmp_lg_u32 %28, 9\n\ts_cbranch_scc1 2f\n\t"
                     MM(9, 10, 11, 12, 4) W(3) MM(13, 14, 15, 16, 5) W(2) MM(17, 18, 19, 20, 6) W(1) MM(21, 22, 23, 24, 7)
                     "s_branch 3f\n2:\n\t" LD(4, 0) LD(5, 1024) LD(6, 2048) LD(7, 3072) "\n3:\n\ts_nop 7"
                     : "+v"(acc[0]), "+v"(acc[1]), "+v"(acc[2]), "+v"(acc[3]), "+v"(a[4]), "+v"(a[5]), "+v"(a[6]), "+v"(a[7]), "+v"(a[0])
                     : BF4(4), BF4(5), BF4(6), BF4(7), "v"(ahi), "s"(ars), "s"(soff), "s"(md) : "memory", "scc");
        if (md == FETCH) { if (pend >= 0 && !mid) announce(); continue; }
        const unsigned pb = (u & 3) * PBUF;
        asm volatile("ds_write_b128 %0, %1\n\tds_write_b128 %0, %2 offset:64\n\tds_write_b128 %0, %3 offset:128\n\tds_write_b128 %0, %4 offset:192"
                     :: "v"(pw0 + pb), "v"(acc[0]), "v"(acc[1]), "v"(acc[2]), "v"(acc[3]) : "memory");
        ++u;
        if (!foreign && (u & 1) && u < nunit - NSGL) { paired = true; continue; }
        const bool mine = (wave >> 2) != (u & 1);
        const bool after = pend >= 0 && !mine && !mid;
        if (pend >= 0 && mine) publish();
        if (hold >= 0) { pace(hdue); emit(); }
        asm volatile("s_waitcnt lgkmcnt(0)\n\ts_barrier" ::: "memory");
        if (after) publish();
        if (mine || paired) {
            const unsigned info = __builtin_amdgcn_readlane(vinfo, mine ? u - 1 : u - 2);
            i32x4 r[8];
            int poison;
            asm volatile("ds_read_b128 %0, %9\n\tds_read_b128 %1, %9 offset:%11\n\tds_read_b128 %2, %9 offset:2*%11\n\tds_read_b128 %3, %9 offset:3*%11\n\t"
                         "ds_read_b128 %4, %9 offset:4*%11\n\tds_read_b128 %5, %9 offset:5*%11\n\tds_read_b128 %6, %9 offset:6*%11\n\tds_read_b128 %7, %9 offset:7*%11\n\t"
                         "ds_read_b32 %8, %10\n\ts_waitcnt lgkmcnt(0)"
                         : "=&v"(r[0]), "=&v"(r[1]), "=&v"(r[2]), "=&v"(r[3]), "=&v"(r[4]), "=&v"(r[5]), "=&v"(r[6]), "=&v"(r[7]), "=&v"(poison)
                         : "v"(pr0 + (mine ? pb : pb - PBUF)), "v"(lbase + BAD0), "n"(PTILE) : "memory");
            const i32x4 sum = ((r[0] + r[1]) + (r[2] + r[3])) + ((r[4] + r[5]) + (r[6] + r[7]));
            const float sc = poison ? __builtin_nanf("") : scale;
            char* const orow = (char*)obase + (size_t)(info & 255) * (16 * NL * 4);
            const unsigned roff = g * (NL * 4) + (lane & 15) * 16;
#pragma unroll
            for (int e = 0; e < 4; ++e) o[e] = __builtin_bit_cast(unsigned, (float)sum[e] * sc);
            if (!foreign) {
                asm volatile("global_store_dwordx4 %0, %1, %2 sc0 sc1\n\ts_nop 1" :: "v"(roff), "v"(o), "s"(orow) : "memory");
            } else {
                hold = (info >> 8) & 7;
                hdue = info >> 12;
                op = orow; ooff = roff;
                holdv = 1 + (16u << (wave & 3));
                asm volatile("" : "+v"(o));
                const unsigned big = ((unsigned)sum[0] + 0x200000u) | ((unsigned)sum[1] + 0x200000u) | ((unsigned)sum[2] + 0x200000u) | ((unsigned)sum[3] + 0x200000u);
                if (!(info & 0x800) && !__any((big >> 22) | poison)) {
                    const unsigned s0 = sum[0], s1 = sum[1], s2 = sum[2], s3 = sum[3];
                    const unsigned hi = ((s0 >> 16) & 63) | ((s1 >> 10) & (63 << 6)) | ((s2 >> 4) & (63 << 12)) | ((s3 << 2) & (63 << 18));
                    const unsigned pk = lbase + PACK0 + wave * 768;
                    asm volatile("ds_write_b32 %1, %2\n\tds_write_b32 %1, %3 offset:256\n\tds_write_b16 %5, %4 offset:512\n\t"
                                 "ds_write_b8_d16_hi %6, %4 offset:640\n\tds_read_b128 %0, %7\n\ts_waitcnt lgkmcnt(0)"
                                 : "=&v"(o) : "v"(pk + lane * 4), "v"((s0 & 0xffffu) | (s1 << 16)), "v"((s2 & 0xffffu) | (s3 << 16)), "v"(hi),
                                   "v"(pk + lane * 2), "v"(pk + lane), "v"(pk + l16) : "memory");
                    op = ydst + hold * YTILE; ooff = l16;
                    holdv = 1;
                }
                if (u >= nunit) { pace(hdue); emit(); }
                else {
                    const int left = hdue - (int)((unsigned)__builtin_amdgcn_s_memrealtime() - ts);
                    if (left <= HOLDMIN) { if (left > 4) pace(hdue); emit(); }
                }
            }
        }
        paired = false;
    }
    asm volatile("s_waitcnt vmcnt(0)" : "+v"(a[0]), "+v"(a[1]), "+v"(a[2]), "+v"(a[3]), "+v"(a[4]), "+v"(a[5]), "+v"(a[6]), "+v"(a[7]), "+v"(fv) :: "memory");
    if (pend >= 0) publish();
    if (tid == 0) __hip_atomic_store(cnt, call, __ATOMIC_RELAXED, __HIP_MEMORY_SCOPE_AGENT);
    EXITSTAMP();
}

extern "C" void kernel_launch(int rank, int nranks, ncclComm_t comm,
                              hipStream_t stream,
                              void* const* d_in, const int* in_sizes, int n_in,
                              void* d_out, int out_size,
                              const int* in_dtypes, int out_dtype,
                              void* const* d_out_all,
                              void* const* d_ws_all, size_t ws_bytes) {
    if (nranks != NR || (unsigned)rank >= (unsigned)NR || n_in != 4 || ws_bytes < WS_NEED
        || in_sizes[0] != ML * KDIM || in_sizes[1] != KDIM * NL || out_size != NR * ML * NL) {
        fprintf(stderr, "ag_gemm: unexpected problem shape\n");
        return;
    }
    HIP_CHECK(hipFuncSetAttribute((const void*)ag_gemm, hipFuncAttributeMaxDynamicSharedMemorySize, LDS_BYTES));
    Params p;
    p.x = (const char*)d_in[0]; p.w = (const char*)d_in[1];
    p.sx = (const float*)d_in[2]; p.sw = (const float*)d_in[3];
    p.rank = rank; p.pad = 0;
    const int me = rank & 3, other = (rank & 4) ^ 4;
    p.myws = (char*)d_ws_all[rank]; p.out = (float*)d_out;
    for (int i = 0; i < NR - 1; ++i) {
        p.link[i].ws = (char*)d_ws_all[i < 3 ? (rank & 4) | ((me + 1 + i) & 3) : other | (i - 3)];
        p.link[i].fin = p.link[i].fout = p.link[i].pad[0] = p.link[i].pad[1] = 0;
    }
    for (int k = 0; k < 4; ++k) p.pout[k] = (float*)d_out_all[other | k];
    p.extra[0] = p.extra[1] = 0;
    int ne = 0;
    for (int f = 0; f < NF; ++f) {
        p.link[3 + ((me + offs(f)) & 3)].fin |= 1u << f;
        for (int m = 0; m < 4; ++m) {
            if (((m + offs(f)) & 3) != me) continue;
            for (int k = 0; k < 4; ++k) if (k != m) p.link[3 + k].fout |= 1u << (4 * f + m);
            p.extra[ne >> 2] |= (unsigned)((other | m) * NRB + ND + f) << (8 * (ne & 3));
            ++ne;
        }
    }
    for (int i = 0; i < NR - 1; ++i) {
        const unsigned fin = p.link[i].fin, rest = fin & (fin - 1);
        const unsigned s0 = rank * NRB + (i < 3 ? 0 : ND + __builtin_ctz(fin));
        const unsigned s1 = i < 3 ? rank * NRB + 1 : rest ? rank * NRB + ND + __builtin_ctz(rest) : NR * NRB + me * 8;
        p.link[i].first = s0 | s1 << 16;
        p.link[i].firstx = i == 3 + me && !rest ? s0 | (rank * NRB + ND - 1) << 16 : p.link[i].first;
    }
    hipLaunchKernelGGL(ag_gemm, dim3(GRID), dim3(512), LDS_BYTES, stream, p);
    HIP_CHECK(hipGetLastError());
}
```

```cpp
#include <hip/hip_runtime.h>
#include <rccl/rccl.h>
#include <cstdio>
#include <cstdint>

#define HIP_CHECK(expr) do { hipError_t _e = (expr); if (_e != hipSuccess) { \
    fprintf(stderr, "HIP error %s at %s:%d\n", hipGetErrorString(_e), __FILE__, __LINE__); \
    return; } } while (0)

typedef unsigned u32x4 __attribute__((ext_vector_type(4)));
typedef int i32x4 __attribute__((ext_vector_type(4)));
typedef int i32x2 __attribute__((ext_vector_type(2)));
typedef float f32x4 __attribute__((ext_vector_type(4)));
#define LDS_AS __attribute__((address_space(3)))
#define GLB_AS __attribute__((address_space(1)))

constexpr int NF = 7;
constexpr int HJ = 4;
constexpr int D1 = 13;
constexpr int ROUND = 1116;
constexpr int YSLOT = 490;
constexpr int LEAD = 125;
constexpr int SLEAD = -20;
constexpr int GAP = 110;
constexpr int HINTR = 4;
constexpr int YLEAD = 8;
constexpr int YROT = 112;
constexpr int HOLDMIN = 25;
constexpr int EPI = 20;
constexpr int MIDLEAD = 10;
constexpr int RAWW = 4;
constexpr int NRAW = 10;
constexpr int NSGL = 3;

constexpr int NR = 8;
constexpr int KDIM = 4096;
constexpr int ML = 512;
constexpr int NL = 512;
constexpr int NRB = ML / 16;
constexpr int ND = NRB - NF;
constexpr int RBB = 16 * KDIM;
constexpr unsigned OFFS = NF == 5 ? 0x20321u : NF == 6 ? 0x310321u : 0x03210321u;
__host__ __device__ constexpr int offs(int f) { return (OFFS >> (4 * f)) & 3; }
static_assert(NF >= 4 && NF <= 7 && D1 < ND && HJ >= 1 && HJ <= 7, "plan");
constexpr int GEMM0 = (NR - 1) * 8;
constexpr int GRID = GEMM0 + 8 * 24;
constexpr int WPB = 8;
constexpr int LDS_BYTES = 148 << 10;
constexpr int NSLOT = NR * NRB + 64;
constexpr size_t WS_CNT = 0;
constexpr size_t WS_TMO = 48 << 10;
constexpr size_t WS_ZERO = 50 << 10;
constexpr size_t WS_HINT = 52 << 10;
constexpr size_t WS_FLAG = 256 << 10;
constexpr size_t WS_YFLAG = 128 << 10;
constexpr size_t WS_X = 1 << 20;
static_assert(WS_FLAG + NSLOT * 1024 <= WS_X, "flags");
constexpr size_t XPAR = (size_t)NSLOT * RBB;
constexpr size_t WS_Y = WS_X + 2 * XPAR;
constexpr int YT = 8;
constexpr int YPART = 704, YTILE = 4 * YPART;
constexpr size_t YPAR = (size_t)4 * 32 * YT * YTILE;
constexpr size_t WS_NEED = WS_Y + 2 * YPAR;
constexpr unsigned long long DEADLINE_TICKS = 100000000ull;

struct Link {
    char* ws;
    unsigned first;
    unsigned fin;
    unsigned fout;
    unsigned firstx;
    unsigned pad[2];
};
struct Params {
    const char* x; const char* w; char* myws;
    int rank; int pad;
    Link link[NR - 1];
    const float* sx; const float* sw;
    float* out; float* pout[4];
    unsigned extra[2];
};

__device__ __forceinline__ int list_slot(int r, int pos) {
    if (pos < NRB) return r * NRB + pos;
    const int q = pos - NRB, t = q / 3, s = q - 3 * t;
    const int rb = t < D1 ? t : (t < D1 + NF ? ND + t - D1 : t - NF);
    return ((r & 4) | ((r + 1 + s) & 3)) * NRB + rb;
}
__device__ __forceinline__ bool held_by(int slot, int k) {
    const int rb = slot & (NRB - 1);
    return rb >= ND && (((slot >> 5) + offs(rb - ND)) & 3) == k;
}

#define MF(d, b, a, c) "v_mfma_i32_16x16x64_i8 %" #d ", %" #b ", %" #a ", " c "\n\t"
#define MM(b0, b1, b2, b3, a) MF(0, b0, a, "%0") MF(1, b1, a, "%1") MF(2, b2, a, "%2") MF(3, b3, a, "%3")
#define MM0 MF(0, 9, 4, "0") MF(1, 10, 4, "0") MF(2, 11, 4, "0") MF(3, 12, 4, "0")
#define LD(a, o) "buffer_load_dwordx4 %" #a ", %25, %26, %27 offen offset:" #o " sc0 sc1\n\t"
#define W(n) "s_waitcnt vmcnt(" #n ")\n\t"
#define BF4(s) "v"(bf[s][0]), "v"(bf[s][1]), "v"(bf[s][2]), "v"(bf[s][3])
#define WAITV(n) asm volatile("s_waitcnt vmcnt(" #n ")" ::: "memory")

__global__ void __launch_bounds__(512, 2) ag_gemm(Params p) {
    extern __shared__ __attribute__((aligned(16))) char smem[];
    const int tid = threadIdx.x, lane = tid & 63;
    const int wave = __builtin_amdgcn_readfirstlane(tid >> 6);
    const int bid = blockIdx.x;
    const unsigned long long t0 = __builtin_amdgcn_s_memrealtime();
    unsigned ts = (unsigned)t0 + LEAD;
    const Link lk = p.link[bid < GEMM0 ? bid >> 3 : 3 + (((bid - GEMM0) >> 4) & 3)];
    const char* const px = p.x; const char* const pw = p.w;
    const int rank = p.rank, me = rank & 3, other = (rank & 4) ^ 4;
    char* myws = p.myws;
    asm volatile("" :: "s"(lk.ws), "s"(lk.first), "s"(lk.firstx), "s"(px), "s"(pw), "s"(myws), "s"(rank));
    GLB_AS unsigned* cnt = (GLB_AS unsigned*)(myws + WS_CNT + (size_t)bid * 128);
    GLB_AS unsigned* tmo = (GLB_AS unsigned*)(myws + WS_TMO);
    const unsigned lbase = (unsigned)(size_t)(LDS_AS char*)smem;
    unsigned call;
    bool bad = false;
    auto put_word = [&](const void* f, unsigned v) {
        asm volatile("s_mov_b64 exec, 1\n\tglobal_store_dword %0, %1, %2 sc0 sc1\n\ts_mov_b64 exec, -1" :: "v"(0u), "v"(v), "s"(f) : "memory");
    };
    auto give_up = [&]() {
        if (__builtin_amdgcn_readfirstlane(__hip_atomic_load(tmo, __ATOMIC_RELAXED, __HIP_MEMORY_SCOPE_AGENT)) == call) bad = true;
        else if (__builtin_amdgcn_s_memrealtime() - t0 > DEADLINE_TICKS) { put_word(myws + WS_TMO, call); bad = true; }
        return bad;
    };
    auto wait_flag = [&](const char* f, unsigned want) -> unsigned {
        unsigned v = 0;
        if (bad) return v;
        do {
            unsigned x, n;
            asm volatile("s_mov_b64 exec, 1\n\ts_mov_b32 %2, 16\n"
                         "1:\n\tglobal_load_dword %1, %3, %4 sc0 sc1\n\ts_waitcnt vmcnt(0)\n\tv_readfirstlane_b32 %0, %1\n\t"
                         "s_cmp_ge_u32 %0, %5\n\ts_cbranch_scc1 2f\n\ts_sleep 1\n\t"
                         "s_sub_u32 %2, %2, 1\n\ts_cmp_lg_u32 %2, 0\n\ts_cbranch_scc1 1b\n"
                         "2:\n\ts_mov_b64 exec, -1"
                         : "=&s"(v), "=&v"(x), "=&s"(n) : "v"(0u), "s"(f), "s"(want) : "memory", "scc");
            if (__builtin_expect(v >= want, 1)) return v;
        } while (!give_up());
        return 0;
    };
    auto wait_flags4 = [&](const char* f) {
        do {
#pragma unroll 1
            for (int it = 0; it < 8; ++it) {
                unsigned fv;
                asm volatile("s_mov_b64 exec, 15\n\tglobal_load_dword %0, %1, %2 sc0 sc1\n\ts_mov_b64 exec, -1\n\ts_waitcnt vmcnt(0)"
                             : "=&v"(fv) : "v"(lane * 128), "s"(f) : "memory");
                if ((__ballot(fv < call) & 15) == 0) return;
                __builtin_amdgcn_s_sleep(4);
            }
        } while (!give_up());
    };
    auto pace = [&](int due) { while ((int)((unsigned)__builtin_amdgcn_s_memrealtime() - ts) < due) __builtin_amdgcn_s_sleep(1); };
    auto snooze = [&](int due) { while ((int)((unsigned)__builtin_amdgcn_s_memrealtime() - ts) < due) __builtin_amdgcn_s_sleep(8); };
    auto lds_add = [&](unsigned addr, unsigned v) -> unsigned {
        unsigned old;
        asm volatile("s_mov_b64 exec, 1\n\tds_add_rtn_u32 %0, %1, %2\n\ts_mov_b64 exec, -1\n\ts_waitcnt lgkmcnt(0)" : "=&v"(old) : "v"(addr), "v"(v) : "memory");
        return __builtin_amdgcn_readfirstlane(old);
    };

    const int g = lane >> 4;
    if (__builtin_expect(bid < GEMM0, 1)) {
        const int j = bid & 7, pi = bid >> 3;
        const bool cross = pi >= 3;
        const int k = (pi - 3) & 3;
        const int wq = wave & 3, frag = j * 8 + wq * 2;
        const bool ext = j >= HJ;
        const int nfull = cross ? 8 + NF : ND - 1, nround = nfull + (cross == ext);
        const int nmix = cross ? nfull : nfull - 1;
        const int tail = (cross ? (j + pi) % (8 - HJ) : (j + pi) % HJ) * 4 + wq;
        const unsigned stg = lbase + 1024 + wave * 4096;
        const unsigned xo = (lane >> 3) * KDIM + (((lane & 7) ^ g) << 4), wo = (lane >> 2) * NL + (((lane & 3) ^ (g & 1)) << 4);
        auto fetch = [&](int slot, int i) {
            const bool isw = slot >= NR * NRB;
            const char* base = isw ? pw + (size_t)(((slot & 3) * 8 + j) * 128 + wq * 32) * NL + k * 128 + (slot & 4) * 16
                                   : px + (size_t)(slot & 31) * RBB + frag * 64;
            unsigned keep;
            asm volatile("s_mov_b32 %0, m0\n\ts_mov_b32 m0, %4\n\ts_nop 0\n\t"
                         "global_load_lds_dwordx4 %1, %3\n\ts_add_u32 m0, m0, 1024\n\ts_nop 0\n\t"
                         "global_load_lds_dwordx4 %2, %3\n\ts_mov_b32 m0, %0"
                         : "=&s"(keep) : "v"(isw ? wo : xo), "v"(isw ? (wo ^ 32) + 16 * NL : (xo ^ 64) + 8 * KDIM), "s"(base),
                           "s"(stg + ((i >> 1) & 1) * 2048) : "memory", "scc");
        };
        fetch(((ext ? lk.firstx : lk.first) >> (16 * (wave >> 2))) & 0xffff, 0);
        unsigned c0;
        asm volatile("global_load_dword %0, %1, %2 sc1\n\tds_write_b32 %3, %1" : "=&v"(c0) : "v"(0u), "s"(cnt), "v"(lbase + 4 * lane) : "memory");
        int vslot;
        {
            const unsigned fin = lk.fin | (ext && k == me ? 0x80u : 0u), fout = lk.fout | (ext && k != me ? 0x10000000u << me : 0u);
            const int nin = __builtin_popcount(fin);
            unsigned m = lane < nin ? fin : fout;
            const int c = lane < nin ? lane : lane - nin - 8;
#pragma unroll
            for (int t = 0; t < NF; ++t) m &= m - (c > t);
            const int b = __builtin_ctz(m | 0x80000000u);
            const int f = lane < nin ? b : b >> 2, rb = f < NF ? ND + f : ND - 1;
            vslot = !cross ? rank * NRB + (lane & 31) : lane < nin ? rank * NRB + rb
                  : lane < nin + 8 ? NR * NRB + me * 8 + lane - nin : (other | (b & 3)) * NRB + rb;
        }
        unsigned fseen = 0;
        asm volatile("s_waitcnt lgkmcnt(0)\n\ts_barrier" ::: "memory");
        int early = 0, drift = 0, left = 0;
        unsigned par = 0;
        call = 0;
        char* dst = lk.ws + WS_X + frag * 1024;
        char* flag = lk.ws + WS_FLAG + j * 128;
        const unsigned l16 = lane * 16;
#pragma unroll 1
        for (int i = wave >> 2; i < nround; i += 2) {
            asm volatile("s_cmp_gt_u32 %1, 1\n\ts_cbranch_scc1 1f\n\ts_waitcnt vmcnt(0)\n1:" : "+v"(c0) : "s"(i) : "memory", "scc");
            if (i < 2) {
                call = __builtin_amdgcn_readfirstlane(c0) + 1;
                par = (call & 1) * (unsigned)XPAR;
                ts = (unsigned)__builtin_amdgcn_s_memrealtime() + SLEAD;
            }
            const int slot = __builtin_amdgcn_readlane(vslot, i);
            const bool own = (slot >> 5) == rank;
            const unsigned r0 = stg + ((i >> 1) & 1) * 2048 + (own ? (lane & 15) * 128 + ((g ^ ((lane & 15) >> 1)) << 4) : l16);
            u32x4 v0, v1;
            if (slot < NR * NRB) {
                asm volatile("ds_read_b128 %0, %2\n\tds_read_b128 %1, %3\n\ts_waitcnt lgkmcnt(0)"
                             : "=&v"(v0), "=&v"(v1) : "v"(r0), "v"(own ? r0 ^ 64 : r0 + 1024) : "memory");
            } else {
                const int q = (lane & 15) >> 1;
                const unsigned rt = stg + ((i >> 1) & 1) * 2048 + (16 * (g & 1) + q) * 64 + 8 * (lane & 1) + (((g >> 1) ^ (q >> 2) ^ ((g & 1) << 1)) << 4);
                i32x2 a, b, c, d;
                asm volatile("ds_read_b64_tr_b8 %0, %4\n\tds_read_b64_tr_b8 %1, %4 offset:512\n\t"
                             "ds_read_b64_tr_b8 %2, %5\n\tds_read_b64_tr_b8 %3, %5 offset:512\n\ts_waitcnt lgkmcnt(0)"
                             : "=&v"(a), "=&v"(b), "=&v"(c), "=&v"(d) : "v"(rt), "v"(rt ^ 32) : "memory");
                v0 = (u32x4){(unsigned)a[0], (unsigned)a[1], (unsigned)b[0], (unsigned)b[1]};
                v1 = (u32x4){(unsigned)c[0], (unsigned)c[1], (unsigned)d[0], (unsigned)d[1]};
            }
            const int due = ((i < nmix ? (i * 4 + wq) * 8 + j : i * 32 + (i < nfull ? ((j + pi) & 7) * 4 + wq : tail)) * ROUND) / 320;
            pace(max(due, early));
            asm volatile("global_store_dwordx4 %0, %2, %1 sc0 sc1\n\tglobal_store_dwordx4 %0, %3, %1 offset:1024 sc0 sc1\n\ts_nop 1"
                         :: "v"(l16), "s"(dst + (par + (unsigned)slot * RBB)), "v"(v0), "v"(v1) : "memory");
            if (i < 2) fetch(__builtin_amdgcn_readlane(vslot, i + 2), 2);
            if ((i < 2 || (i >> 1) == HINTR) && lane == 0)
                __hip_atomic_store((GLB_AS unsigned long long*)(myws + WS_HINT) + ((i < 2 ? 7 : 0) + pi) * 64 + j * 8 + wave,
                                   ((unsigned long long)call << 32) | (ts + drift * left + max(early - due, 0)), __ATOMIC_RELAXED, __HIP_MEMORY_SCOPE_AGENT);
            unsigned long long hv = 0;
            if ((i >> 1) == 2) hv = __hip_atomic_load((GLB_AS unsigned long long*)(myws + WS_HINT) + (7 + pi) * 64 + lane, __ATOMIC_RELAXED, __HIP_MEMORY_SCOPE_AGENT);
            if (cross && (i >> 1) == 2 && lane < 16) fseen = __hip_atomic_load((GLB_AS unsigned*)(myws + WS_FLAG + (vslot & 255) * 1024 + j * 128), __ATOMIC_RELAXED, __HIP_MEMORY_SCOPE_SYSTEM);
            asm volatile("s_waitcnt vmcnt(0)" : "+v"(fseen), "+v"(hv) :: "memory");
            early = i + 2 < nmix ? (int)((unsigned)__builtin_amdgcn_s_memrealtime() - ts) + GAP : 0;
            if (!bad && lds_add(lbase + 4 * i, 1) == 3) put_word(flag + slot * 1024, call);
            if ((i >> 1) == 2 && __all((unsigned)(hv >> 32) == call)) {
                int d = (int)((unsigned)hv - ts);
#pragma unroll
                for (int o = 32; o; o >>= 1) d += __shfl_xor(d, o);
                drift = __builtin_amdgcn_readfirstlane(min(max(d, -6400), 6400)) / 256;
                left = 4;
            }
            if (left > 0) { ts += drift; early -= drift; --left; }
            if (i + 4 < nround) {
                const int ns = __builtin_amdgcn_readlane(vslot, i + 4);
                if ((ns >> 7) != (other >> 2)) fetch(ns, i);
                else {
                    if (__builtin_amdgcn_readlane(fseen, i + 4) < call) wait_flag(myws + WS_FLAG + ns * 1024 + j * 128, call);
                    unsigned keep;
                    asm volatile("s_mov_b32 %0, m0\n\ts_mov_b32 m0, %3\n\ts_nop 0\n\tglobal_load_lds_dwordx4 %1, %2 sc0 sc1\n\t"
                                 "global_load_lds_dwordx4 %1, %2 offset:1024 sc0 sc1\n\ts_mov_b32 m0, %0"
                                 : "=&s"(keep) : "v"(l16), "s"(myws + WS_X + frag * 1024 + (par + (unsigned)ns * RBB)),
                                   "s"(stg + ((i >> 1) & 1) * 2048) : "memory");
                }
            }
        }
        if (lds_add(lbase + 4 * 63, 1) == WPB - 1) put_word((const void*)cnt, call);
        if (!cross) return;

        const float scale = p.sx[0] * p.sw[0];
        const int prod = j * 4 + wq, h = prod >> 4, rep = prod & 15, e0 = (wave >> 2) * 2;
        const char* yf = myws + WS_YFLAG + (k * 32 + prod) * 128;
        const char* yb = myws + WS_Y + (call & 1) * YPAR + (size_t)(k * 32 + prod) * YT * YTILE + e0 * YPART;
        const int yslot = list_slot(other | k, rep + 16 * (lane & 7));
        unsigned todo = (unsigned)__ballot(lane < 8 && !held_by(yslot, me));
#pragma unroll 1
        for (; todo; todo &= todo - 1) {
            const int n = __builtin_ctz(todo), slot = __builtin_amdgcn_readlane(yslot, n);
            const unsigned raw = (wait_flag(yf + n * 4, call * 16) >> e0) & 3;
            if (raw == 3) continue;
            unsigned d[8], tv;
            asm volatile("global_load_dword %0, %9, %12 sc0 sc1\n\tglobal_load_dword %1, %9, %12 offset:256 sc0 sc1\n\t"
                         "global_load_ushort %2, %10, %12 offset:512 sc0 sc1\n\tglobal_load_ubyte %3, %11, %12 offset:640 sc0 sc1\n\t"
                         "global_load_dword %4, %9, %12 offset:704 sc0 sc1\n\tglobal_load_dword %5, %9, %12 offset:960 sc0 sc1\n\t"
                         "global_load_ushort %6, %10, %12 offset:1216 sc0 sc1\n\tglobal_load_ubyte %7, %11, %12 offset:1344 sc0 sc1\n\t"
                         "global_load_dword %8, %13, %14 sc1\n\ts_waitcnt vmcnt(0)"
                         : "=&v"(d[0]), "=&v"(d[1]), "=&v"(d[2]), "=&v"(d[3]), "=&v"(d[4]), "=&v"(d[5]), "=&v"(d[6]), "=&v"(d[7]), "=&v"(tv)
                         : "v"(lane * 4), "v"(lane * 2), "v"(lane), "s"(yb + n * YTILE), "v"(0u), "s"(myws + WS_TMO) : "memory");
            const float sc = bad || tv == call ? __builtin_nanf("") : scale;
            float* op = p.out + (size_t)(slot * 16 + e0 * 4 + g) * NL + k * 128 + h * 64 + (lane & 15) * 4;
#pragma unroll
            for (int e = 0; e < 2; ++e) {
                if ((raw >> e) & 1) continue;
                const unsigned d0 = d[4 * e], d1 = d[4 * e + 1], hi = d[4 * e + 2] | d[4 * e + 3] << 16;
                f32x4 o;
                o[0] = (float)(((int)(hi << 26) >> 10) | (int)(d0 & 0xffffu)) * sc;
                o[1] = (float)((((int)(hi << 20) >> 26) << 16) | (int)(d0 >> 16)) * sc;
                o[2] = (float)((((int)(hi << 14) >> 26) << 16) | (int)(d1 & 0xffffu)) * sc;
                o[3] = (float)((((int)(hi << 8) >> 26) << 16) | (int)(d1 >> 16)) * sc;
                asm volatile("global_store_dwordx4 %0, %1, off sc0 sc1\n\ts_nop 1" :: "v"(op + e * 4 * NL), "v"(o) : "memory");
            }
        }
        return;
    }

    const int gb = bid - GEMM0;
    const int xcd = gb & 7, sl = (gb >> 3) & 7, set = gb >> 6;
    const bool foreign = set > 0;
    const int rep = foreign ? xcd + 8 * (set - 1) : xcd;
    const unsigned c0 = __hip_atomic_load(cnt, __ATOMIC_RELAXED, __HIP_MEMORY_SCOPE_AGENT);
    const float scale = p.sx[0] * p.sw[0];

    constexpr int PTILE = 16 * 272, PBUF = 8 * PTILE, PACK0 = 4 * PBUF, ARR0 = PACK0 + 8 * 768, BAD0 = ARR0 + 32;
    static_assert(ARR0 + 256 + 64 <= LDS_BYTES, "LDS");
    const unsigned l16 = lane * 16;
    const unsigned ast = lbase + wave * 4096;

    if (foreign) {
        const int c = gb - 64, sn = c >> 2, frag = (c & 3) * 16 + wave * 2;
        const char* xs = px + (size_t)sn * RBB + frag * 64;
        const char* wsrc = pw + (size_t)((sn & 3) * 1024 + frag * 16) * NL + (sn >> 2) * 64;
        const unsigned xo = (lane >> 3) * KDIM + (((lane & 7) ^ g) << 4), wo = (lane >> 2) * NL + (((lane & 3) ^ (g & 1)) << 4);
        const int q = (lane & 15) >> 1;
        const unsigned rx = ast + (lane & 15) * 128 + ((g ^ q) << 4);
        const unsigned rt = ast + 2048 + (16 * (g & 1) + q) * 64 + 8 * (lane & 1) + (((g >> 1) ^ (q >> 2) ^ ((g & 1) << 1)) << 4);
        u32x4 v0, v1; i32x2 t0, t1, t2, t3;
        unsigned keep;
        asm volatile("s_mov_b32 %6, m0\n\ts_mov_b32 m0, %13\n\ts_nop 4\n\t"
                     "global_load_lds_dwordx4 %7, %11\n\ts_add_u32 m0, m0, 1024\n\ts_nop 0\n\t"
                     "global_load_lds_dwordx4 %8, %11\n\ts_add_u32 m0, m0, 1024\n\ts_nop 0\n\t"
                     "global_load_lds_dwordx4 %9, %12\n\ts_add_u32 m0, m0, 1024\n\ts_nop 0\n\t"
                     "global_load_lds_dwordx4 %10, %12\n\ts_mov_b32 m0, %6\n\ts_waitcnt vmcnt(0)\n\t"
                     "ds_read_b128 %0, %14\n\tds_read_b128 %1, %15\n\t"
                     "ds_read_b64_tr_b8 %2, %16\n\tds_read_b64_tr_b8 %3, %16 offset:512\n\t"
                     "ds_read_b64_tr_b8 %4, %17\n\tds_read_b64_tr_b8 %5, %17 offset:512\n\ts_waitcnt lgkmcnt(0)"
                     : "=&v"(v0), "=&v"(v1), "=&v"(t0), "=&v"(t1), "=&v"(t2), "=&v"(t3), "=&s"(keep)
                     : "v"(xo), "v"((xo ^ 64) + 8 * KDIM), "v"(wo), "v"((wo ^ 32) + 16 * NL), "s"(xs), "s"(wsrc), "s"(ast),
                       "v"(rx), "v"(rx ^ 64), "v"(rt), "v"(rt ^ 32) : "memory", "scc");
        call = __builtin_amdgcn_readfirstlane(c0) + 1;
        const char* xw = myws + WS_X + (call & 1) * XPAR + frag * 1024;
        const i32x4 v2 = {t0[0], t0[1], t1[0], t1[1]}, v3 = {t2[0], t2[1], t3[0], t3[1]};
        asm volatile("global_store_dwordx4 %0, %5, %2 sc0 sc1\n\tglobal_store_dwordx4 %0, %6, %2 offset:1024 sc0 sc1\n\t"
                     "global_store_dwordx4 %0, %3, %1 sc0 sc1\n\tglobal_store_dwordx4 %0, %4, %1 offset:1024 sc0 sc1\n\t"
                     "s_waitcnt vmcnt(0)\n\ts_barrier"
                     :: "v"(l16), "s"(xw + (size_t)(rank * NRB + sn) * RBB), "s"(xw + (size_t)(NR * NRB + 32 + sn) * RBB),
                        "v"(v0), "v"(v1), "v"(v2), "v"(v3) : "memory");
        if (tid < 4) __hip_atomic_store((GLB_AS unsigned*)(myws + WS_FLAG + ((tid & 2 ? NR * NRB + 32 : rank * NRB) + sn) * 1024 + ((c & 3) * 2 + (tid & 1)) * 128),
                                        call, __ATOMIC_RELAXED, __HIP_MEMORY_SCOPE_SYSTEM);
    }
    call = __builtin_amdgcn_readfirstlane(c0) + 1;
    const char* xws = myws + WS_X + (call & 1) * XPAR;
    if (tid < 16) *(unsigned*)(smem + ARR0 + tid * 4) = 0;
    asm volatile("s_waitcnt lgkmcnt(0)\n\ts_barrier" ::: "memory");

    unsigned vinfo, vflag;
    int nunit;
    {
        const int pos = min(foreign ? rep + 16 * lane : rep + 8 * (lane - (lane > 4)), 127);
        const int slot = !foreign && lane == 4 ? (int)((p.extra[xcd >> 2] >> (8 * (xcd & 3))) & 255) : list_slot(rank, pos);
        const bool use = foreign ? lane < 8 && !held_by(slot, sl >> 1) : lane < 17 && (lane != 4 || xcd < NF);
        const unsigned mask = (unsigned)__ballot(use);
        const int pr = pos < YROT ? (pos & ~7) | ((pos + (sl & 6)) & 7) : pos;
        const int c4 = 4 * pr - min(max(pr - ND, 0), NF) - min(max(pr - NRB - 3 * D1, 0), 3 * NF) + max(pr - (128 - NRAW), 0) * RAWW / 4;
        const int due = ((8 * (9 + NF) - HJ) * ROUND) / 80 + ((c4 + 2 * (sl & 1)) * YSLOT) / 200;
        nunit = __builtin_popcount(mask);
        vinfo = __builtin_amdgcn_ds_permute((use ? __builtin_amdgcn_mbcnt_lo(mask, 0) : 63) * 4, slot | (lane & 7) << 8 | (pos >= 128 - NRAW) << 11 | due << 12);
        vflag = lane < nunit ? (unsigned)WS_FLAG + wave * 128 + (vinfo & 255) * 1024 : (unsigned)WS_ZERO;
    }
    const unsigned vaoff = (vinfo & 255) << 16;
    char* ydst = lk.ws + WS_Y + (call & 1) * YPAR + (size_t)(me * 32 + (sl & 1) * 16 + rep) * YT * YTILE + (wave & 3) * YPART;
    char* yflag = lk.ws + WS_YFLAG + (me * 32 + (sl & 1) * 16 + rep) * 128;
    float* obase = (foreign ? p.pout[sl >> 1] + me * 128 + (sl & 1) * 64 : p.out + sl * 64) + (wave & 3) * 4 * NL;

    int hold = -1, hdue = 0;
    unsigned holdv = 0;
    u32x4 o = {0, 0, 0, 0};
    char* op = ydst;
    unsigned ooff = l16;
    int u = 0;
    int pend = -1;
    auto emit = [&]() {
        asm volatile("s_mov_b32 exec_hi, %3\n\tglobal_store_dwordx4 %0, %1, %2 sc0 sc1\n\ts_mov_b32 exec_hi, -1\n\ts_nop 0"
                     :: "v"(ooff), "v"(o), "s"(op), "s"(holdv > 1 ? ~0u : (1u << (YPART / 16 - 32)) - 1) : "memory");
        pend = hold; hold = -1;
    };
    auto announce = [&]() {
        const unsigned now = lds_add(lbase + ARR0 + pend * 4, holdv) + holdv;
        if ((now & 15) == 4) put_word(yflag + pend * 4, call * 16 + (now >> 4));
        pend = -1;
    };
    auto publish = [&]() { WAITV(0); announce(); };

    i32x4 a[8], bf[8][4];
    wait_flag(myws + __builtin_amdgcn_readlane(vflag, 0), call);
    {
        const __amdgpu_buffer_rsrc_t rs = __builtin_amdgcn_make_buffer_rsrc((void*)(xws + __builtin_amdgcn_readlane(vaoff, 0) + wave * 8192), 0, 8192, 0x00020000);
#pragma unroll
        for (int s = 0; s < 8; ++s) a[s] = __builtin_bit_cast(i32x4, __builtin_amdgcn_raw_buffer_load_b128(rs, l16, s * 1024, 17));
    }
    {
        const int wslot = NR * NRB + (foreign ? 0 : 32) + sl * 4 + (wave >> 1);
        if (foreign) snooze(((2 + (sl & 1) * 4 + (wave >> 1)) * ROUND) / 10);
        wait_flags4(myws + WS_FLAG + wslot * 1024 + (wave & 1) * 512);
        asm volatile("" ::: "memory");
        const __amdgpu_buffer_rsrc_t rs = __builtin_amdgcn_make_buffer_rsrc((void*)(xws + (size_t)wslot * RBB + (wave & 1) * 32768), 0, 32768, 0x00020000);
#pragma unroll
        for (int s = 0; s < 8; ++s)
#pragma unroll
            for (int t = 0; t < 4; ++t)
                bf[s][t] = __builtin_bit_cast(i32x4, __builtin_amdgcn_raw_buffer_load_b128(rs, (lane >> 5) * 2048 + (lane & 31) * 16 + t * 512, s * 4096, 17));
#pragma unroll
        for (int s = 0; s < 8; ++s) asm volatile("" : "+v"(bf[s][0]), "+v"(bf[s][1]), "+v"(bf[s][2]), "+v"(bf[s][3]), "+v"(a[s]));
    }
    if (foreign) {
        snooze(((2 * HINTR + 3) * ROUND) / 10);
        const unsigned long long h = __hip_atomic_load((GLB_AS unsigned long long*)(myws + WS_HINT) + (3 + (sl >> 1)) * 64 + lane, __ATOMIC_RELAXED, __HIP_MEMORY_SCOPE_AGENT);
        int d = (unsigned)(h >> 32) == call ? (int)((unsigned)h - (unsigned)t0) : LEAD;
#pragma unroll
        for (int o = 32; o; o >>= 1) d += __shfl_xor(d, o);
        d >>= 6;
        ts = (unsigned)t0 + __builtin_amdgcn_readfirstlane(min(max(d, 0), 1000)) + YLEAD;
    }
    unsigned fv = 0, fa = (unsigned)WS_TMO;
    u32x4 ars;
    ars[0] = (unsigned)(size_t)xws; ars[1] = (unsigned)((size_t)xws >> 32) & 0xffff; ars[2] = (unsigned)XPAR; ars[3] = 0x00020000;
    const unsigned alo = wave * 8192 + l16, ahi = alo + 4096;
    const unsigned pw0 = lbase + wave * PTILE + (lane & 15) * 272 + g * 16;
    const unsigned pr0 = lbase + ((wave & 3) * 4 + g) * 272 + (lane & 15) * 16;
    auto poison_all = [&]() { asm volatile("ds_write_b32 %0, %1" :: "v"(lbase + BAD0), "v"(1) : "memory"); };
    constexpr int RUN = 3, LAST = 9, FETCH = 5;
    int md = FETCH;
    if (bad) poison_all();
    auto peek = [&](const char* f) -> unsigned {
        unsigned v;
        asm volatile("s_mov_b64 exec, 1\n\tglobal_load_dword %0, %1, %2 sc0 sc1\n\ts_mov_b64 exec, -1\n\ts_waitcnt vmcnt(0)"
                     : "=&v"(v) : "v"(0u), "s"(f) : "memory");
        return __builtin_amdgcn_readfirstlane(v);
    };
    bool paired = false;
#pragma unroll 1
    while (u < nunit) {
        asm volatile("s_waitcnt vmcnt(7)" : "+v"(a[0]), "+v"(fv) :: "memory");
        const int un = u + (md != LAST);
        const unsigned soff = __builtin_amdgcn_readlane(vaoff, un), fnext = __builtin_amdgcn_readlane(vflag, un + 1);
        if (__builtin_expect(md != LAST, 1)) {
            unsigned nx = __builtin_amdgcn_readlane(fv, 0);
            if (nx < call && u + 1 < nunit) nx = peek(myws + __builtin_amdgcn_readlane(vflag, u + 1));
            md = nx >= call ? RUN : LAST;
            if (__builtin_expect(__builtin_amdgcn_readlane(fv, 1) == call, 0)) poison_all();
        } else {
            const char* f = myws + __builtin_amdgcn_readlane(vflag, u);
            bool seen = false;
            while ((hold & pend) >= 0) {
                seen = peek(f) >= call;
                if (seen) break;
                if (pend >= 0) announce();
                else if ((int)((unsigned)__builtin_amdgcn_s_memrealtime() - ts) >= hdue) emit();
            }
            if (!seen && wait_flag(f, call) == 0) poison_all();
            md = FETCH;
        }
        asm volatile("s_nop 1\n\tv_writelane_b32 %1, %3, 0\n\tglobal_load_dword %0, %1, %2 sc0 sc1"
                     : "+v"(fv), "+v"(fa) : "s"(myws), "s"(fnext), "s"(soff) : "memory");
        i32x4 acc[4];
        asm volatile("s_cmp_lg_u32 %28, 3\n\ts_cbranch_scc1 1f\n\t"
                     MM0 LD(4, 0) W(8) MM(13, 14, 15, 16, 5) LD(5, 1024) W(8) MM(17, 18, 19, 20, 6) LD(6, 2048) W(8) MM(21, 22, 23, 24, 7) LD(7, 3072) W(8)
                     "s_branch 3f\n1:\n\ts_cmp_lg_u32 %28, 9\n\ts_cbranch_scc1 2f\n\t"
                     MM0 W(7) MM(13, 14, 15, 16, 5) W(6) MM(17, 18, 19, 20, 6) W(5) MM(21, 22, 23, 24, 7) W(4)
                     "s_branch 3f\n2:\n\t" LD(4, 0) LD(5, 1024) LD(6, 2048) LD(7, 3072) "\n3:"
                     : "=&v"(acc[0]), "=&v"(acc[1]), "=&v"(acc[2]), "=&v"(acc[3]), "+v"(a[0]), "+v"(a[1]), "+v"(a[2]), "+v"(a[3]), "+v"(a[4])
                     : BF4(0), BF4(1), BF4(2), BF4(3), "v"(alo), "s"(ars), "s"(soff), "s"(md) : "memory", "scc");
        bool mid = false;
        if (hold >= 0 && (int)((unsigned)__builtin_amdgcn_s_memrealtime() - ts) + MIDLEAD >= hdue) { emit(); mid = true; }
        asm volatile("s_cmp_lg_u32 %28, 3\n\ts_cbranch_scc1 1f\n\t"
                     MM(9, 10, 11, 12, 4) LD(4, 0) W(8) MM(13, 14, 15, 16, 5) LD(5, 1024) W(8) MM(17, 18, 19, 20, 6) LD(6, 2048) W(8) MM(21, 22, 23, 24, 7) LD(7, 3072)
                     "s_branch 3f\n1:\n\ts_cmp_lg_u32 %28, 9\n\ts_cbranch_scc1 2f\n\t"
                     MM(9, 10, 11, 12, 4) W(3) MM(13, 14, 15, 16, 5) W(2) MM(17, 18, 19, 20, 6) W(1) MM(21, 22, 23, 24, 7)
                     "s_branch 3f\n2:\n\t" LD(4, 0) LD(5, 1024) LD(6, 2048) LD(7, 3072) "\n3:\n\ts_nop 7"
                     : "+v"(acc[0]), "+v"(acc[1]), "+v"(acc[2]), "+v"(acc[3]), "+v"(a[4]), "+v"(a[5]), "+v"(a[6]), "+v"(a[7]), "+v"(a[0])
                     : BF4(4), BF4(5), BF4(6), BF4(7), "v"(ahi), "s"(ars), "s"(soff), "s"(md) : "memory", "scc");
        if (md == FETCH) { if (pend >= 0 && !mid) announce(); continue; }
        const unsigned pb = (u & 3) * PBUF;
        asm volatile("ds_write_b128 %0, %1\n\tds_write_b128 %0, %2 offset:64\n\tds_write_b128 %0, %3 offset:128\n\tds_write_b128 %0, %4 offset:192"
                     :: "v"(pw0 + pb), "v"(acc[0]), "v"(acc[1]), "v"(acc[2]), "v"(acc[3]) : "memory");
        ++u;
        if (!foreign && (u & 1) && u < nunit - NSGL) { paired = true; continue; }
        const bool mine = (wave >> 2) != (u & 1);
        const bool after = pend >= 0 && !mine && !mid;
        if (pend >= 0 && mine) publish();
        if (hold >= 0) { pace(hdue); emit(); }
        asm volatile("s_waitcnt lgkmcnt(0)\n\ts_barrier" ::: "memory");
        if (after) publish();
        if (mine || paired) {
            const unsigned info = __builtin_amdgcn_readlane(vinfo, mine ? u - 1 : u - 2);
            i32x4 r[8];
            int poison;
            unsigned long long now;
            asm volatile("s_memrealtime %9\n\tds_read_b128 %0, %10\n\tds_read_b128 %1, %10 offset:%12\n\tds_read_b128 %2, %10 offset:2*%12\n\t"
                         "ds_read_b128 %3, %10 offset:3*%12\n\tds_read_b128 %4, %10 offset:4*%12\n\tds_read_b128 %5, %10 offset:5*%12\n\t"
                         "ds_read_b128 %6, %10 offset:6*%12\n\tds_read_b128 %7, %10 offset:7*%12\n\tds_read_b32 %8, %11\n\ts_waitcnt lgkmcnt(0)"
                         : "=&v"(r[0]), "=&v"(r[1]), "=&v"(r[2]), "=&v"(r[3]), "=&v"(r[4]), "=&v"(r[5]), "=&v"(r[6]), "=&v"(r[7]), "=&v"(poison), "=&s"(now)
                         : "v"(pr0 + (mine ? pb : pb - PBUF)), "v"(lbase + BAD0), "n"(PTILE) : "memory");
            const i32x4 sum = ((r[0] + r[1]) + (r[2] + r[3])) + ((r[4] + r[5]) + (r[6] + r[7]));
            const float sc = poison ? __builtin_nanf("") : scale;
            char* const orow = (char*)obase + (size_t)(info & 255) * (16 * NL * 4);
            const unsigned roff = g * (NL * 4) + (lane & 15) * 16;
#pragma unroll
            for (int e = 0; e < 4; ++e) o[e] = __builtin_bit_cast(unsigned, (float)sum[e] * sc);
            if (!foreign) {
                asm volatile("global_store_dwordx4 %0, %1, %2 sc0 sc1\n\ts_nop 1" :: "v"(roff), "v"(o), "s"(orow) : "memory");
            } else {
                hold = (info >> 8) & 7;
                hdue = info >> 12;
                op = orow; ooff = roff;
                holdv = 1 + (16u << (wave & 3));
                asm volatile("" : "+v"(o));
                const unsigned big = ((unsigned)sum[0] + 0x200000u) | ((unsigned)sum[1] + 0x200000u) | ((unsigned)sum[2] + 0x200000u) | ((unsigned)sum[3] + 0x200000u);
                if (!(info & 0x800) && !__any((big >> 22) | poison)) {
                    const unsigned s0 = sum[0], s1 = sum[1], s2 = sum[2], s3 = sum[3];
                    const unsigned hi = ((s0 >> 16) & 63) | ((s1 >> 10) & (63 << 6)) | ((s2 >> 4) & (63 << 12)) | ((s3 << 2) & (63 << 18));
                    const unsigned pk = lbase + PACK0 + wave * 768;
                    asm volatile("ds_write_b32 %1, %2\n\tds_write_b32 %1, %3 offset:256\n\tds_write_b16 %5, %4 offset:512\n\t"
                                 "ds_write_b8_d16_hi %6, %4 offset:640\n\tds_read_b128 %0, %7\n\ts_waitcnt lgkmcnt(0)"
                                 : "=&v"(o) : "v"(pk + lane * 4), "v"((s0 & 0xffffu) | (s1 << 16)), "v"((s2 & 0xffffu) | (s3 << 16)), "v"(hi),
                                   "v"(pk + lane * 2), "v"(pk + lane), "v"(pk + l16) : "memory");
                    op = ydst + hold * YTILE; ooff = l16;
                    holdv = 1;
                }
                const int left = hdue - (int)((unsigned)now - ts) - EPI;
                if (u >= nunit || left <= HOLDMIN) { if (left > 0) pace(hdue); emit(); }
            }
        }
        paired = false;
    }
    asm volatile("s_waitcnt vmcnt(0)" : "+v"(a[0]), "+v"(a[1]), "+v"(a[2]), "+v"(a[3]), "+v"(a[4]), "+v"(a[5]), "+v"(a[6]), "+v"(a[7]), "+v"(fv) :: "memory");
    if (pend >= 0) publish();
    if (tid == 0) __hip_atomic_store(cnt, call, __ATOMIC_RELAXED, __HIP_MEMORY_SCOPE_AGENT);
}

extern "C" void kernel_launch(int rank, int nranks, ncclComm_t comm,
                              hipStream_t stream,
                              void* const* d_in, const int* in_sizes, int n_in,
                              void* d_out, int out_size,
                              const int* in_dtypes, int out_dtype,
                              void* const* d_out_all,
                              void* const* d_ws_all, size_t ws_bytes) {
    if (nranks != NR || (unsigned)rank >= (unsigned)NR || n_in != 4 || ws_bytes < WS_NEED
        || in_sizes[0] != ML * KDIM || in_sizes[1] != KDIM * NL || out_size != NR * ML * NL) {
        fprintf(stderr, "ag_gemm: unexpected problem shape\n");
        return;
    }
    HIP_CHECK(hipFuncSetAttribute((const void*)ag_gemm, hipFuncAttributeMaxDynamicSharedMemorySize, LDS_BYTES));
    Params p;
    p.x = (const char*)d_in[0]; p.w = (const char*)d_in[1];
    p.sx = (const float*)d_in[2]; p.sw = (const float*)d_in[3];
    p.rank = rank; p.pad = 0;
    const int me = rank & 3, other = (rank & 4) ^ 4;
    p.myws = (char*)d_ws_all[rank]; p.out = (float*)d_out;
    for (int i = 0; i < NR - 1; ++i) {
        p.link[i].ws = (char*)d_ws_all[i < 3 ? (rank & 4) | ((me + 1 + i) & 3) : other | (i - 3)];
        p.link[i].fin = p.link[i].fout = p.link[i].pad[0] = p.link[i].pad[1] = 0;
    }
    for (int k = 0; k < 4; ++k) p.pout[k] = (float*)d_out_all[other | k];
    p.extra[0] = p.extra[1] = 0;
    int ne = 0;
    for (int f = 0; f < NF; ++f) {
        p.link[3 + ((me + offs(f)) & 3)].fin |= 1u << f;
        for (int m = 0; m < 4; ++m) {
            if (((m + offs(f)) & 3) != me) continue;
            for (int k = 0; k < 4; ++k) if (k != m) p.link[3 + k].fout |= 1u << (4 * f + m);
            p.extra[ne >> 2] |= (unsigned)((other | m) * NRB + ND + f) << (8 * (ne & 3));
            ++ne;
        }
    }
    for (int i = 0; i < NR - 1; ++i) {
        const unsigned fin = p.link[i].fin, rest = fin & (fin - 1);
        const unsigned s0 = rank * NRB + (i < 3 ? 0 : ND + __builtin_ctz(fin));
        const unsigned s1 = i < 3 ? rank * NRB + 1 : rest ? rank * NRB + ND + __builtin_ctz(rest) : NR * NRB + me * 8;
        p.link[i].first = s0 | s1 << 16;
        p.link[i].firstx = i == 3 + me && !rest ? s0 | (rank * NRB + ND - 1) << 16 : p.link[i].first;
    }
    hipLaunchKernelGGL(ag_gemm, dim3(GRID), dim3(512), LDS_BYTES, stream, p);
    HIP_CHECK(hipGetLastError());
}
```

```cpp
#include <hip/hip_runtime.h>
#include <rccl/rccl.h>
#include <cstdio>
#include <cstdint>

#define HIP_CHECK(expr) do { hipError_t _e = (expr); if (_e != hipSuccess) { \
    fprintf(stderr, "HIP error %s at %s:%d\n", hipGetErrorString(_e), __FILE__, __LINE__); \
    return; } } while (0)

typedef unsigned u32x4 __attribute__((ext_vector_type(4)));
typedef int i32x4 __attribute__((ext_vector_type(4)));
typedef int i32x2 __attribute__((ext_vector_type(2)));
typedef float f32x4 __attribute__((ext_vector_type(4)));
#define LDS_AS __attribute__((address_space(3)))
#define GLB_AS __attribute__((address_space(1)))

constexpr int NF = 7;
constexpr int D1 = 13;
constexpr int ROUND = 1116;
constexpr int YSLOT = 490;
constexpr int LEAD = 125;
constexpr int SLEAD = -20;
constexpr int GAP = 110;
constexpr int HINTR = 4;
constexpr int YLEAD = 8;
constexpr int YROT = 96;
constexpr int HOLDMIN = 25;
constexpr int EPI = 20;
constexpr int XL = 13;
constexpr int MIDLEAD = 10;
constexpr int RAWW = 4;
constexpr int NRAW = 10;
constexpr int NSGL = 3;

constexpr int NR = 8;
constexpr int KDIM = 4096;
constexpr int ML = 512;
constexpr int NL = 512;
constexpr int NRB = ML / 16;
constexpr int ND = NRB - NF;
constexpr int RBB = 16 * KDIM;
constexpr unsigned OFFS = NF == 5 ? 0x20321u : NF == 6 ? 0x310321u : 0x03210321u;
__host__ __device__ constexpr int offs(int f) { return (OFFS >> (4 * f)) & 3; }
static_assert(NF >= 4 && NF <= 7 && D1 < ND - 1, "plan");
constexpr int GEMM0 = (NR - 1) * 8;
constexpr int GRID = GEMM0 + 8 * 24;
constexpr int WPB = 8;
constexpr int LDS_BYTES = 148 << 10;
constexpr int NSLOT = NR * NRB + 64;
constexpr size_t WS_CNT = 0;
constexpr size_t WS_TMO = 48 << 10;
constexpr size_t WS_ZERO = 50 << 10;
constexpr size_t WS_HINT = 52 << 10;
constexpr size_t WS_FLAG = 256 << 10;
constexpr size_t WS_YFLAG = 128 << 10;
constexpr size_t WS_X = 1 << 20;
static_assert(WS_FLAG + NSLOT * 1024 <= WS_X, "flags");
constexpr size_t XPAR = (size_t)NSLOT * RBB;
constexpr size_t WS_Y = WS_X + 2 * XPAR;
constexpr int YT = 8;
constexpr int YPART = 704, YTILE = 4 * YPART;
constexpr size_t YPAR = (size_t)4 * 32 * YT * YTILE;
constexpr size_t WS_NEED = WS_Y + 2 * YPAR;
constexpr unsigned long long DEADLINE_TICKS = 100000000ull;

struct Link {
    char* ws;
    unsigned first;
    unsigned fin;
    unsigned fout;
    unsigned pad[3];
};
struct Params {
    const char* x; const char* w; char* myws;
    int rank; int pad;
    Link link[NR - 1];
    const float* sx; const float* sw;
    float* out; float* pout[4];
    unsigned extra[2];
};

__device__ __forceinline__ int list_slot(int r, int pos) {
    if (pos < NRB) return r * NRB + pos;
    const int q = pos - NRB, t = q / 3, s = q - 3 * t;
    const int rb = t < D1 ? t : (t < D1 + NF ? ND + t - D1 : t - NF);
    return ((r & 4) | ((r + 1 + s) & 3)) * NRB + rb;
}
__device__ __forceinline__ int tile_pos(int rep, int n) { return n < 6 ? rep + 16 * n : n == 6 ? (rep < 13 ? 96 + rep : 127) : 109 + rep; }
__device__ __forceinline__ bool held_by(int slot, int k) {
    const int rb = slot & (NRB - 1);
    return rb == ND - 1 || (rb >= ND && (((slot >> 5) + offs(rb - ND)) & 3) == k);
}

#define MF(d, b, a, c) "v_mfma_i32_16x16x64_i8 %" #d ", %" #b ", %" #a ", " c "\n\t"
#define MM(b0, b1, b2, b3, a) MF(0, b0, a, "%0") MF(1, b1, a, "%1") MF(2, b2, a, "%2") MF(3, b3, a, "%3")
#define MM0 MF(0, 9, 4, "0") MF(1, 10, 4, "0") MF(2, 11, 4, "0") MF(3, 12, 4, "0")
#define LD(a, o) "buffer_load_dwordx4 %" #a ", %25, %26, %27 offen offset:" #o " sc0 sc1\n\t"
#define W(n) "s_waitcnt vmcnt(" #n ")\n\t"
#define BF4(s) "v"(bf[s][0]), "v"(bf[s][1]), "v"(bf[s][2]), "v"(bf[s][3])
#define WAITV(n) asm volatile("s_waitcnt vmcnt(" #n ")" ::: "memory")

__global__ void __launch_bounds__(512, 2) ag_gemm(Params p) {
    extern __shared__ __attribute__((aligned(16))) char smem[];
    const int tid = threadIdx.x, lane = tid & 63;
    const int wave = __builtin_amdgcn_readfirstlane(tid >> 6);
    const int bid = blockIdx.x;
    const unsigned long long t0 = __builtin_amdgcn_s_memrealtime();
    unsigned ts = (unsigned)t0 + LEAD;
    const Link lk = p.link[bid < GEMM0 ? bid >> 3 : 3 + (((bid - GEMM0) >> 4) & 3)];
    const char* const px = p.x; const char* const pw = p.w;
    const int rank = p.rank, me = rank & 3, other = (rank & 4) ^ 4;
    char* myws = p.myws;
    asm volatile("" :: "s"(lk.ws), "s"(lk.first), "s"(lk.fout), "s"(px), "s"(pw), "s"(myws), "s"(rank));
    GLB_AS unsigned* cnt = (GLB_AS unsigned*)(myws + WS_CNT + (size_t)bid * 128);
    GLB_AS unsigned* tmo = (GLB_AS unsigned*)(myws + WS_TMO);
    const unsigned lbase = (unsigned)(size_t)(LDS_AS char*)smem;
    unsigned call;
    bool bad = false;
    auto put_word = [&](const void* f, unsigned v) {
        asm volatile("s_mov_b64 exec, 1\n\tglobal_store_dword %0, %1, %2 sc0 sc1\n\ts_mov_b64 exec, -1" :: "v"(0u), "v"(v), "s"(f) : "memory");
    };
    auto give_up = [&]() {
        if (__builtin_amdgcn_readfirstlane(__hip_atomic_load(tmo, __ATOMIC_RELAXED, __HIP_MEMORY_SCOPE_AGENT)) == call) bad = true;
        else if (__builtin_amdgcn_s_memrealtime() - t0 > DEADLINE_TICKS) { put_word(myws + WS_TMO, call); bad = true; }
        return bad;
    };
    auto wait_flag = [&](const char* f, unsigned want) -> unsigned {
        unsigned v = 0;
        if (bad) return v;
        do {
            unsigned x, n;
            asm volatile("s_mov_b64 exec, 1\n\ts_movk_i32 %2, 0x200\n"
                         "1:\n\tglobal_load_dword %1, %3, %4 sc0 sc1\n\ts_waitcnt vmcnt(0)\n\tv_readfirstlane_b32 %0, %1\n\t"
                         "s_cmp_ge_u32 %0, %5\n\ts_cbranch_scc1 2f\n\ts_sleep 1\n\t"
                         "s_sub_u32 %2, %2, 1\n\ts_cmp_lg_u32 %2, 0\n\ts_cbranch_scc1 1b\n"
                         "2:\n\ts_mov_b64 exec, -1"
                         : "=&s"(v), "=&v"(x), "=&s"(n) : "v"(0u), "s"(f), "s"(want) : "memory", "scc");
            if (__builtin_expect(v >= want, 1)) return v;
        } while (!give_up());
        return 0;
    };
    auto wait_flags4 = [&](const char* f) {
        do {
#pragma unroll 1
            for (int it = 0; it < 8; ++it) {
                unsigned fv;
                asm volatile("s_mov_b64 exec, 15\n\tglobal_load_dword %0, %1, %2 sc0 sc1\n\ts_mov_b64 exec, -1\n\ts_waitcnt vmcnt(0)"
                             : "=&v"(fv) : "v"(lane * 128), "s"(f) : "memory");
                if ((__ballot(fv < call) & 15) == 0) return;
                __builtin_amdgcn_s_sleep(4);
            }
        } while (!give_up());
    };
    auto pace = [&](int due) { while ((int)((unsigned)__builtin_amdgcn_s_memrealtime() - ts) < due) __builtin_amdgcn_s_sleep(1); };
    auto snooze = [&](int due) { while ((int)((unsigned)__builtin_amdgcn_s_memrealtime() - ts) < due) __builtin_amdgcn_s_sleep(8); };
    auto lds_add = [&](unsigned addr, unsigned v) -> unsigned {
        unsigned old;
        asm volatile("s_mov_b64 exec, 1\n\tds_add_rtn_u32 %0, %1, %2\n\ts_mov_b64 exec, -1\n\ts_waitcnt lgkmcnt(0)" : "=&v"(old) : "v"(addr), "v"(v) : "memory");
        return __builtin_amdgcn_readfirstlane(old);
    };

    const int g = lane >> 4;
    if (__builtin_expect(bid < GEMM0, 1)) {
        const int j = bid & 7, pi = bid >> 3;
        const bool cross = pi >= 3;
        const int k = (pi - 3) & 3;
        const int wq = wave & 3, frag = j * 8 + wq * 2;
        const int nround = cross ? 9 + NF : ND, nmix = cross ? nround : nround - 1;
        const unsigned stg = lbase + 1024 + wave * 4096;
        const unsigned xo = (lane >> 3) * KDIM + (((lane & 7) ^ g) << 4), wo = (lane >> 2) * NL + (((lane & 3) ^ (g & 1)) << 4);
        auto fetch = [&](int slot, int i) {
            const bool isw = slot >= NR * NRB;
            const char* base = isw ? pw + (size_t)(((slot & 3) * 8 + j) * 128 + wq * 32) * NL + k * 128 + (slot & 4) * 16
                                   : px + (size_t)(slot & 31) * RBB + frag * 64;
            unsigned keep;
            asm volatile("s_mov_b32 %0, m0\n\ts_mov_b32 m0, %4\n\ts_nop 0\n\t"
                         "global_load_lds_dwordx4 %1, %3\n\ts_add_u32 m0, m0, 1024\n\ts_nop 0\n\t"
                         "global_load_lds_dwordx4 %2, %3\n\ts_mov_b32 m0, %0"
                         : "=&s"(keep) : "v"(isw ? wo : xo), "v"(isw ? (wo ^ 32) + 16 * NL : (xo ^ 64) + 8 * KDIM), "s"(base),
                           "s"(stg + ((i >> 1) & 1) * 2048) : "memory", "scc");
        };
        fetch((lk.first >> (16 * (wave >> 2))) & 0xffff, 0);
        unsigned c0;
        asm volatile("global_load_dword %0, %1, %2 sc1\n\tds_write_b32 %3, %1" : "=&v"(c0) : "v"(0u), "s"(cnt), "v"(lbase + 4 * lane) : "memory");
        int vslot;
        {
            const unsigned fin = lk.fin, fout = lk.fout;
            const int nin = __builtin_popcount(fin);
            unsigned m = lane < nin ? fin : fout;
            const int c = lane < nin ? lane : lane - nin - 8;
#pragma unroll
            for (int t = 0; t < NF - 1; ++t) m &= m - (c > t);
            const int b = __builtin_ctz(m | 0x80000000u);
            const int rb = min(ND + (lane < nin ? b : b >> 2), NRB - 1);
            vslot = !cross ? rank * NRB + (lane & 31) : lane < nin ? rank * NRB + rb
                  : lane < nin + 8 ? NR * NRB + me * 8 + lane - nin : lane == 8 + NF ? rank * NRB + ND - 1 : (other | (b & 3)) * NRB + rb;
        }
        unsigned fseen = 0;
        asm volatile("s_waitcnt lgkmcnt(0)\n\ts_barrier" ::: "memory");
        int early = 0, drift = 0, left = 0;
        unsigned par = 0;
        call = 0;
        char* dst = lk.ws + WS_X + frag * 1024;
        char* flag = lk.ws + WS_FLAG + j * 128;
        const unsigned l16 = lane * 16;
#pragma unroll 1
        for (int i = wave >> 2; i < nround; i += 2) {
            asm volatile("s_cmp_gt_u32 %1, 1\n\ts_cbranch_scc1 1f\n\ts_waitcnt vmcnt(0)\n1:" : "+v"(c0) : "s"(i) : "memory", "scc");
            if (i < 2) {
                call = __builtin_amdgcn_readfirstlane(c0) + 1;
                par = (call & 1) * (unsigned)XPAR;
                ts = (unsigned)__builtin_amdgcn_s_memrealtime() + SLEAD;
            }
            const int slot = __builtin_amdgcn_readlane(vslot, i);
            const bool own = (slot >> 5) == rank;
            const unsigned r0 = stg + ((i >> 1) & 1) * 2048 + (own ? (lane & 15) * 128 + ((g ^ ((lane & 15) >> 1)) << 4) : l16);
            u32x4 v0, v1;
            if (slot < NR * NRB) {
                asm volatile("ds_read_b128 %0, %2\n\tds_read_b128 %1, %3\n\ts_waitcnt lgkmcnt(0)"
                             : "=&v"(v0), "=&v"(v1) : "v"(r0), "v"(own ? r0 ^ 64 : r0 + 1024) : "memory");
            } else {
                const int q = (lane & 15) >> 1;
                const unsigned rt = stg + ((i >> 1) & 1) * 2048 + (16 * (g & 1) + q) * 64 + 8 * (lane & 1) + (((g >> 1) ^ (q >> 2) ^ ((g & 1) << 1)) << 4);
                i32x2 a, b, c, d;
                asm volatile("ds_read_b64_tr_b8 %0, %4\n\tds_read_b64_tr_b8 %1, %4 offset:512\n\t"
                             "ds_read_b64_tr_b8 %2, %5\n\tds_read_b64_tr_b8 %3, %5 offset:512\n\ts_waitcnt lgkmcnt(0)"
                             : "=&v"(a), "=&v"(b), "=&v"(c), "=&v"(d) : "v"(rt), "v"(rt ^ 32) : "memory");
                v0 = (u32x4){(unsigned)a[0], (unsigned)a[1], (unsigned)b[0], (unsigned)b[1]};
                v1 = (u32x4){(unsigned)c[0], (unsigned)c[1], (unsigned)d[0], (unsigned)d[1]};
            }
            const int due = ((i < nmix ? (i * 4 + wq) * 8 + j : (i * 8 + ((j + pi) & 7)) * 4 + wq) * ROUND) / 320;
            pace(max(due, early));
            asm volatile("global_store_dwordx4 %0, %2, %1 sc0 sc1\n\tglobal_store_dwordx4 %0, %3, %1 offset:1024 sc0 sc1\n\ts_nop 1"
                         :: "v"(l16), "s"(dst + (par + (unsigned)slot * RBB)), "v"(v0), "v"(v1) : "memory");
            if (i < 2) fetch(__builtin_amdgcn_readlane(vslot, i + 2), 2);
            if ((i < 2 || (i >> 1) == HINTR) && lane == 0)
                __hip_atomic_store((GLB_AS unsigned long long*)(myws + WS_HINT) + ((i < 2 ? 7 : 0) + pi) * 64 + j * 8 + wave,
                                   ((unsigned long long)call << 32) | (ts + drift * left + max(early - due, 0)), __ATOMIC_RELAXED, __HIP_MEMORY_SCOPE_AGENT);
            unsigned long long hv = 0;
            if ((i >> 1) == 2) hv = __hip_atomic_load((GLB_AS unsigned long long*)(myws + WS_HINT) + (7 + pi) * 64 + lane, __ATOMIC_RELAXED, __HIP_MEMORY_SCOPE_AGENT);
            if (cross && (i >> 1) == 2 && lane < 16) fseen = __hip_atomic_load((GLB_AS unsigned*)(myws + WS_FLAG + (vslot & 255) * 1024 + j * 128), __ATOMIC_RELAXED, __HIP_MEMORY_SCOPE_SYSTEM);
            asm volatile("s_waitcnt vmcnt(0)" : "+v"(fseen), "+v"(hv) :: "memory");
            early = i + 2 < nmix ? (int)((unsigned)__builtin_amdgcn_s_memrealtime() - ts) + GAP : 0;
            if (!bad && lds_add(lbase + 4 * i, 1) == 3) put_word(flag + slot * 1024, call);
            if ((i >> 1) == 2 && __all((unsigned)(hv >> 32) == call)) {
                int d = (int)((unsigned)hv - ts);
#pragma unroll
                for (int o = 32; o; o >>= 1) d += __shfl_xor(d, o);
                drift = __builtin_amdgcn_readfirstlane(min(max(d, -6400), 6400)) / 256;
                left = 4;
            }
            if (left > 0) { ts += drift; early -= drift; --left; }
            if (i + 4 < nround) {
                const int ns = __builtin_amdgcn_readlane(vslot, i + 4);
                if ((ns >> 7) != (other >> 2)) fetch(ns, i);
                else {
                    if (__builtin_amdgcn_readlane(fseen, i + 4) < call) wait_flag(myws + WS_FLAG + ns * 1024 + j * 128, call);
                    unsigned keep;
                    asm volatile("s_mov_b32 %0, m0\n\ts_mov_b32 m0, %3\n\ts_nop 0\n\tglobal_load_lds_dwordx4 %1, %2 sc0 sc1\n\t"
                                 "global_load_lds_dwordx4 %1, %2 offset:1024 sc0 sc1\n\ts_mov_b32 m0, %0"
                                 : "=&s"(keep) : "v"(l16), "s"(myws + WS_X + frag * 1024 + (par + (unsigned)ns * RBB)),
                                   "s"(stg + ((i >> 1) & 1) * 2048) : "memory");
                }
            }
        }
        if (lds_add(lbase + 4 * 63, 1) == WPB - 1) put_word((const void*)cnt, call);
        if (!cross) return;

        const float scale = p.sx[0] * p.sw[0];
        const int prod = j * 4 + wq, h = prod >> 4, rep = prod & 15, e0 = (wave >> 2) * 2;
        const char* yf = myws + WS_YFLAG + (k * 32 + prod) * 128;
        const char* yb = myws + WS_Y + (call & 1) * YPAR + (size_t)(k * 32 + prod) * YT * YTILE + e0 * YPART;
        const int yslot = list_slot(other | k, tile_pos(rep, lane & 7));
        unsigned todo = (unsigned)__ballot(lane < 8 && !held_by(yslot, me));
#pragma unroll 1
        for (; todo; todo &= todo - 1) {
            const int n = __builtin_ctz(todo), slot = __builtin_amdgcn_readlane(yslot, n);
            const unsigned raw = (wait_flag(yf + n * 4, call * 16) >> e0) & 3;
            if (raw == 3) continue;
            unsigned d[8], tv;
            asm volatile("global_load_dword %0, %9, %12 sc0 sc1\n\tglobal_load_dword %1, %9, %12 offset:256 sc0 sc1\n\t"
                         "global_load_ushort %2, %10, %12 offset:512 sc0 sc1\n\tglobal_load_ubyte %3, %11, %12 offset:640 sc0 sc1\n\t"
                         "global_load_dword %4, %9, %12 offset:704 sc0 sc1\n\tglobal_load_dword %5, %9, %12 offset:960 sc0 sc1\n\t"
                         "global_load_ushort %6, %10, %12 offset:1216 sc0 sc1\n\tglobal_load_ubyte %7, %11, %12 offset:1344 sc0 sc1\n\t"
                         "global_load_dword %8, %13, %14 sc1\n\ts_waitcnt vmcnt(0)"
                         : "=&v"(d[0]), "=&v"(d[1]), "=&v"(d[2]), "=&v"(d[3]), "=&v"(d[4]), "=&v"(d[5]), "=&v"(d[6]), "=&v"(d[7]), "=&v"(tv)
                         : "v"(lane * 4), "v"(lane * 2), "v"(lane), "s"(yb + n * YTILE), "v"(0u), "s"(myws + WS_TMO) : "memory");
            const float sc = bad || tv == call ? __builtin_nanf("") : scale;
            float* op = p.out + (size_t)(slot * 16 + e0 * 4 + g) * NL + k * 128 + h * 64 + (lane & 15) * 4;
#pragma unroll
            for (int e = 0; e < 2; ++e) {
                if ((raw >> e) & 1) continue;
                const unsigned d0 = d[4 * e], d1 = d[4 * e + 1], hi = d[4 * e + 2] | d[4 * e + 3] << 16;
                f32x4 o;
                o[0] = (float)(((int)(hi << 26) >> 10) | (int)(d0 & 0xffffu)) * sc;
                o[1] = (float)((((int)(hi << 20) >> 26) << 16) | (int)(d0 >> 16)) * sc;
                o[2] = (float)((((int)(hi << 14) >> 26) << 16) | (int)(d1 & 0xffffu)) * sc;
                o[3] = (float)((((int)(hi << 8) >> 26) << 16) | (int)(d1 >> 16)) * sc;
                asm volatile("global_store_dwordx4 %0, %1, off sc0 sc1\n\ts_nop 1" :: "v"(op + e * 4 * NL), "v"(o) : "memory");
            }
        }
        return;
    }

    const int gb = bid - GEMM0;
    const int xcd = gb & 7, sl = (gb >> 3) & 7, set = gb >> 6;
    const bool foreign = set > 0;
    const int rep = foreign ? xcd + 8 * (set - 1) : xcd;
    const unsigned c0 = __hip_atomic_load(cnt, __ATOMIC_RELAXED, __HIP_MEMORY_SCOPE_AGENT);
    const float scale = p.sx[0] * p.sw[0];

    constexpr int PTILE = 16 * 272, PBUF = 8 * PTILE, PACK0 = 4 * PBUF, ARR0 = PACK0 + 8 * 768, BAD0 = ARR0 + 32;
    static_assert(ARR0 + 256 + 64 <= LDS_BYTES, "LDS");
    const unsigned l16 = lane * 16;
    const unsigned ast = lbase + wave * 4096;

    if (foreign) {
        const int c = gb - 64, sn = c >> 2, frag = (c & 3) * 16 + wave * 2;
        const char* xs = px + (size_t)sn * RBB + frag * 64;
        const char* wsrc = pw + (size_t)((sn & 3) * 1024 + frag * 16) * NL + (sn >> 2) * 64;
        const unsigned xo = (lane >> 3) * KDIM + (((lane & 7) ^ g) << 4), wo = (lane >> 2) * NL + (((lane & 3) ^ (g & 1)) << 4);
        const int q = (lane & 15) >> 1;
        const unsigned rx = ast + (lane & 15) * 128 + ((g ^ q) << 4);
        const unsigned rt = ast + 2048 + (16 * (g & 1) + q) * 64 + 8 * (lane & 1) + (((g >> 1) ^ (q >> 2) ^ ((g & 1) << 1)) << 4);
        u32x4 v0, v1; i32x2 t0, t1, t2, t3;
        unsigned keep;
        asm volatile("s_mov_b32 %6, m0\n\ts_mov_b32 m0, %13\n\ts_nop 4\n\t"
                     "global_load_lds_dwordx4 %7, %11\n\ts_add_u32 m0, m0, 1024\n\ts_nop 0\n\t"
                     "global_load_lds_dwordx4 %8, %11\n\ts_add_u32 m0, m0, 1024\n\ts_nop 0\n\t"
                     "global_load_lds_dwordx4 %9, %12\n\ts_add_u32 m0, m0, 1024\n\ts_nop 0\n\t"
                     "global_load_lds_dwordx4 %10, %12\n\ts_mov_b32 m0, %6\n\ts_waitcnt vmcnt(0)\n\t"
                     "ds_read_b128 %0, %14\n\tds_read_b128 %1, %15\n\t"
                     "ds_read_b64_tr_b8 %2, %16\n\tds_read_b64_tr_b8 %3, %16 offset:512\n\t"
                     "ds_read_b64_tr_b8 %4, %17\n\tds_read_b64_tr_b8 %5, %17 offset:512\n\ts_waitcnt lgkmcnt(0)"
                     : "=&v"(v0), "=&v"(v1), "=&v"(t0), "=&v"(t1), "=&v"(t2), "=&v"(t3), "=&s"(keep)
                     : "v"(xo), "v"((xo ^ 64) + 8 * KDIM), "v"(wo), "v"((wo ^ 32) + 16 * NL), "s"(xs), "s"(wsrc), "s"(ast),
                       "v"(rx), "v"(rx ^ 64), "v"(rt), "v"(rt ^ 32) : "memory", "scc");
        call = __builtin_amdgcn_readfirstlane(c0) + 1;
        const char* xw = myws + WS_X + (call & 1) * XPAR + frag * 1024;
        const i32x4 v2 = {t0[0], t0[1], t1[0], t1[1]}, v3 = {t2[0], t2[1], t3[0], t3[1]};
        asm volatile("global_store_dwordx4 %0, %5, %2 sc0 sc1\n\tglobal_store_dwordx4 %0, %6, %2 offset:1024 sc0 sc1\n\t"
                     "global_store_dwordx4 %0, %3, %1 sc0 sc1\n\tglobal_store_dwordx4 %0, %4, %1 offset:1024 sc0 sc1\n\t"
                     "s_waitcnt vmcnt(0)\n\ts_barrier"
                     :: "v"(l16), "s"(xw + (size_t)(rank * NRB + sn) * RBB), "s"(xw + (size_t)(NR * NRB + 32 + sn) * RBB),
                        "v"(v0), "v"(v1), "v"(v2), "v"(v3) : "memory");
        if (tid < 4) __hip_atomic_store((GLB_AS unsigned*)(myws + WS_FLAG + ((tid & 2 ? NR * NRB + 32 : rank * NRB) + sn) * 1024 + ((c & 3) * 2 + (tid & 1)) * 128),
                                        call, __ATOMIC_RELAXED, __HIP_MEMORY_SCOPE_SYSTEM);
    }
    call = __builtin_amdgcn_readfirstlane(c0) + 1;
    const char* xws = myws + WS_X + (call & 1) * XPAR;
    if (tid < 16) *(unsigned*)(smem + ARR0 + tid * 4) = 0;
    asm volatile("s_waitcnt lgkmcnt(0)\n\ts_barrier" ::: "memory");

    unsigned vinfo, vflag;
    int nunit;
    {
        const int pos = min(foreign ? tile_pos(rep, lane) : rep + 8 * (lane - (lane > 4) - (lane > XL)), 127);
        const int slot = !foreign && lane == 4 ? (int)((p.extra[xcd >> 2] >> (8 * (xcd & 3))) & 255)
                       : !foreign && lane == XL ? (other | (xcd & 3)) * NRB + ND - 1 : list_slot(rank, pos);
        const bool use = foreign ? lane < 8 && !held_by(slot, sl >> 1) : lane < 18 && (lane != 4 || xcd < NF) && (lane != XL || xcd >= 4);
        const unsigned mask = (unsigned)__ballot(use);
        const int pr = pos < YROT ? (pos & ~7) | ((pos + (sl & 6)) & 7) : pos;
        const int c4 = 4 * pr - min(max(pr - ND, 0), NF) - min(max(pr - NRB - 3 * D1, 0), 3 * NF) + max(pr - (125 - NRAW), 0) * RAWW / 4 - (pr >= ND ? 4 : 0);
        const int due = ((9 + NF) * ROUND) / 10 + ((c4 + 2 * (sl & 1)) * YSLOT) / 200;
        nunit = __builtin_popcount(mask);
        vinfo = __builtin_amdgcn_ds_permute((use ? __builtin_amdgcn_mbcnt_lo(mask, 0) : 63) * 4, slot | (lane & 7) << 8 | (pos >= 125 - NRAW) << 11 | due << 12);
        vflag = lane < nunit ? (unsigned)WS_FLAG + wave * 128 + (vinfo & 255) * 1024 : (unsigned)WS_ZERO;
    }
    const unsigned vaoff = (vinfo & 255) << 16;
    char* ydst = lk.ws + WS_Y + (call & 1) * YPAR + (size_t)(me * 32 + (sl & 1) * 16 + rep) * YT * YTILE + (wave & 3) * YPART;
    char* yflag = lk.ws + WS_YFLAG + (me * 32 + (sl & 1) * 16 + rep) * 128;
    float* obase = (foreign ? p.pout[sl >> 1] + me * 128 + (sl & 1) * 64 : p.out + sl * 64) + (wave & 3) * 4 * NL;

    int hold = -1, hdue = 0;
    unsigned holdv = 0;
    u32x4 o = {0, 0, 0, 0};
    char* op = ydst;
    unsigned ooff = l16;
    int u = 0;
    int pend = -1;
    auto emit = [&]() {
        asm volatile("s_mov_b32 exec_hi, %3\n\tglobal_store_dwordx4 %0, %1, %2 sc0 sc1\n\ts_mov_b32 exec_hi, -1\n\ts_nop 0"
                     :: "v"(ooff), "v"(o), "s"(op), "s"(holdv > 1 ? ~0u : (1u << (YPART / 16 - 32)) - 1) : "memory");
        pend = hold; hold = -1;
    };
    auto announce = [&]() {
        const unsigned now = lds_add(lbase + ARR0 + pend * 4, holdv) + holdv;
        if ((now & 15) == 4) put_word(yflag + pend * 4, call * 16 + (now >> 4));
        pend = -1;
    };
    auto publish = [&]() { WAITV(0); announce(); };

    i32x4 a[8], bf[8][4];
    wait_flag(myws + __builtin_amdgcn_readlane(vflag, 0), call);
    {
        const __amdgpu_buffer_rsrc_t rs = __builtin_amdgcn_make_buffer_rsrc((void*)(xws + __builtin_amdgcn_readlane(vaoff, 0) + wave * 8192), 0, 8192, 0x00020000);
#pragma unroll
        for (int s = 0; s < 8; ++s) a[s] = __builtin_bit_cast(i32x4, __builtin_amdgcn_raw_buffer_load_b128(rs, l16, s * 1024, 17));
    }
    {
        const int wslot = NR * NRB + (foreign ? 0 : 32) + sl * 4 + (wave >> 1);
        if (foreign) snooze(((2 + (sl & 1) * 4 + (wave >> 1)) * ROUND) / 10);
        wait_flags4(myws + WS_FLAG + wslot * 1024 + (wave & 1) * 512);
        asm volatile("" ::: "memory");
        const __amdgpu_buffer_rsrc_t rs = __builtin_amdgcn_make_buffer_rsrc((void*)(xws + (size_t)wslot * RBB + (wave & 1) * 32768), 0, 32768, 0x00020000);
#pragma unroll
        for (int s = 0; s < 8; ++s)
#pragma unroll
            for (int t = 0; t < 4; ++t)
                bf[s][t] = __builtin_bit_cast(i32x4, __builtin_amdgcn_raw_buffer_load_b128(rs, (lane >> 5) * 2048 + (lane & 31) * 16 + t * 512, s * 4096, 17));
#pragma unroll
        for (int s = 0; s < 8; ++s) asm volatile("" : "+v"(bf[s][0]), "+v"(bf[s][1]), "+v"(bf[s][2]), "+v"(bf[s][3]), "+v"(a[s]));
    }
    if (foreign) {
        snooze(((2 * HINTR + 3) * ROUND) / 10);
        const unsigned long long h = __hip_atomic_load((GLB_AS unsigned long long*)(myws + WS_HINT) + (3 + (sl >> 1)) * 64 + lane, __ATOMIC_RELAXED, __HIP_MEMORY_SCOPE_AGENT);
        int d = (unsigned)(h >> 32) == call ? (int)((unsigned)h - (unsigned)t0) : LEAD;
#pragma unroll
        for (int o = 32; o; o >>= 1) d += __shfl_xor(d, o);
        d >>= 6;
        ts = (unsigned)t0 + __builtin_amdgcn_readfirstlane(min(max(d, 0), 1000)) + YLEAD;
    }
    unsigned fv = 0, fa = (unsigned)WS_TMO;
    u32x4 ars;
    ars[0] = (unsigned)(size_t)xws; ars[1] = (unsigned)((size_t)xws >> 32) & 0xffff; ars[2] = (unsigned)XPAR; ars[3] = 0x00020000;
    const unsigned alo = wave * 8192 + l16, ahi = alo + 4096;
    const unsigned pw0 = lbase + wave * PTILE + (lane & 15) * 272 + g * 16;
    const unsigned pr0 = lbase + ((wave & 3) * 4 + g) * 272 + (lane & 15) * 16;
    auto poison_all = [&]() { asm volatile("ds_write_b32 %0, %1" :: "v"(lbase + BAD0), "v"(1) : "memory"); };
    constexpr int RUN = 3, LAST = 9, FETCH = 5;
    int md = FETCH;
    if (bad) poison_all();
    auto peek = [&](const char* f) -> unsigned {
        unsigned v;
        asm volatile("s_mov_b64 exec, 1\n\tglobal_load_dword %0, %1, %2 sc0 sc1\n\ts_mov_b64 exec, -1\n\ts_waitcnt vmcnt(0)"
                     : "=&v"(v) : "v"(0u), "s"(f) : "memory");
        return __builtin_amdgcn_readfirstlane(v);
    };
    bool paired = false;
#pragma unroll 1
    while (u < nunit) {
        asm volatile("s_waitcnt vmcnt(7)" : "+v"(a[0]), "+v"(fv) :: "memory");
        const int un = u + (md != LAST);
        const unsigned soff = __builtin_amdgcn_readlane(vaoff, un), fnext = __builtin_amdgcn_readlane(vflag, un + 1);
        if (__builtin_expect(md != LAST, 1)) {
            unsigned nx = __builtin_amdgcn_readlane(fv, 0);
            if (nx < call && u + 1 < nunit) nx = peek(myws + __builtin_amdgcn_readlane(vflag, u + 1));
            md = nx >= call ? RUN : LAST;
            if (__builtin_expect(__builtin_amdgcn_readlane(fv, 1) == call, 0)) poison_all();
        } else {
            const char* f = myws + __builtin_amdgcn_readlane(vflag, u);
            bool seen = false;
            while ((hold & pend) >= 0) {
                seen = peek(f) >= call;
                if (seen) break;
                if (pend >= 0) announce();
                else if ((int)((unsigned)__builtin_amdgcn_s_memrealtime() - ts) >= hdue) emit();
            }
            if (!seen && wait_flag(f, call) == 0) poison_all();
            md = FETCH;
        }
        asm volatile("s_nop 1\n\tv_writelane_b32 %1, %3, 0\n\tglobal_load_dword %0, %1, %2 sc0 sc1"
                     : "+v"(fv), "+v"(fa) : "s"(myws), "s"(fnext), "s"(soff) : "memory");
        i32x4 acc[4];
        asm volatile("s_cmp_lg_u32 %28, 3\n\ts_cbranch_scc1 1f\n\t"
                     MM0 LD(4, 0) W(8) MM(13, 14, 15, 16, 5) LD(5, 1024) W(8) MM(17, 18, 19, 20, 6) LD(6, 2048) W(8) MM(21, 22, 23, 24, 7) LD(7, 3072) W(8)
                     "s_branch 3f\n1:\n\ts_cmp_lg_u32 %28, 9\n\ts_cbranch_scc1 2f\n\t"
                     MM0 W(7) MM(13, 14, 15, 16, 5) W(6) MM(17, 18, 19, 20, 6) W(5) MM(21, 22, 23, 24, 7) W(4)
                     "s_branch 3f\n2:\n\t" LD(4, 0) LD(5, 1024) LD(6, 2048) LD(7, 3072) "\n3:"
                     : "=&v"(acc[0]), "=&v"(acc[1]), "=&v"(acc[2]), "=&v"(acc[3]), "+v"(a[0]), "+v"(a[1]), "+v"(a[2]), "+v"(a[3]), "+v"(a[4])
                     : BF4(0), BF4(1), BF4(2), BF4(3), "v"(alo), "s"(ars), "s"(soff), "s"(md) : "memory", "scc");
        bool mid = false;
        if (hold >= 0 && (int)((unsigned)__builtin_amdgcn_s_memrealtime() - ts) + MIDLEAD >= hdue) { emit(); mid = true; }
        asm volatile("s_cmp_lg_u32 %28, 3\n\ts_cbranch_scc1 1f\n\t"
                     MM(9, 10, 11, 12, 4) LD(4, 0) W(8) MM(13, 14, 15, 16, 5) LD(5, 1024) W(8) MM(17, 18, 19, 20, 6) LD(6, 2048) W(8) MM(21, 22, 23, 24, 7) LD(7, 3072)
                     "s_branch 3f\n1:\n\ts_cmp_lg_u32 %28, 9\n\ts_cbranch_scc1 2f\n\t"
                     MM(9, 10, 11, 12, 4) W(3) MM(13, 14, 15, 16, 5) W(2) MM(17, 18, 19, 20, 6) W(1) MM(21, 22, 23, 24, 7)
                     "s_branch 3f\n2:\n\t" LD(4, 0) LD(5, 1024) LD(6, 2048) LD(7, 3072) "\n3:\n\ts_nop 7"
                     : "+v"(acc[0]), "+v"(acc[1]), "+v"(acc[2]), "+v"(acc[3]), "+v"(a[4]), "+v"(a[5]), "+v"(a[6]), "+v"(a[7]), "+v"(a[0])
                     : BF4(4), BF4(5), BF4(6), BF4(7), "v"(ahi), "s"(ars), "s"(soff), "s"(md) : "memory", "scc");
        if (md == FETCH) { if (pend >= 0 && !mid) announce(); continue; }
        const unsigned pb = (u & 3) * PBUF;
        asm volatile("ds_write_b128 %0, %1\n\tds_write_b128 %0, %2 offset:64\n\tds_write_b128 %0, %3 offset:128\n\tds_write_b128 %0, %4 offset:192"
                     :: "v"(pw0 + pb), "v"(acc[0]), "v"(acc[1]), "v"(acc[2]), "v"(acc[3]) : "memory");
        ++u;
        if (!foreign && (u & 1) && u < nunit - NSGL) { paired = true; continue; }
        const bool mine = (wave >> 2) != (u & 1);
        const bool after = pend >= 0 && !mine && !mid;
        if (pend >= 0 && mine) publish();
        if (hold >= 0) { pace(hdue); emit(); }
        asm volatile("s_waitcnt lgkmcnt(0)\n\ts_barrier" ::: "memory");
        if (after) publish();
        if (mine || paired) {
            const unsigned info = __builtin_amdgcn_readlane(vinfo, mine ? u - 1 : u - 2);
            i32x4 r[8];
            int poison;
            unsigned long long now;
            asm volatile("s_memrealtime %9\n\tds_read_b128 %0, %10\n\tds_read_b128 %1, %10 offset:%12\n\tds_read_b128 %2, %10 offset:2*%12\n\t"
                         "ds_read_b128 %3, %10 offset:3*%12\n\tds_read_b128 %4, %10 offset:4*%12\n\tds_read_b128 %5, %10 offset:5*%12\n\t"
                         "ds_read_b128 %6, %10 offset:6*%12\n\tds_read_b128 %7, %10 offset:7*%12\n\tds_read_b32 %8, %11\n\ts_waitcnt lgkmcnt(0)"
                         : "=&v"(r[0]), "=&v"(r[1]), "=&v"(r[2]), "=&v"(r[3]), "=&v"(r[4]), "=&v"(r[5]), "=&v"(r[6]), "=&v"(r[7]), "=&v"(poison), "=&s"(now)
                         : "v"(pr0 + (mine ? pb : pb - PBUF)), "v"(lbase + BAD0), "n"(PTILE) : "memory");
            const i32x4 sum = ((r[0] + r[1]) + (r[2] + r[3])) + ((r[4] + r[5]) + (r[6] + r[7]));
            const float sc = poison ? __builtin_nanf("") : scale;
            char* const orow = (char*)obase + (size_t)(info & 255) * (16 * NL * 4);
            const unsigned roff = g * (NL * 4) + (lane & 15) * 16;
#pragma unroll
            for (int e = 0; e < 4; ++e) o[e] = __builtin_bit_cast(unsigned, (float)sum[e] * sc);
            if (!foreign) {
                asm volatile("global_store_dwordx4 %0, %1, %2 sc0 sc1\n\ts_nop 1" :: "v"(roff), "v"(o), "s"(orow) : "memory");
            } else {
                hold = (info >> 8) & 7;
                hdue = info >> 12;
                op = orow; ooff = roff;
                holdv = 1 + (16u << (wave & 3));
                asm volatile("" : "+v"(o));
                const unsigned big = ((unsigned)sum[0] + 0x200000u) | ((unsigned)sum[1] + 0x200000u) | ((unsigned)sum[2] + 0x200000u) | ((unsigned)sum[3] + 0x200000u);
                if (!(info & 0x800) && !__any((big >> 22) | poison)) {
                    const unsigned s0 = sum[0], s1 = sum[1], s2 = sum[2], s3 = sum[3];
                    const unsigned hi = ((s0 >> 16) & 63) | ((s1 >> 10) & (63 << 6)) | ((s2 >> 4) & (63 << 12)) | ((s3 << 2) & (63 << 18));
                    const unsigned pk = lbase + PACK0 + wave * 768;
                    asm volatile("ds_write_b32 %1, %2\n\tds_write_b32 %1, %3 offset:256\n\tds_write_b16 %5, %4 offset:512\n\t"
                                 "ds_write_b8_d16_hi %6, %4 offset:640\n\tds_read_b128 %0, %7\n\ts_waitcnt lgkmcnt(0)"
                                 : "=&v"(o) : "v"(pk + lane * 4), "v"((s0 & 0xffffu) | (s1 << 16)), "v"((s2 & 0xffffu) | (s3 << 16)), "v"(hi),
                                   "v"(pk + lane * 2), "v"(pk + lane), "v"(pk + l16) : "memory");
                    op = ydst + hold * YTILE; ooff = l16;
                    holdv = 1;
                }
                const int left = hdue - (int)((unsigned)now - ts) - EPI;
                if (u >= nunit || left <= HOLDMIN) { if (left > 0) pace(hdue); emit(); }
            }
        }
        paired = false;
    }
    asm volatile("s_waitcnt vmcnt(0)" : "+v"(a[0]), "+v"(a[1]), "+v"(a[2]), "+v"(a[3]), "+v"(a[4]), "+v"(a[5]), "+v"(a[6]), "+v"(a[7]), "+v"(fv) :: "memory");
    if (pend >= 0) publish();
    if (tid == 0) __hip_atomic_store(cnt, call, __ATOMIC_RELAXED, __HIP_MEMORY_SCOPE_AGENT);
}

extern "C" void kernel_launch(int rank, int nranks, ncclComm_t comm,
                              hipStream_t stream,
                              void* const* d_in, const int* in_sizes, int n_in,
                              void* d_out, int out_size,
                              const int* in_dtypes, int out_dtype,
                              void* const* d_out_all,
                              void* const* d_ws_all, size_t ws_bytes) {
    if (nranks != NR || (unsigned)rank >= (unsigned)NR || n_in != 4 || ws_bytes < WS_NEED
        || in_sizes[0] != ML * KDIM || in_sizes[1] != KDIM * NL || out_size != NR * ML * NL) {
        fprintf(stderr, "ag_gemm: unexpected problem shape\n");
        return;
    }
    HIP_CHECK(hipFuncSetAttribute((const void*)ag_gemm, hipFuncAttributeMaxDynamicSharedMemorySize, LDS_BYTES));
    Params p;
    p.x = (const char*)d_in[0]; p.w = (const char*)d_in[1];
    p.sx = (const float*)d_in[2]; p.sw = (const float*)d_in[3];
    p.rank = rank; p.pad = 0;
    const int me = rank & 3, other = (rank & 4) ^ 4;
    p.myws = (char*)d_ws_all[rank]; p.out = (float*)d_out;
    for (int i = 0; i < NR - 1; ++i) {
        p.link[i].ws = (char*)d_ws_all[i < 3 ? (rank & 4) | ((me + 1 + i) & 3) : other | (i - 3)];
        p.link[i].fin = p.link[i].fout = p.link[i].pad[0] = p.link[i].pad[1] = p.link[i].pad[2] = 0;
    }
    for (int k = 0; k < 4; ++k) p.pout[k] = (float*)d_out_all[other | k];
    p.extra[0] = p.extra[1] = 0;
    int ne = 0;
    for (int f = 0; f < NF; ++f) {
        p.link[3 + ((me + offs(f)) & 3)].fin |= 1u << f;
        for (int m = 0; m < 4; ++m) {
            if (((m + offs(f)) & 3) != me) continue;
            for (int k = 0; k < 4; ++k) if (k != m) p.link[3 + k].fout |= 1u << (4 * f + m);
            p.extra[ne >> 2] |= (unsigned)((other | m) * NRB + ND + f) << (8 * (ne & 3));
            ++ne;
        }
    }
    for (int i = 0; i < NR - 1; ++i) {
        const unsigned fin = p.link[i].fin, rest = fin & (fin - 1);
        const unsigned s0 = rank * NRB + (i < 3 ? 0 : ND + __builtin_ctz(fin));
        const unsigned s1 = i < 3 ? rank * NRB + 1 : rest ? rank * NRB + ND + __builtin_ctz(rest) : NR * NRB + me * 8;
        p.link[i].first = s0 | s1 << 16;
    }
    hipLaunchKernelGGL(ag_gemm, dim3(GRID), dim3(512), LDS_BYTES, stream, p);
    HIP_CHECK(hipGetLastError());
}
```
